```python
import numpy as np

kernel_name = "gpu_ag_gemm_m4096_k4096_n4096_f32_gelu"


ABSMAX_RTOL = 0.001

COMPUTE_DTYPE = "float32"
EPILOGUE = "gelu"
IN_QUANT = None
OUT_QUANT = None
IN_DTYPES = {"x": "float32", "w_mat": "float32"}
WORLD_SIZE = 8

def setup_inputs():

    x = np.random.randn(4096, 4096).astype(np.float32)
    w_mat = (np.random.randn(4096, 4096) / np.sqrt(4096)).astype(np.float32)
    return {"x": x, "w_mat": w_mat}

def _epilogue(y):
    c = 0.7978845608028654
    return (0.5 * y * (1.0 + np.tanh(c * (y + 0.044715 * y ** 3)))).astype(np.float32)

def reference(x, w_mat):
    y = (x @ w_mat).astype(np.float32)
    return _epilogue(y)
```

```cpp
#include <hip/hip_runtime.h>
#include <rccl/rccl.h>
#include <cstdio>

#define HIP_CHECK(expr) do { hipError_t _e = (expr); if (_e != hipSuccess) { \
    fprintf(stderr, "HIP error %s at %s:%d\n", hipGetErrorString(_e), __FILE__, __LINE__); \
    return; } } while (0)

namespace {

typedef unsigned u32x4 __attribute__((ext_vector_type(4)));
typedef float f32x4 __attribute__((ext_vector_type(4)));
typedef float f32x16 __attribute__((ext_vector_type(16)));
typedef _Float16 h2 __attribute__((ext_vector_type(2)));
typedef _Float16 h8 __attribute__((ext_vector_type(8)));
typedef short s4 __attribute__((ext_vector_type(4)));
typedef short s8 __attribute__((ext_vector_type(8)));
typedef unsigned long long u64;

constexpr int NR = 8;
constexpr int KDIM = 4096;
constexpr int ROWS = 512;
constexpr int COLS = 512;
constexpr unsigned OBJ = 256u * 1024u;
constexpr int NG = 6;
constexpr int NA = 12;
constexpr size_t WS_CNT = 0;
constexpr size_t WS_TMO = 16 * 1024;
constexpr size_t WS_PROG = 32 * 1024;
constexpr size_t WS_DONE = 64 * 1024;
constexpr size_t WS_DIAG = 128 * 1024;
constexpr size_t WS_DATA = 1024 * 1024;
constexpr unsigned NSLOT = 160;
constexpr size_t PAR_BYTES = (size_t)NSLOT * OBJ;
constexpr size_t WS_NEED = WS_DATA + 2 * PAR_BYTES;
constexpr u64 DEADLINE = 100000000ull;
constexpr unsigned L_PART = 128 * 1024;
constexpr unsigned L_CTRL = L_PART + 16 * 1024;
constexpr unsigned L_TOTAL = L_CTRL + 64;

struct Params {
  const char* x;
  const char* w;
  char* ws[NR];
  char* out[NR];
  unsigned units[NG][4];
  int rank;
};

__device__ __forceinline__ unsigned rfl(unsigned v) { return __builtin_amdgcn_readfirstlane(v); }
__device__ __forceinline__ unsigned pk(float a, float b) {
  h2 v; v.x = (_Float16)a; v.y = (_Float16)b; return __builtin_bit_cast(unsigned, v);
}
__device__ __forceinline__ u32x4 cvt8(u32x4 a, u32x4 b) {
  f32x4 fa = __builtin_bit_cast(f32x4, a), fb = __builtin_bit_cast(f32x4, b);
  u32x4 r; r.x = pk(fa.x, fa.y); r.y = pk(fa.z, fa.w); r.z = pk(fb.x, fb.y); r.w = pk(fb.z, fb.w);
  return r;
}
__device__ __forceinline__ float gelu(float y) {
  const float c2 = -2.0f * 0.7978845608028654f * 1.4426950408889634f;
  float u = y + 0.044715f * y * y * y;
  float e = __builtin_amdgcn_exp2f(c2 * u);
  return y * __builtin_amdgcn_rcpf(1.0f + e);
}

#define STAMPV(idx, val) do {} while (0)
#define STAMP(idx) STAMPV(idx, __builtin_amdgcn_s_memrealtime())

__device__ __forceinline__ bool give_up(u64 t0, unsigned* tmo, unsigned call) {
  u64 el = __builtin_amdgcn_s_memrealtime() - t0;
  if (el < 100000ull) return false;
  if (el > DEADLINE ||
      rfl(__hip_atomic_load(tmo, __ATOMIC_RELAXED, __HIP_MEMORY_SCOPE_AGENT)) == call) {
    __hip_atomic_store(tmo, call, __ATOMIC_RELAXED, __HIP_MEMORY_SCOPE_AGENT);
    return true;
  }
  return false;
}

__device__ __forceinline__ void wait_prog(const u64* p, unsigned target, unsigned& seen, bool& bad,
                                          u64 t0, unsigned* tmo, unsigned call) {
  while (seen < target) {
    u64 v = __hip_atomic_load(p, __ATOMIC_RELAXED, __HIP_MEMORY_SCOPE_SYSTEM);
    unsigned lo = rfl((unsigned)v), hi = rfl((unsigned)(v >> 32));
    seen = lo < hi ? lo : hi;
    if (seen >= target) break;
    if (give_up(t0, tmo, call)) { bad = true; break; }
    __builtin_amdgcn_s_sleep(2);
  }
  asm volatile("" ::: "memory");
}

struct Piece { u32x4 r[16]; };

__device__ __forceinline__ void piece_load(Piece& s, const char* sb, unsigned fstride, unsigned voff) {
#pragma unroll
  for (int u = 0; u < 8; ++u) {
    const char* p = sb + (size_t)u * fstride;
    asm volatile("global_load_dwordx4 %0, %1, %2" : "=v"(s.r[2 * u]) : "v"(voff), "s"(p) : "memory");
    asm volatile("global_load_dwordx4 %0, %1, %2 offset:16" : "=v"(s.r[2 * u + 1]) : "v"(voff), "s"(p) : "memory");
  }
}
template <int N>
__device__ __forceinline__ void piece_wait(Piece& s) {
  asm volatile("s_waitcnt vmcnt(%0)" :: "i"(N) : "memory");
#pragma unroll
  for (int u = 0; u < 16; ++u) asm volatile("" : "+v"(s.r[u]));
}
template <int U>
__device__ __forceinline__ void piece_store1(Piece& s, char* db, unsigned voff) {
  u32x4 v = cvt8(s.r[2 * U], s.r[2 * U + 1]);
  asm volatile("global_store_dwordx4 %0, %1, %2 offset:%3 sc0 sc1\n\ts_nop 1"
               :: "v"(voff), "v"(v), "s"(db), "i"((U - 4) * 1024) : "memory");
}
__device__ __forceinline__ void piece_store(Piece& s, char* db  , unsigned voff) {
  piece_store1<0>(s, db, voff); piece_store1<1>(s, db, voff); piece_store1<2>(s, db, voff);
  piece_store1<3>(s, db, voff); piece_store1<4>(s, db, voff); piece_store1<5>(s, db, voff);
  piece_store1<6>(s, db, voff); piece_store1<7>(s, db, voff);
}

typedef __attribute__((address_space(3))) char lds_char;

__device__ __forceinline__ void glds16(const char* src, unsigned voff, unsigned lds) {
  unsigned keep;
  asm volatile("s_mov_b32 %0, m0\n\ts_mov_b32 m0, %3\n\ts_nop 0\n\t"
               "global_load_lds_dwordx4 %1, %2 sc0 sc1\n\ts_mov_b32 m0, %0"
               : "=&s"(keep) : "v"(voff), "s"(src), "s"(lds) : "memory");
}

__device__ __forceinline__ h8 tr_frag(lds_char* p) {
  s4 a = __builtin_amdgcn_ds_read_tr16_b64_v4i16((__attribute__((address_space(3))) s4*)p);
  s4 c = __builtin_amdgcn_ds_read_tr16_b64_v4i16((__attribute__((address_space(3))) s4*)(p + 256));
  s8 r = __builtin_shufflevector(a, c, 0, 1, 2, 3, 4, 5, 6, 7);
  return __builtin_bit_cast(h8, r);
}

__global__ void __launch_bounds__(384) ag_gemm_gelu(Params P) {
  __shared__ __attribute__((aligned(1024))) char smem[L_TOTAL];
  const int lane = threadIdx.x & 63;
  const int wave = rfl(threadIdx.x >> 6);
  const int b = blockIdx.x;
  const int rank = P.rank & (NR - 1);
  const int side = rank >> 2, q = rank & 3;
  char* const myws = P.ws[rank];
  unsigned* const cntp = (unsigned*)(myws + WS_CNT + b * 32 + wave * 4);
  unsigned* const lcnt = (unsigned*)(smem + L_CTRL);
  unsigned* const lfree = (unsigned*)(smem + L_CTRL + 4);

  if (wave == 0) {
    if (b >= 64 + 128 || (b >> 3) == 7) return;
    __builtin_amdgcn_s_setprio(3);
    const int j = b & 7, c = b >> 3;
    int dest, nW, strip0, wslot0, xu0, nobj, ch;
    if (c < 3)      { dest = side * 4 + ((q + 1 + c) & 3); nW = 0; strip0 = 0; wslot0 = 0; xu0 = 0; nobj = NA; ch = rank; }
    else if (c < 7) { dest = (side ^ 1) * 4 + ((q + c - 3) & 3); nW = 4; strip0 = (dest & 3) * 4; wslot0 = 16 + q * 4;
                      xu0 = NA; nobj = 4 + 16 - NA; ch = rank; }
    else            { dest = rank; nW = 1; strip0 = c - 8; wslot0 = c - 8; xu0 = c - 8; nobj = 2; ch = c; }
    dest &= NR - 1;
    const int nit = nobj * 4;
    const unsigned laneX = (lane & 31) * (KDIM * 4) + (lane >> 5) * 32;
    const unsigned laneW = (lane >> 2) * (COLS * 4) + (lane & 3) * 32;
    const unsigned lane16 = lane * 16;
    const char* px = P.x;
    const char* pw = P.w;
    asm volatile("" : "+s"(px), "+s"(pw));
    auto load = [&](Piece& s, int it) {
      it = it < nit ? it : nit - 1;
      const int obj = it >> 2, f0 = (it & 3) * 64 + j * 8;
      const bool isW = obj < nW;
      const char* sb = isW ? pw + (size_t)(strip0 + obj) * 128 + (size_t)f0 * (16 * COLS * 4)
                           : px + (size_t)(xu0 + obj - nW) * (32 * KDIM * 4) + (size_t)f0 * 64;
      piece_load(s, sb, isW ? 16 * COLS * 4 : 64, isW ? laneW : laneX);
    };
    unsigned cnt;
    asm volatile("global_load_dword %0, %1, off sc1" : "=v"(cnt) : "v"(cntp) : "memory");
    Piece A, B;
    load(A, 0);
    load(B, 1);
    __builtin_amdgcn_s_barrier();
    piece_wait<16>(A);
    asm volatile("" : "+v"(cnt));
    const unsigned call = rfl(cnt) + 1;
    char* const dws = P.ws[dest];
    char* const ddata = dws + WS_DATA + (size_t)(call & 1) * PAR_BYTES + 4096 + j * 8192;
    unsigned* const fp = (unsigned*)(dws + WS_PROG + (ch * 4 + (j >> 1)) * 128 + (j & 1) * 4);
    auto store = [&](Piece& s, int it) {
      const int obj = it >> 2;
      const unsigned slot = obj < nW ? 128 + wslot0 + obj : rank * 16 + xu0 + obj - nW;
      piece_store(s, ddata + (size_t)slot * OBJ + (it & 3) * 65536, lane16);
    };
    auto flag = [&](int it) {
      unsigned v = call * 64 + it + 1;
      if (lane == 0) asm volatile("global_store_dword %0, %1, off sc0 sc1" :: "v"(fp), "v"(v) : "memory");
    };
    STAMP(0);
    store(A, 0);
    load(A, 2);
    piece_wait<24>(B);
    if (lane == 0) asm volatile("global_store_dword %0, %1, off sc1" :: "v"(cntp), "v"(call) : "memory");
    store(B, 1);
    load(B, 3);
    for (int it = 2; it < nit; it += 2) {
      piece_wait<25>(A);
      flag(it - 2);
      store(A, it);
      load(A, it + 2);
      piece_wait<25>(B);
      flag(it - 1);
      store(B, it + 1);
      load(B, it + 3);
    }
    piece_wait<25>(A);
    flag(nit - 2);
    piece_wait<0>(B);
    flag(nit - 1);
    STAMP(1);
    return;
  }

  if (b < 64) return;
  const u64 t0 = __builtin_amdgcn_s_memrealtime();
  const unsigned call = rfl(__hip_atomic_load(cntp, __ATOMIC_RELAXED, __HIP_MEMORY_SCOPE_AGENT)) + 1;
  if (lane == 0) __hip_atomic_store(cntp, call, __ATOMIC_RELAXED, __HIP_MEMORY_SCOPE_AGENT);
  unsigned* const tmo = (unsigned*)(myws + WS_TMO);
  const int nq = (b & 3) * 24 + (b >> 3) - 8;
  const int g = ((b >> 2) & 1) * 3 + (nq >> 5), s = nq & 31;
  const bool foreign = s >= 16;
  const int owner = (foreign ? (side ^ 1) * 4 + ((s - 16) >> 2) : rank) & (NR - 1);
  const unsigned ul0 = P.units[g][0], ul1 = P.units[g][1], ul2 = P.units[g][2];
  auto unit_at = [&](int k) -> unsigned {
    unsigned wv = k < 4 ? ul0 : (k < 8 ? ul1 : ul2);
    return (wv >> ((k & 3) * 8)) & 0xFFu;
  };
  auto skip = [&](unsigned e) -> bool {
    return foreign && (e & 15) >= NA && ((int)(e >> 4) == rank || (int)(e >> 4) == owner);
  };
  bool bad = false;

  if (wave == 1) {
    STAMPV(5, t0);
    if (lane == 0) { *lcnt = 0; *lfree = 0; }
    asm volatile("s_waitcnt lgkmcnt(0)" ::: "memory");
    __builtin_amdgcn_s_barrier();
    const int col0 = foreign ? q * 128 + (s & 3) * 32 : s * 32;
    char* const obase = P.out[owner] + col0 * 4;
    const int rr = lane >> 3, chn = lane & 7;
    const unsigned ovoff = rr * (COLS * 4) + chn * 16;
    const char* const pl = smem + L_PART + rr * 128 + ((chn ^ rr) * 16);
    unsigned done = 0;
    char* lastp = obase;
    for (int k = 0; k < 12; ++k) {
      const unsigned e = unit_at(k);
      if (e == 0xFFu) break;
      if (skip(e)) continue;
      ++done;
      while (rfl(__hip_atomic_load(lcnt, __ATOMIC_RELAXED, __HIP_MEMORY_SCOPE_WORKGROUP)) < done * 4) {
        if (__builtin_amdgcn_s_memrealtime() - t0 > 2 * DEADLINE) { bad = true; break; }
        __builtin_amdgcn_s_sleep(1);
      }
      asm volatile("" ::: "memory");
      f32x4 y[4];
#pragma unroll
      for (int it = 0; it < 4; ++it) {
        f32x4 p0 = *(const f32x4*)(pl + it * 1024);
        f32x4 p1 = *(const f32x4*)(pl + it * 1024 + 4096);
        f32x4 p2 = *(const f32x4*)(pl + it * 1024 + 8192);
        f32x4 p3 = *(const f32x4*)(pl + it * 1024 + 12288);
        y[it] = (p0 + p1) + (p2 + p3);
      }
#pragma unroll
      for (int it = 0; it < 4; ++it) asm volatile("" : "+v"(y[it]));
      asm volatile("s_waitcnt lgkmcnt(0)" ::: "memory");
      if (lane == 0) __hip_atomic_store(lfree, done, __ATOMIC_RELAXED, __HIP_MEMORY_SCOPE_WORKGROUP);
      char* const op = obase + (size_t)(e & 127) * (32 * COLS * 4);
      lastp = op;
#pragma unroll
      for (int it = 0; it < 4; ++it) {
        f32x4 r;
        r.x = gelu(y[it].x); r.y = gelu(y[it].y); r.z = gelu(y[it].z); r.w = gelu(y[it].w);
        if (bad) r.x = __builtin_nanf("");
        char* o2 = op + it * (8 * COLS * 4);
        asm volatile("global_store_dwordx4 %0, %1, %2 sc0 sc1\n\ts_nop 1"
                     :: "v"(ovoff), "v"(__builtin_bit_cast(u32x4, r)), "s"(o2) : "memory");
      }
      STAMP(32 + k);
    }
    asm volatile("s_waitcnt vmcnt(0)" ::: "memory");
    STAMP(2);
    if (foreign) {
      unsigned* dp = (unsigned*)(P.ws[owner] + WS_DONE + q * 128 + ((s & 3) * 8 + g) * 4);
      if (lane == 0) __hip_atomic_store(dp, call, __ATOMIC_RELAXED, __HIP_MEMORY_SCOPE_SYSTEM);
    } else if (b == 64) {
      const u64* dp = (const u64*)(myws + WS_DONE + lane * 8);
      while (true) {
        u64 v = __hip_atomic_load(dp, __ATOMIC_RELAXED, __HIP_MEMORY_SCOPE_SYSTEM);
        bool ok = ((unsigned)v >= call && (unsigned)(v >> 32) >= call) || (lane & 3) == 3;
        if (__all(ok)) break;
        if (give_up(t0, tmo, call)) {
          if (lane == 0) __hip_atomic_store((float*)lastp, __builtin_nanf(""), __ATOMIC_RELAXED, __HIP_MEMORY_SCOPE_SYSTEM);
          break;
        }
        __builtin_amdgcn_s_sleep(1);
      }
      STAMP(3);
    }
    return;
  }

  __builtin_amdgcn_s_barrier();
  const int w = wave - 2;
  char* const data = myws + WS_DATA + (size_t)(call & 1) * PAR_BYTES;
  const unsigned base = call * 64;
  lds_char* const lw = (lds_char*)smem + w * 32768;
  lds_char* const ltr = lw + ((lane >> 5) * 8 + ((lane & 15) >> 2)) * 64 + ((lane >> 4) & 1) * 32 + (lane & 3) * 8;
  h8 wr[32];
  {
    const int ch = foreign ? owner : 8 + s;
    const unsigned pos = foreign ? (s & 3) : 0;
    const u64* pp = (const u64*)(myws + WS_PROG + (ch * 4 + w) * 128);
    const char* src = data + (size_t)(128 + s) * OBJ + w * 16384;
    const unsigned svoff = lane * 16;
    const unsigned lwa = (unsigned)(size_t)lw;
    unsigned seen = 0;
#pragma unroll
    for (int r = 0; r < 4; ++r) {
      wait_prog(pp, base + pos * 4 + r + 1, seen, bad, t0, tmo, call);
#pragma unroll
      for (int i = 0; i < 16; ++i)
        glds16(src + r * 65536 + i * 1024, svoff, lwa + (r & 1) * 16384 + i * 1024);
      if (r & 1) asm volatile("s_waitcnt vmcnt(0)" ::: "memory");
      if (r == 1) {
#pragma unroll
        for (int i = 0; i < 32; ++i) wr[i] = tr_frag(ltr + i * 1024);
#pragma unroll
        for (int i = 0; i < 32; ++i) asm volatile("" : "+v"(wr[i]));
        asm volatile("s_waitcnt lgkmcnt(0)" ::: "memory");
      }
    }
  }
  if (w == 0) STAMP(4);
  const __amdgpu_buffer_rsrc_t rsrc =
      __builtin_amdgcn_make_buffer_rsrc(data, 0, 128 * OBJ, 0x00020000);
  const unsigned xvoff = w * 16384 + lane * 16;
  char* const pw = smem + L_PART + w * 4096 + (lane & 31) * 128;
  unsigned done = 0;
  for (int k = 0; k < 12; ++k) {
    const unsigned e = unit_at(k);
    if (e == 0xFFu) break;
    if (skip(e)) continue;
    const int src = e >> 4, i0 = e & 15;
    const int ch = src == rank ? 8 + i0 : src;
    const unsigned pos = src == rank ? 1 : (i0 < NA ? i0 : i0 - NA + 4);
    const u64* pp = (const u64*)(myws + WS_PROG + (ch * 4 + w) * 128);
    const unsigned soff = (e & 127) * OBJ;
    unsigned seen = 0;
    f32x16 acc = {0.f, 0.f, 0.f, 0.f, 0.f, 0.f, 0.f, 0.f, 0.f, 0.f, 0.f, 0.f, 0.f, 0.f, 0.f, 0.f};
#pragma unroll
    for (int r = 0; r < 4; ++r) {
      wait_prog(pp, base + pos * 4 + r + 1, seen, bad, t0, tmo, call);
      u32x4 xa[16];
#pragma unroll
      for (int i = 0; i < 16; ++i)
        xa[i] = __builtin_amdgcn_raw_buffer_load_b128(rsrc, xvoff + (i & 3) * 1024, soff + r * 65536 + (i >> 2) * 4096, 17);
      __builtin_amdgcn_sched_barrier(0);
#pragma unroll
      for (int i = 0; i < 16; ++i) {
        h8 wf = r < 2 ? wr[r * 16 + i] : tr_frag(ltr + ((r - 2) * 16 + i) * 1024);
        acc = __builtin_amdgcn_mfma_f32_32x32x16_f16(wf, __builtin_bit_cast(h8, xa[i]), acc, 0, 0, 0);
      }
    }
    if (bad) {
#pragma unroll
      for (int i = 0; i < 16; ++i) acc[i] = __builtin_nanf("");
    }
    while (rfl(__hip_atomic_load(lfree, __ATOMIC_RELAXED, __HIP_MEMORY_SCOPE_WORKGROUP)) < done) {
      if (__builtin_amdgcn_s_memrealtime() - t0 > 2 * DEADLINE) break;
      __builtin_amdgcn_s_sleep(1);
    }
    asm volatile("" ::: "memory");
#pragma unroll
    for (int gq = 0; gq < 4; ++gq) {
      f32x4 v = {acc[gq * 4], acc[gq * 4 + 1], acc[gq * 4 + 2], acc[gq * 4 + 3]};
      *(f32x4*)(pw + (((gq * 2 + (lane >> 5)) ^ (lane & 7)) * 16)) = v;
    }
    asm volatile("s_waitcnt lgkmcnt(0)" ::: "memory");
    if (lane == 0) __hip_atomic_fetch_add(lcnt, 1u, __ATOMIC_RELAXED, __HIP_MEMORY_SCOPE_WORKGROUP);
    ++done;
    if (w == 0) STAMP(16 + k);
  }
}

}

extern "C" void kernel_launch(int rank, int nranks, ncclComm_t comm,
                              hipStream_t stream,
                              void* const* d_in, const int* in_sizes, int n_in,
                              void* d_out, int out_size,
                              const int* in_dtypes, int out_dtype,
                              void* const* d_out_all,
                              void* const* d_ws_all, size_t ws_bytes) {
  (void)comm; (void)in_dtypes; (void)out_dtype; (void)d_out;
  if (nranks != NR || (unsigned)rank >= (unsigned)NR || n_in < 2 || ws_bytes < WS_NEED ||
      in_sizes[0] != ROWS * KDIM || in_sizes[1] != KDIM * COLS || out_size != NR * ROWS * COLS) {
    fprintf(stderr, "kernel_launch: unexpected problem shape\n");
    return;
  }
  Params P;
  P.x = (const char*)d_in[0];
  P.w = (const char*)d_in[1];
  for (int p = 0; p < NR; ++p) { P.ws[p] = (char*)d_ws_all[p]; P.out[p] = (char*)d_out_all[p]; }
  P.rank = rank;
  const int side = rank >> 2, q = rank & 3;
  static const unsigned char order[NG][12] = {
    {60, 61, 0, 6, 12, 40, 46, 18, 24, 52, 30, 99},
    {62, 63, 1, 7, 13, 41, 47, 19, 25, 53, 31, 99},
    {64, 65, 72, 2, 8, 14, 42, 20, 48, 26, 54, 32},
    {66, 67, 73, 3, 9, 43, 15, 21, 49, 55, 27, 33},
    {68, 69, 74, 4, 10, 16, 44, 22, 50, 28, 34, 99},
    {70, 71, 75, 5, 11, 17, 45, 23, 51, 29, 35, 99}};
  for (int g = 0; g < NG; ++g) {
    unsigned char u[16];
    for (int k = 0; k < 16; ++k) u[k] = 0xFF;
    for (int k = 0; k < 12; ++k) {
      int v = order[g][k];
      if (v == 99) break;
      if (v < 40) u[k] = (side * 4 + ((q + 1 + v % 3) & 3)) * 16 + v / 3;
      else if (v < 60) u[k] = ((side ^ 1) * 4 + ((v - 40) & 3)) * 16 + NA + ((v - 40) >> 2);
      else u[k] = rank * 16 + v - 60;
    }
    for (int k = 0; k < 4; ++k)
      P.units[g][k] = u[4 * k] | (u[4 * k + 1] << 8) | (u[4 * k + 2] << 16) | ((unsigned)u[4 * k + 3] << 24);
  }
  hipLaunchKernelGGL(ag_gemm_gelu, dim3(256), dim3(384), 0, stream, P);
  HIP_CHECK(hipGetLastError());
}
```

```cpp
#include <hip/hip_runtime.h>
#include <rccl/rccl.h>
#include <cstdio>

#define HIP_CHECK(expr) do { hipError_t _e = (expr); if (_e != hipSuccess) { \
    fprintf(stderr, "HIP error %s at %s:%d\n", hipGetErrorString(_e), __FILE__, __LINE__); \
    return; } } while (0)

namespace {

typedef unsigned u32x4 __attribute__((ext_vector_type(4)));
typedef float f32x4 __attribute__((ext_vector_type(4)));
typedef float f32x16 __attribute__((ext_vector_type(16)));
typedef _Float16 h2 __attribute__((ext_vector_type(2)));
typedef _Float16 h8 __attribute__((ext_vector_type(8)));
typedef short s4 __attribute__((ext_vector_type(4)));
typedef short s8 __attribute__((ext_vector_type(8)));
typedef unsigned long long u64;

constexpr int NR = 8;
constexpr int KDIM = 4096;
constexpr int ROWS = 512;
constexpr int COLS = 512;
constexpr unsigned OBJ = 256u * 1024u;
constexpr int NG = 6;
constexpr int NA = 12;
constexpr size_t WS_CNT = 0;
constexpr size_t WS_TMO = 16 * 1024;
constexpr size_t WS_TICK = 20 * 1024;
constexpr size_t WS_FLAG = 32 * 1024;
constexpr size_t WS_DONE = 64 * 1024;
constexpr size_t WS_STGF = 80 * 1024;
constexpr size_t WS_DIAG = 128 * 1024;
constexpr size_t WS_DATA = 1024 * 1024;
constexpr unsigned NSLOT = 160;
constexpr size_t PAR_BYTES = (size_t)NSLOT * OBJ;
constexpr size_t WS_STG = WS_DATA + 2 * PAR_BYTES;
constexpr size_t WS_NEED = WS_STG + 128 * 12 * 4096;
constexpr u64 DEADLINE = 100000000ull;
#define FWD_T0 2500
#define SMODE 1
__host__ __device__ constexpr int lent_strip(int o, int h, int i) {
  return SMODE == 0 ? h * 4 + i : (SMODE == 1 ? ((o + h) & 3) * 4 + i : i * 4 + ((o + h) & 3));
}
#define FW 4
#define KA 4
#define KC 4
#define FWD_QUICK 0
#define KEEP 4
constexpr unsigned L_PART = 128 * 1024;
constexpr unsigned L_CTRL = L_PART + 16 * 1024;
constexpr unsigned L_TOTAL = L_CTRL + 64;

struct Params {
  const char* x;
  const char* w;
  char* ws[NR];
  char* out[NR];
  unsigned units[NG][5][4];
  int rank;
  unsigned xmap;
};

__device__ __forceinline__ unsigned rfl(unsigned v) { return __builtin_amdgcn_readfirstlane(v); }
__device__ __forceinline__ unsigned pk(float a, float b) {
  h2 v; v.x = (_Float16)a; v.y = (_Float16)b; return __builtin_bit_cast(unsigned, v);
}
__device__ __forceinline__ u32x4 cvt8(u32x4 a, u32x4 b) {
  f32x4 fa = __builtin_bit_cast(f32x4, a), fb = __builtin_bit_cast(f32x4, b);
  u32x4 r; r.x = pk(fa.x, fa.y); r.y = pk(fa.z, fa.w); r.z = pk(fb.x, fb.y); r.w = pk(fb.z, fb.w);
  return r;
}
__device__ __forceinline__ float gelu(float y) {
  const float c2 = -2.0f * 0.7978845608028654f * 1.4426950408889634f;
  float u = y + 0.044715f * y * y * y;
  float e = __builtin_amdgcn_exp2f(c2 * u);
  return y * __builtin_amdgcn_rcpf(1.0f + e);
}

#define STAMPV(idx, val) do {} while (0)
#define STAMP(idx) STAMPV(idx, __builtin_amdgcn_s_memrealtime())

__device__ __forceinline__ bool give_up(u64 t0, unsigned* tmo, unsigned call) {
  u64 el = __builtin_amdgcn_s_memrealtime() - t0;
  if (el < 100000ull) return false;
  if (el > DEADLINE ||
      rfl(__hip_atomic_load(tmo, __ATOMIC_RELAXED, __HIP_MEMORY_SCOPE_AGENT)) == call) {
    __hip_atomic_store(tmo, call, __ATOMIC_RELAXED, __HIP_MEMORY_SCOPE_AGENT);
    return true;
  }
  return false;
}

__device__ __forceinline__ unsigned flag_load(const unsigned* p) {
  return __hip_atomic_load(p, __ATOMIC_RELAXED, __HIP_MEMORY_SCOPE_SYSTEM);
}
__device__ __forceinline__ unsigned flag_mask(unsigned v, unsigned call) {
  return (unsigned)__builtin_amdgcn_ballot_w64(v >= call) & 0xFFu;
}
__device__ __forceinline__ void wait_round(const unsigned* p, int r, unsigned& seen, bool& bad, u64 t0,
                                           unsigned* tmo, unsigned call, unsigned* lidle, unsigned mark) {
  while (((seen >> (2 * r)) & 3u) != 3u) {
    seen = flag_mask(flag_load(p), call);
    if (((seen >> (2 * r)) & 3u) == 3u) break;
    if (r < 3) __hip_atomic_store(lidle, mark, __ATOMIC_RELAXED, __HIP_MEMORY_SCOPE_WORKGROUP);
    if (give_up(t0, tmo, call)) { bad = true; break; }
    __builtin_amdgcn_s_sleep(2);
  }
  asm volatile("" ::: "memory");
}

struct Piece { u32x4 r[16]; };

__device__ __forceinline__ void piece_load(Piece& s, const char* sb, unsigned fstride, unsigned voff) {
#pragma unroll
  for (int u = 0; u < 8; ++u) {
    const char* p = sb + (size_t)u * fstride;
    asm volatile("global_load_dwordx4 %0, %1, %2" : "=v"(s.r[2 * u]) : "v"(voff), "s"(p) : "memory");
    asm volatile("global_load_dwordx4 %0, %1, %2 offset:16" : "=v"(s.r[2 * u + 1]) : "v"(voff), "s"(p) : "memory");
  }
}
template <int N>
__device__ __forceinline__ void piece_wait(Piece& s) {
  asm volatile("s_waitcnt vmcnt(%0)" :: "i"(N) : "memory");
#pragma unroll
  for (int u = 0; u < 16; ++u) asm volatile("" : "+v"(s.r[u]));
}
template <int U>
__device__ __forceinline__ void piece_store1(Piece& s, char* db, unsigned voff) {
  u32x4 v = cvt8(s.r[2 * U], s.r[2 * U + 1]);
  asm volatile("global_store_dwordx4 %0, %1, %2 offset:%3 sc0 sc1\n\ts_nop 1"
               :: "v"(voff), "v"(v), "s"(db), "i"((U - 4) * 1024) : "memory");
}
__device__ __forceinline__ void piece_store(Piece& s, char* db  , unsigned voff) {
  piece_store1<0>(s, db, voff); piece_store1<1>(s, db, voff); piece_store1<2>(s, db, voff);
  piece_store1<3>(s, db, voff); piece_store1<4>(s, db, voff); piece_store1<5>(s, db, voff);
  piece_store1<6>(s, db, voff); piece_store1<7>(s, db, voff);
}

typedef __attribute__((address_space(3))) char lds_char;

__device__ __forceinline__ void glds16(const char* src, unsigned voff, unsigned lds) {
  unsigned keep;
  asm volatile("s_mov_b32 %0, m0\n\ts_mov_b32 m0, %3\n\ts_nop 0\n\t"
               "global_load_lds_dwordx4 %1, %2 sc0 sc1\n\ts_mov_b32 m0, %0"
               : "=&s"(keep) : "v"(voff), "s"(src), "s"(lds) : "memory");
}

__device__ __forceinline__ h8 tr_frag(lds_char* p) {
  s4 a = __builtin_amdgcn_ds_read_tr16_b64_v4i16((__attribute__((address_space(3))) s4*)p);
  s4 c = __builtin_amdgcn_ds_read_tr16_b64_v4i16((__attribute__((address_space(3))) s4*)(p + 256));
  s8 r = __builtin_shufflevector(a, c, 0, 1, 2, 3, 4, 5, 6, 7);
  return __builtin_bit_cast(h8, r);
}

__global__ void __launch_bounds__(384) ag_gemm_gelu(Params P) {
  __shared__ __attribute__((aligned(1024))) char smem[L_TOTAL];
  const int lane = threadIdx.x & 63;
  const int wave = rfl(threadIdx.x >> 6);
  const int b = blockIdx.x < 64 ? ((P.xmap >> ((blockIdx.x & 7) * 4)) & 7) * 8 + (blockIdx.x >> 3) : blockIdx.x;
  const int rank = P.rank & (NR - 1);
  const int side = rank >> 2, q = rank & 3;
  char* const myws = P.ws[rank];
  unsigned* const cntp = (unsigned*)(myws + WS_CNT + b * 32 + wave * 4);
  unsigned* const lcnt = (unsigned*)(smem + L_CTRL);
  unsigned* const lidle = (unsigned*)(smem + L_CTRL + 4);
  unsigned* const lfree = (unsigned*)(smem + L_CTRL + 8);

  if (wave == 0) {
    if (b >= 64 + 128 || (b >> 3) == 7) return;
    __builtin_amdgcn_s_setprio(3);
    __builtin_amdgcn_s_waitcnt(0);
    const int j = b & 7, c = b >> 3;
    int dest, nW, strip0, wslot0, xu0, nobj;
    if (c < 3)      { dest = side * 4 + ((q + 1 + c) & 3); nW = 0; strip0 = 0; wslot0 = 0; xu0 = 0; nobj = NA; }
    else if (c < 7) { dest = (side ^ 1) * 4 + ((q + c - 3) & 3); nW = 4; strip0 = lent_strip(q, dest & 3, 0); wslot0 = 16 + q * 4;
                      xu0 = NA; nobj = 4 + 16 - NA; }
    else            { dest = rank; nW = 1; strip0 = c - 8; wslot0 = c - 8; xu0 = c - 8; nobj = 2; }
    dest &= NR - 1;
    const unsigned np = nobj * 32;
    const unsigned laneX = (lane & 31) * (KDIM * 4) + (lane >> 5) * 32;
    const unsigned laneW = (lane >> 2) * (COLS * 4) + (lane & 3) * 32;
    const unsigned lane16 = lane * 16;
    const char* px = P.x;
    const char* pw = P.w;
    asm volatile("" : "+s"(px), "+s"(pw));
    auto load = [&](Piece& s, unsigned pc) {
      pc = pc < np ? pc : np - 1;
      const int obj = pc >> 5, f0 = (pc & 31) * 8;
      const bool isW = obj < nW;
      const char* sb = isW ? pw + (size_t)(strip0 + obj * (lent_strip(0, 0, 1) - lent_strip(0, 0, 0))) * 128 + (size_t)f0 * (16 * COLS * 4)
                           : px + (size_t)(xu0 + obj - nW) * (32 * KDIM * 4) + (size_t)f0 * 64;
      piece_load(s, sb, isW ? 16 * COLS * 4 : 64, isW ? laneW : laneX);
    };
    unsigned cnt;
    asm volatile("global_load_dword %0, %1, off sc1" : "=v"(cnt) : "v"(cntp) : "memory");
    Piece A, B;
    load(A, j);
    load(B, j + 8);
    __builtin_amdgcn_s_barrier();
    piece_wait<16>(A);
    asm volatile("" : "+v"(cnt));
    const unsigned call = rfl(cnt) + 1;
    char* const dws = P.ws[dest];
    char* const ddata = dws + WS_DATA + (size_t)(call & 1) * PAR_BYTES + 4096;
    char* const tick = myws + WS_TICK + c * 128;
    const unsigned tbase = 16 - (call - 1) * (np / 2);
    auto slot_of = [&](unsigned pc) -> unsigned {
      const int obj = pc >> 5;
      return obj < nW ? 128 + wslot0 + obj : rank * 16 + xu0 + obj - nW;
    };
    auto store = [&](Piece& s, unsigned pc) {
      piece_store(s, ddata + (size_t)slot_of(pc) * OBJ + (pc & 31) * 8192, lane16);
    };
    const unsigned zero = 0, one = 1;
    auto flag = [&](unsigned pc) {
      char* fp = dws + WS_FLAG + slot_of(pc) * 128 + (((pc >> 1) & 3) * 8 + ((pc >> 3) & 3) * 2 + (pc & 1)) * 4;
      if (lane == 0) asm volatile("global_store_dword %0, %1, %2 sc0 sc1" :: "v"(zero), "v"(call), "s"(fp) : "memory");
    };
    auto ticket = [&](unsigned& t) {
      if (lane == 0) asm volatile("global_atomic_add %0, %1, %2, %3 sc0" : "=v"(t) : "v"(zero), "v"(one), "s"(tick) : "memory");
    };
    STAMP(0);
    auto piece_of = [&](unsigned t) -> unsigned { return (t >> 3) * 16 + (t & 7); };
    unsigned tk = 0;
    unsigned cur = j, nxt = j + 16;
    store(A, cur);
    ticket(tk);
    load(A, nxt);
    piece_wait<25>(B);
    if (lane == 0) asm volatile("global_store_dword %0, %1, off sc1" :: "v"(cntp), "v"(call) : "memory");
    store(B, cur + 8);
    load(B, nxt + 8);
    do {
      piece_wait<25>(A);
      asm volatile("" : "+v"(tk));
      flag(cur);
      unsigned t = rfl(tk) + tbase;
      const unsigned nn = t < np / 2 ? piece_of(t) : np;
      if (c < 3) asm volatile("s_waitcnt vmcnt(%0)" :: "i"(17 + KA) : "memory");
      else if (c < 7) asm volatile("s_waitcnt vmcnt(%0)" :: "i"(17 + KC) : "memory");
      store(A, nxt);
      ticket(tk);
      load(A, nn);
      piece_wait<26>(B);
      flag(cur + 8);
      if (c < 3) asm volatile("s_waitcnt vmcnt(%0)" :: "i"(18 + KA) : "memory");
      else if (c < 7) asm volatile("s_waitcnt vmcnt(%0)" :: "i"(18 + KC) : "memory");
      store(B, nxt + 8);
      load(B, nn + 8);
      cur = nxt;
      nxt = nn;
    } while (nxt < np);
    if (b < 64) __hip_atomic_store(lfree, 1u, __ATOMIC_RELAXED, __HIP_MEMORY_SCOPE_WORKGROUP);
    piece_wait<25>(A);
    asm volatile("" : "+v"(tk));
    flag(cur);
    piece_wait<0>(B);
    flag(cur + 8);
    STAMP(1);
    return;
  }

  const bool fwd = b < 64;
  if (fwd && (b < 24 || b >= 56)) return;
  const u64 t0 = __builtin_amdgcn_s_memrealtime();
  const unsigned call = rfl(__hip_atomic_load(cntp, __ATOMIC_RELAXED, __HIP_MEMORY_SCOPE_AGENT)) + 1;
  if (lane == 0) __hip_atomic_store(cntp, call, __ATOMIC_RELAXED, __HIP_MEMORY_SCOPE_AGENT);
  unsigned* const tmo = (unsigned*)(myws + WS_TMO);
  const int rr = lane >> 3, chn = lane & 7;
  const unsigned ovoff = rr * (COLS * 4) + chn * 16;
  bool bad = false;
  if (wave == 1) {
    STAMPV(5, t0);
    if (lane < 8) lcnt[lane] = 0;
    asm volatile("s_waitcnt lgkmcnt(0)" ::: "memory");
  }
  __builtin_amdgcn_s_barrier();

  if (fwd) {
    const int j = b & 7, p = (q + (b >> 3) - 3) & 3;
    const int owner = ((side ^ 1) * 4 + p) & (NR - 1);
    char* const obase = P.out[owner];
    unsigned* const lnext = (unsigned*)(smem + L_CTRL + 16);
    const unsigned tl = lane < 3 ? lane : 0;
    const unsigned* const stgf = (const unsigned*)(myws + WS_STGF + ((p * 4 + ((j + tl) & 3)) * 8 + 2 * tl + (j >> 2)) * 128);
    const unsigned n0 = P.units[j >> 2][1 + p][3], n1 = P.units[2 + (j >> 2)][1 + p][3], n2 = P.units[4 + (j >> 2)][1 + p][3];
    const unsigned nl = (tl == 0 ? n0 : (tl == 1 ? n1 : n2)) - 1;
    while (__builtin_amdgcn_s_memrealtime() - t0 < FWD_T0) __builtin_amdgcn_s_sleep(16);
    unsigned fv = 0;
    bool open = false, quick = false;
    u64 tl0 = 0;
    while (wave <= FW) {
      const unsigned nx = __hip_atomic_load(lnext + tl, __ATOMIC_RELAXED, __HIP_MEMORY_SCOPE_WORKGROUP);
      const unsigned left = (unsigned)__builtin_amdgcn_ballot_w64(nx < nl) & 7u;
      if (left == 0) break;
      const unsigned rdy = bad ? left : (unsigned)__builtin_amdgcn_ballot_w64(fv > call * 16 + nx) & left;
      if (rdy == 0) {
        fv = __hip_atomic_load(stgf, __ATOMIC_RELAXED, __HIP_MEMORY_SCOPE_SYSTEM);
        if (give_up(t0, tmo, call)) bad = true;
        __builtin_amdgcn_s_sleep(2);
        continue;
      }
      const unsigned rdy2 = quick ? (unsigned)__builtin_amdgcn_ballot_w64(fv > call * 16 + nx + 1 && nx + 1 < nl) & 7u : 0u;
      const unsigned t = __builtin_ctz(rdy2 ? rdy2 : rdy);
      const unsigned k = __builtin_amdgcn_readlane(nx, t);
      unsigned got = k;
      if (lane == 0) __hip_atomic_compare_exchange_strong(lnext + t, &got, k + (rdy2 ? 2 : 1), __ATOMIC_RELAXED, __ATOMIC_RELAXED, __HIP_MEMORY_SCOPE_WORKGROUP);
      if (rfl(got) != k) continue;
      const unsigned g = 2 * t + (j >> 2), i = (j + t) & 3;
      const unsigned unit = (P.units[g][1 + p][k >> 2] >> ((k & 3) * 8)) & 127u;
      const unsigned unit2 = (P.units[g][1 + p][(k + 1) >> 2] >> (((k + 1) & 3) * 8)) & 127u;
      const char* const src = myws + WS_STG + (size_t)(((p * 4 + i) * 8 + g) * 12 + k) * 4096;
      f32x4 y0, y1, y2, y3, z0, z1, z2, z3;
      asm volatile("global_load_dwordx4 %0, %9, %10 sc0 sc1\n\t"
                   "global_load_dwordx4 %1, %9, %10 offset:1024 sc0 sc1\n\t"
                   "global_load_dwordx4 %2, %9, %10 offset:2048 sc0 sc1\n\t"
                   "global_load_dwordx4 %3, %9, %10 offset:3072 sc0 sc1\n\t"
                   "global_load_dwordx4 %4, %9, %11 sc0 sc1\n\t"
                   "global_load_dwordx4 %5, %9, %11 offset:1024 sc0 sc1\n\t"
                   "global_load_dwordx4 %6, %9, %11 offset:2048 sc0 sc1\n\t"
                   "global_load_dwordx4 %7, %9, %11 offset:3072 sc0 sc1\n\t"
                   "global_load_dword %8, %12, off sc0 sc1\n\t"
                   "s_waitcnt vmcnt(0)"
                   : "=&v"(y0), "=&v"(y1), "=&v"(y2), "=&v"(y3), "=&v"(z0), "=&v"(z1), "=&v"(z2), "=&v"(z3), "=&v"(fv)
                   : "v"(lane * 16), "s"(src), "s"(src + (rdy2 ? 4096 : 0)), "v"(stgf) : "memory");
      if (!open) {
        while (rfl(__hip_atomic_load(lfree, __ATOMIC_RELAXED, __HIP_MEMORY_SCOPE_WORKGROUP)) == 0) {
          if (__builtin_amdgcn_s_memrealtime() - t0 > 2 * DEADLINE) break;
          __builtin_amdgcn_s_sleep(2);
        }
        open = true;
        if (wave < 5) STAMP(6 + wave);
      }
      const u64 now = __builtin_amdgcn_s_memrealtime();
      quick = now - tl0 < FWD_QUICK;
      tl0 = now;
      STAMP(16 + t * 10 + k);
      if (bad) y0.x = __builtin_nanf("");
      auto put = [&](unsigned u, f32x4 v0, f32x4 v1, f32x4 v2, f32x4 v3) {
        const unsigned vo = ovoff + u * (32 * COLS * 4) + lent_strip(p, q, i) * 128;
        asm volatile("global_store_dwordx4 %0, %4, %8 sc0 sc1\n\t"
                     "global_store_dwordx4 %1, %5, %8 sc0 sc1\n\t"
                     "global_store_dwordx4 %2, %6, %8 sc0 sc1\n\t"
                     "global_store_dwordx4 %3, %7, %8 sc0 sc1\n\ts_nop 1"
                     :: "v"(vo), "v"(vo + 8 * COLS * 4), "v"(vo + 16 * COLS * 4), "v"(vo + 24 * COLS * 4),
                        "v"(v0), "v"(v1), "v"(v2), "v"(v3), "s"(obase) : "memory");
      };
      put(unit, y0, y1, y2, y3);
      if (rdy2) {
        STAMP(16 + t * 10 + k + 1);
        put(unit2, z0, z1, z2, z3);
      }
    }
    asm volatile("s_waitcnt vmcnt(0)" ::: "memory");
    unsigned* const dp = (unsigned*)(P.ws[owner] + WS_DONE + q * 256 + (j * 5 + wave - 1) * 4);
    if (lane == 0) __hip_atomic_store(dp, call, __ATOMIC_RELAXED, __HIP_MEMORY_SCOPE_SYSTEM);
    STAMP(10 + wave);
    return;
  }

  const int nq = (b & 3) * 24 + (b >> 3) - 8;
  const int g = ((b >> 2) & 1) * 3 + (nq >> 5), s = nq & 31;
  const bool foreign = s >= 16;
  const int owner = (foreign ? (side ^ 1) * 4 + ((s - 16) >> 2) : rank) & (NR - 1);
  const int cls = foreign ? 1 + (owner & 3) : 0;
  const unsigned ul0 = P.units[g][cls][0], ul1 = P.units[g][cls][1], ul2 = P.units[g][cls][2];
  const unsigned nu = P.units[g][cls][3] < 12u ? P.units[g][cls][3] : 12u;
  auto unit_at = [&](unsigned k) -> unsigned {
    unsigned wv = k < 4 ? ul0 : (k < 8 ? ul1 : ul2);
    return (wv >> ((k & 3) * 8)) & 127u;
  };

  if (wave == 1) {
    const int col0 = (foreign ? lent_strip(owner & 3, q, s & 3) : s) * 32;
    char* const obase = P.out[owner] + col0 * 4;
    const unsigned prod = (s & 15) * 8 + g;
    char* const stg = myws + WS_STG + (size_t)prod * (12 * 4096);
    unsigned* const stgf = (unsigned*)(myws + WS_STGF + prod * 128);
    unsigned* const donep = (unsigned*)(P.ws[owner] + WS_DONE + q * 256 + 160 + ((s & 3) * 6 + g) * 4);
    const char* const pl = smem + L_PART + rr * 128 + ((chn ^ rr) * 16);
    f32x4 keep[KEEP][4];
    unsigned kd = 0, ks = 0;
    while (ks < nu) {
      const u64 st = __hip_atomic_load((u64*)lcnt, __ATOMIC_RELAXED, __HIP_MEMORY_SCOPE_WORKGROUP);
      const unsigned have = rfl((unsigned)st), idle = rfl((unsigned)(st >> 32));
      if (__builtin_amdgcn_s_memrealtime() - t0 > 2 * DEADLINE) bad = true;
      if (have >= (kd + 1) * 4 || bad) {
        asm volatile("" ::: "memory");
        f32x4 y[4];
#pragma unroll
        for (int it = 0; it < 4; ++it) {
          f32x4 p0 = *(const f32x4*)(pl + it * 1024);
          f32x4 p1 = *(const f32x4*)(pl + it * 1024 + 4096);
          f32x4 p2 = *(const f32x4*)(pl + it * 1024 + 8192);
          f32x4 p3 = *(const f32x4*)(pl + it * 1024 + 12288);
          y[it] = (p0 + p1) + (p2 + p3);
        }
#pragma unroll
        for (int it = 0; it < 4; ++it) asm volatile("" : "+v"(y[it]));
        asm volatile("s_waitcnt lgkmcnt(0)" ::: "memory");
        if (lane == 0) __hip_atomic_store(lfree, kd + 1, __ATOMIC_RELAXED, __HIP_MEMORY_SCOPE_WORKGROUP);
#pragma unroll
        for (int it = 0; it < 4; ++it) {
          y[it].x = gelu(y[it].x); y[it].y = gelu(y[it].y); y[it].z = gelu(y[it].z); y[it].w = gelu(y[it].w);
          if (bad) y[it].x = __builtin_nanf("");
        }
#pragma unroll
        for (unsigned k = 0; k < KEEP; ++k)
          if (kd - ks == k) {
#pragma unroll
            for (int it = 0; it < 4; ++it) { keep[k][it] = y[it]; asm volatile("" : "+v"(keep[k][it])); }
          }
        ++kd;
      } else if (ks == kd || idle != kd + 1) {
        __builtin_amdgcn_s_sleep(1);
        continue;
      }
      if (kd - ks == KEEP || kd + 1 >= nu || idle == kd + 1) {
        const bool stage = foreign && kd < nu;
        char* const base = stage ? stg : obase;
        const unsigned step = stage ? 1024 : 8 * COLS * 4;
#pragma unroll
        for (unsigned k = 0; k < KEEP; ++k)
          if (ks + k < kd) {
            const unsigned vo = stage ? lane * 16 + (ks + k) * 4096 : ovoff + unit_at(ks + k) * (32 * COLS * 4);
#pragma unroll
            for (int it = 0; it < 4; ++it)
              asm volatile("global_store_dwordx4 %0, %1, %2 sc0 sc1\n\ts_nop 1"
                           :: "v"(vo + it * step), "v"(keep[k][it]), "s"(base) : "memory");
            STAMP(32 + ks + k);
          }
        ks = kd;
        if (stage) {
          asm volatile("s_waitcnt vmcnt(0)" ::: "memory");
          if (lane == 0) __hip_atomic_store(stgf, call * 16 + kd, __ATOMIC_RELAXED, __HIP_MEMORY_SCOPE_SYSTEM);
        }
      }
    }
    char* const lastp = obase + (size_t)unit_at(0) * (32 * COLS * 4);
    asm volatile("s_waitcnt vmcnt(0)" ::: "memory");
    STAMP(2);
    if (foreign) {
      if (lane == 0) __hip_atomic_store(donep, call, __ATOMIC_RELAXED, __HIP_MEMORY_SCOPE_SYSTEM);
    } else if (b == 64) {
      const char* dp = myws + WS_DONE + lane * 16;
      while (true) {
        u32x4 v;
        asm volatile("global_load_dwordx4 %0, %1, off sc0 sc1\n\ts_waitcnt vmcnt(0)" : "=&v"(v) : "v"(dp) : "memory");
        bool ok = v.x >= call && v.y >= call && v.z >= call && v.w >= call;
        if (__all(ok)) break;
        if (give_up(t0, tmo, call)) {
          if (lane == 0) __hip_atomic_store((float*)lastp, __builtin_nanf(""), __ATOMIC_RELAXED, __HIP_MEMORY_SCOPE_SYSTEM);
          break;
        }
        __builtin_amdgcn_s_sleep(1);
      }
      STAMP(3);
    }
    return;
  }

  const int w = wave - 2;
  char* const data = myws + WS_DATA + (size_t)(call & 1) * PAR_BYTES;
  const unsigned* const flags = (const unsigned*)(myws + WS_FLAG) + w * 8 + (lane & 7);
  lds_char* const lw = (lds_char*)smem + w * 32768;
  lds_char* const ltr = lw + ((lane >> 5) * 8 + ((lane & 15) >> 2)) * 64 + ((lane >> 4) & 1) * 32 + (lane & 3) * 8;
  h8 wr[32];
  {
    const unsigned* pp = flags + (128 + s) * 32;
    const char* src = data + (size_t)(128 + s) * OBJ + w * 16384;
    const unsigned svoff = lane * 16;
    const unsigned lwa = (unsigned)(size_t)lw;
    unsigned seen = 0;
#pragma unroll
    for (int r = 0; r < 4; ++r) {
      wait_round(pp, r, seen, bad, t0, tmo, call, lidle, 0);
#pragma unroll
      for (int i = 0; i < 16; ++i)
        glds16(src + r * 65536 + i * 1024, svoff, lwa + (r & 1) * 16384 + i * 1024);
      if (r & 1) asm volatile("s_waitcnt vmcnt(0)" ::: "memory");
      if (r == 1) {
#pragma unroll
        for (int i = 0; i < 32; ++i) wr[i] = tr_frag(ltr + i * 1024);
#pragma unroll
        for (int i = 0; i < 32; ++i) asm volatile("" : "+v"(wr[i]));
        asm volatile("s_waitcnt lgkmcnt(0)" ::: "memory");
      }
    }
  }
  if (w == 0) STAMP(4);
  const __amdgpu_buffer_rsrc_t rsrc =
      __builtin_amdgcn_make_buffer_rsrc(data, 0, 128 * OBJ, 0x00020000);
  const unsigned xvoff = w * 16384 + lane * 16;
  char* const pw = smem + L_PART + w * 4096 + (lane & 31) * 128;
  unsigned nextf = 0;
  for (unsigned k = 0; k < nu; ++k) {
    const unsigned e = unit_at(k);
    const unsigned* pp = flags + e * 32;
    const unsigned* pn = flags + unit_at(k + 1 < nu ? k + 1 : k) * 32;
    const unsigned soff = e * OBJ;
    unsigned seen = flag_mask(nextf, call);
    f32x16 acc = {0.f, 0.f, 0.f, 0.f, 0.f, 0.f, 0.f, 0.f, 0.f, 0.f, 0.f, 0.f, 0.f, 0.f, 0.f, 0.f};
#pragma unroll
    for (int r = 0; r < 4; ++r) {
      wait_round(pp, r, seen, bad, t0, tmo, call, lidle, k + 1);
      u32x4 xa[16];
#pragma unroll
      for (int i = 0; i < 16; ++i)
        xa[i] = __builtin_amdgcn_raw_buffer_load_b128(rsrc, xvoff + (i & 3) * 1024, soff + r * 65536 + (i >> 2) * 4096, 17);
      __builtin_amdgcn_sched_barrier(0);
      if (r == 3) { nextf = flag_load(pn); __builtin_amdgcn_sched_barrier(0); }
#pragma unroll
      for (int i = 0; i < 16; ++i) {
        h8 wf = r < 2 ? wr[r * 16 + i] : tr_frag(ltr + ((r - 2) * 16 + i) * 1024);
        acc = __builtin_amdgcn_mfma_f32_32x32x16_f16(wf, __builtin_bit_cast(h8, xa[i]), acc, 0, 0, 0);
      }
    }
    if (bad) {
#pragma unroll
      for (int i = 0; i < 16; ++i) acc[i] = __builtin_nanf("");
    }
    while (rfl(__hip_atomic_load(lfree, __ATOMIC_RELAXED, __HIP_MEMORY_SCOPE_WORKGROUP)) < k) {
      if (__builtin_amdgcn_s_memrealtime() - t0 > 2 * DEADLINE) break;
      __builtin_amdgcn_s_sleep(1);
    }
    asm volatile("" ::: "memory");
#pragma unroll
    for (int gq = 0; gq < 4; ++gq) {
      f32x4 v = {acc[gq * 4], acc[gq * 4 + 1], acc[gq * 4 + 2], acc[gq * 4 + 3]};
      *(f32x4*)(pw + (((gq * 2 + (lane >> 5)) ^ (lane & 7)) * 16)) = v;
    }
    asm volatile("s_waitcnt lgkmcnt(0)" ::: "memory");
    if (lane == 0) __hip_atomic_fetch_add(lcnt, 1u, __ATOMIC_RELAXED, __HIP_MEMORY_SCOPE_WORKGROUP);
    if (w == 0) STAMP(16 + k);
  }
}

unsigned link_map(int rank) {
  const unsigned dflt = 0x76543210u;
  int dev = -1, n = 0, node[NR], port[NR + 1];
  if (hipGetDevice(&dev) != hipSuccess || hipGetDeviceCount(&n) != hipSuccess || dev != rank || n != NR) return dflt;
  for (int i = 0; i <= NR; ++i) port[i] = -1;
  for (int d = 0; d < NR; ++d) {
    char id[32], path[96];
    if (hipDeviceGetPCIBusId(id, sizeof id, d) != hipSuccess) return dflt;
    for (char* c = id; *c; ++c) if (*c >= 'A' && *c <= 'F') *c += 'a' - 'A';
    snprintf(path, sizeof path, "/sys/bus/pci/devices/%s/xgmi_port_num", id);
    FILE* f = fopen(path, "r");
    if (!f) return dflt;
    unsigned a, pa, z, pz;
    node[d] = -1;
    while (fscanf(f, " %x:%x -> %x:%x", &a, &pa, &z, &pz) == 4) {
      node[d] = (int)a;
      if (d == rank && z <= (unsigned)NR && pa < 8u) port[z] = (int)pa;
    }
    fclose(f);
    if (node[d] < 0 || node[d] > NR) return dflt;
  }
  unsigned map = 0, used = 0;
  const int side = rank >> 2, q = rank & 3;
  for (int c = 0; c < 7; ++c) {
    const int x = port[node[c < 3 ? side * 4 + ((q + 1 + c) & 3) : (side ^ 1) * 4 + ((q + c - 3) & 3)]];
    if (x < 0 || ((used >> x) & 1)) return dflt;
    used |= 1u << x;
    map |= (unsigned)c << (4 * x);
  }
  return map | 7u << (4 * __builtin_ctz(~used));
}

}

extern "C" void kernel_launch(int rank, int nranks, ncclComm_t comm,
                              hipStream_t stream,
                              void* const* d_in, const int* in_sizes, int n_in,
                              void* d_out, int out_size,
                              const int* in_dtypes, int out_dtype,
                              void* const* d_out_all,
                              void* const* d_ws_all, size_t ws_bytes) {
  (void)comm; (void)in_dtypes; (void)out_dtype; (void)d_out;
  if (nranks != NR || (unsigned)rank >= (unsigned)NR || n_in < 2 || ws_bytes < WS_NEED ||
      in_sizes[0] != ROWS * KDIM || in_sizes[1] != KDIM * COLS || out_size != NR * ROWS * COLS) {
    fprintf(stderr, "kernel_launch: unexpected problem shape\n");
    return;
  }
  Params P;
  P.x = (const char*)d_in[0];
  P.w = (const char*)d_in[1];
  for (int p = 0; p < NR; ++p) { P.ws[p] = (char*)d_ws_all[p]; P.out[p] = (char*)d_out_all[p]; }
  P.rank = rank;
  P.xmap = link_map(rank);
  const int side = rank >> 2, q = rank & 3;
  static const unsigned char order[NG][12] = {
    {60, 61, 0, 6, 12, 40, 46, 18, 24, 52, 30, 99},
    {62, 63, 1, 7, 13, 41, 47, 19, 25, 53, 31, 99},
    {64, 65, 72, 2, 8, 14, 42, 20, 48, 26, 54, 32},
    {66, 67, 73, 3, 9, 43, 15, 21, 49, 55, 27, 33},
    {68, 69, 74, 4, 10, 16, 44, 22, 50, 28, 34, 99},
    {70, 71, 75, 5, 11, 17, 45, 23, 51, 29, 35, 99}};
  for (int g = 0; g < NG; ++g)
    for (int cls = 0; cls < 5; ++cls) {
      unsigned u[12] = {0}, n = 0;
      for (int k = 0; k < 12; ++k) {
        int v = order[g][k];
        if (v == 99) break;
        if (cls > 0 && (v >= 60 + NA || (v >= 40 && v < 60 && ((v - 40) & 3) == cls - 1))) continue;
        if (v < 40) u[n++] = (side * 4 + ((q + 1 + v % 3) & 3)) * 16 + v / 3;
        else if (v < 60) u[n++] = ((side ^ 1) * 4 + ((v - 40) & 3)) * 16 + NA + ((v - 40) >> 2);
        else u[n++] = rank * 16 + v - 60;
      }
      for (int k = 0; k < 3; ++k)
        P.units[g][cls][k] = u[4 * k] | (u[4 * k + 1] << 8) | (u[4 * k + 2] << 16) | (u[4 * k + 3] << 24);
      P.units[g][cls][3] = n;
    }
  hipLaunchKernelGGL(ag_gemm_gelu, dim3(256), dim3(384), 0, stream, P);
  HIP_CHECK(hipGetLastError());
}
```

```cpp
#include <hip/hip_runtime.h>
#include <rccl/rccl.h>
#include <cstdio>
#define DIAG
#include <algorithm>

#define HIP_CHECK(expr) do { hipError_t _e = (expr); if (_e != hipSuccess) { \
    fprintf(stderr, "HIP error %s at %s:%d\n", hipGetErrorString(_e), __FILE__, __LINE__); \
    return; } } while (0)

namespace {

typedef unsigned u32x4 __attribute__((ext_vector_type(4)));
typedef float f32x4 __attribute__((ext_vector_type(4)));
typedef float f32x16 __attribute__((ext_vector_type(16)));
typedef _Float16 h2 __attribute__((ext_vector_type(2)));
typedef _Float16 h8 __attribute__((ext_vector_type(8)));
typedef short s4 __attribute__((ext_vector_type(4)));
typedef short s8 __attribute__((ext_vector_type(8)));
typedef unsigned long long u64;

constexpr int NR = 8;
constexpr int KDIM = 4096;
constexpr int ROWS = 512;
constexpr int COLS = 512;
constexpr unsigned OBJ = 256u * 1024u;
constexpr int NG = 6;
constexpr int NA = 12;
constexpr size_t WS_CNT = 0;
constexpr size_t WS_TMO = 16 * 1024;
constexpr size_t WS_TICK = 20 * 1024;
constexpr size_t WS_FLAG = 32 * 1024;
constexpr size_t WS_DONE = 64 * 1024;
constexpr size_t WS_STGF = 80 * 1024;
constexpr size_t WS_DIAG = 128 * 1024;
constexpr size_t WS_DATA = 1024 * 1024;
constexpr unsigned NSLOT = 160;
constexpr size_t PAR_BYTES = (size_t)NSLOT * OBJ;
constexpr size_t WS_STG = WS_DATA + 2 * PAR_BYTES;
constexpr size_t WS_NEED = WS_STG + 128 * 12 * 4096;
constexpr u64 DEADLINE = 100000000ull;
#define FWD_T0 2500
#define SMODE 1
__host__ __device__ constexpr int lent_strip(int o, int h, int i) {
  return SMODE == 0 ? h * 4 + i : (SMODE == 1 ? ((o + h) & 3) * 4 + i : i * 4 + ((o + h) & 3));
}
#define FW 4
#define KA 4
#define KC 4
#define FWD_QUICK 0
#define KEEP 4
constexpr unsigned L_PART = 128 * 1024;
constexpr unsigned L_CTRL = L_PART + 16 * 1024;
constexpr unsigned L_TOTAL = L_CTRL + 64;

struct Params {
  const char* x;
  const char* w;
  char* ws[NR];
  char* out[NR];
  unsigned units[NG][5][4];
  int rank;
  unsigned xmap;
};

__device__ __forceinline__ unsigned rfl(unsigned v) { return __builtin_amdgcn_readfirstlane(v); }
__device__ __forceinline__ unsigned pk(float a, float b) {
  h2 v; v.x = (_Float16)a; v.y = (_Float16)b; return __builtin_bit_cast(unsigned, v);
}
__device__ __forceinline__ u32x4 cvt8(u32x4 a, u32x4 b) {
  f32x4 fa = __builtin_bit_cast(f32x4, a), fb = __builtin_bit_cast(f32x4, b);
  u32x4 r; r.x = pk(fa.x, fa.y); r.y = pk(fa.z, fa.w); r.z = pk(fb.x, fb.y); r.w = pk(fb.z, fb.w);
  return r;
}
__device__ __forceinline__ float gelu(float y) {
  const float c2 = -2.0f * 0.7978845608028654f * 1.4426950408889634f;
  float u = y + 0.044715f * y * y * y;
  float e = __builtin_amdgcn_exp2f(c2 * u);
  return y * __builtin_amdgcn_rcpf(1.0f + e);
}

#define STAMPV(idx, val) do { if (lane == 0) { u64 _t = (val); \
  __hip_atomic_store((u64*)(myws + WS_DIAG + ((size_t)b * 64 + (idx)) * 8), _t, __ATOMIC_RELAXED, __HIP_MEMORY_SCOPE_AGENT); } } while (0)
#define STAMP(idx) STAMPV(idx, __builtin_amdgcn_s_memrealtime())

__device__ __forceinline__ bool give_up(u64 t0, unsigned* tmo, unsigned call) {
  u64 el = __builtin_amdgcn_s_memrealtime() - t0;
  if (el < 100000ull) return false;
  if (el > DEADLINE ||
      rfl(__hip_atomic_load(tmo, __ATOMIC_RELAXED, __HIP_MEMORY_SCOPE_AGENT)) == call) {
    __hip_atomic_store(tmo, call, __ATOMIC_RELAXED, __HIP_MEMORY_SCOPE_AGENT);
    return true;
  }
  return false;
}

__device__ __forceinline__ unsigned flag_load(const unsigned* p) {
  return __hip_atomic_load(p, __ATOMIC_RELAXED, __HIP_MEMORY_SCOPE_SYSTEM);
}
__device__ __forceinline__ unsigned flag_mask(unsigned v, unsigned call) {
  return (unsigned)__builtin_amdgcn_ballot_w64(v >= call) & 0xFFu;
}
__device__ __forceinline__ void wait_round(const unsigned* p, int r, unsigned& seen, bool& bad, u64 t0,
                                           unsigned* tmo, unsigned call, unsigned* lidle, unsigned mark) {
  while (((seen >> (2 * r)) & 3u) != 3u) {
    seen = flag_mask(flag_load(p), call);
    if (((seen >> (2 * r)) & 3u) == 3u) break;
    if (r < 3) __hip_atomic_store(lidle, mark, __ATOMIC_RELAXED, __HIP_MEMORY_SCOPE_WORKGROUP);
    if (give_up(t0, tmo, call)) { bad = true; break; }
    __builtin_amdgcn_s_sleep(2);
  }
  asm volatile("" ::: "memory");
}

struct Piece { u32x4 r[16]; };

__device__ __forceinline__ void piece_load(Piece& s, const char* sb, unsigned fstride, unsigned voff) {
#pragma unroll
  for (int u = 0; u < 8; ++u) {
    const char* p = sb + (size_t)u * fstride;
    asm volatile("global_load_dwordx4 %0, %1, %2" : "=v"(s.r[2 * u]) : "v"(voff), "s"(p) : "memory");
    asm volatile("global_load_dwordx4 %0, %1, %2 offset:16" : "=v"(s.r[2 * u + 1]) : "v"(voff), "s"(p) : "memory");
  }
}
template <int N>
__device__ __forceinline__ void piece_wait(Piece& s) {
  asm volatile("s_waitcnt vmcnt(%0)" :: "i"(N) : "memory");
#pragma unroll
  for (int u = 0; u < 16; ++u) asm volatile("" : "+v"(s.r[u]));
}
template <int U>
__device__ __forceinline__ void piece_store1(Piece& s, char* db, unsigned voff) {
  u32x4 v = cvt8(s.r[2 * U], s.r[2 * U + 1]);
  asm volatile("global_store_dwordx4 %0, %1, %2 offset:%3 sc0 sc1\n\ts_nop 1"
               :: "v"(voff), "v"(v), "s"(db), "i"((U - 4) * 1024) : "memory");
}
__device__ __forceinline__ void piece_store(Piece& s, char* db  , unsigned voff) {
  piece_store1<0>(s, db, voff); piece_store1<1>(s, db, voff); piece_store1<2>(s, db, voff);
  piece_store1<3>(s, db, voff); piece_store1<4>(s, db, voff); piece_store1<5>(s, db, voff);
  piece_store1<6>(s, db, voff); piece_store1<7>(s, db, voff);
}

typedef __attribute__((address_space(3))) char lds_char;

__device__ __forceinline__ void glds16(const char* src, unsigned voff, unsigned lds) {
  unsigned keep;
  asm volatile("s_mov_b32 %0, m0\n\ts_mov_b32 m0, %3\n\ts_nop 0\n\t"
               "global_load_lds_dwordx4 %1, %2 sc0 sc1\n\ts_mov_b32 m0, %0"
               : "=&s"(keep) : "v"(voff), "s"(src), "s"(lds) : "memory");
}

__device__ __forceinline__ h8 tr_frag(lds_char* p) {
  s4 a = __builtin_amdgcn_ds_read_tr16_b64_v4i16((__attribute__((address_space(3))) s4*)p);
  s4 c = __builtin_amdgcn_ds_read_tr16_b64_v4i16((__attribute__((address_space(3))) s4*)(p + 256));
  s8 r = __builtin_shufflevector(a, c, 0, 1, 2, 3, 4, 5, 6, 7);
  return __builtin_bit_cast(h8, r);
}

__global__ void __launch_bounds__(384) ag_gemm_gelu(Params P) {
  __shared__ __attribute__((aligned(1024))) char smem[L_TOTAL];
  const int lane = threadIdx.x & 63;
  const int wave = rfl(threadIdx.x >> 6);
  const int b = blockIdx.x < 64 ? ((P.xmap >> ((blockIdx.x & 7) * 4)) & 7) * 8 + (blockIdx.x >> 3) : blockIdx.x;
  const int rank = P.rank & (NR - 1);
  const int side = rank >> 2, q = rank & 3;
  char* const myws = P.ws[rank];
  unsigned* const cntp = (unsigned*)(myws + WS_CNT + b * 32 + wave * 4);
  unsigned* const lcnt = (unsigned*)(smem + L_CTRL);
  unsigned* const lidle = (unsigned*)(smem + L_CTRL + 4);
  unsigned* const lfree = (unsigned*)(smem + L_CTRL + 8);

  if (wave == 0) {
    if (b >= 64 + 128 || (b >> 3) == 7) return;
    __builtin_amdgcn_s_setprio(3);
    __builtin_amdgcn_s_waitcnt(0);
    const int j = b & 7, c = b >> 3;
    int dest, nW, strip0, wslot0, xu0, nobj;
    if (c < 3)      { dest = side * 4 + ((q + 1 + c) & 3); nW = 0; strip0 = 0; wslot0 = 0; xu0 = 0; nobj = NA; }
    else if (c < 7) { dest = (side ^ 1) * 4 + ((q + c - 3) & 3); nW = 4; strip0 = lent_strip(q, dest & 3, 0); wslot0 = 16 + q * 4;
                      xu0 = NA; nobj = 4 + 16 - NA; }
    else            { dest = rank; nW = 1; strip0 = c - 8; wslot0 = c - 8; xu0 = c - 8; nobj = 2; }
    dest &= NR - 1;
    const unsigned np = nobj * 32;
    const unsigned laneX = (lane & 31) * (KDIM * 4) + (lane >> 5) * 32;
    const unsigned laneW = (lane >> 2) * (COLS * 4) + (lane & 3) * 32;
    const unsigned lane16 = lane * 16;
    const char* px = P.x;
    const char* pw = P.w;
    asm volatile("" : "+s"(px), "+s"(pw));
    auto load = [&](Piece& s, unsigned pc) {
      pc = pc < np ? pc : np - 1;
      const int obj = pc >> 5, f0 = (pc & 31) * 8;
      const bool isW = obj < nW;
      const char* sb = isW ? pw + (size_t)(strip0 + obj * (lent_strip(0, 0, 1) - lent_strip(0, 0, 0))) * 128 + (size_t)f0 * (16 * COLS * 4)
                           : px + (size_t)(xu0 + obj - nW) * (32 * KDIM * 4) + (size_t)f0 * 64;
      piece_load(s, sb, isW ? 16 * COLS * 4 : 64, isW ? laneW : laneX);
    };
    unsigned cnt;
    asm volatile("global_load_dword %0, %1, off sc1" : "=v"(cnt) : "v"(cntp) : "memory");
    Piece A, B;
    load(A, j);
    load(B, j + 8);
    __builtin_amdgcn_s_barrier();
    piece_wait<16>(A);
    asm volatile("" : "+v"(cnt));
    const unsigned call = rfl(cnt) + 1;
    char* const dws = P.ws[dest];
    char* const ddata = dws + WS_DATA + (size_t)(call & 1) * PAR_BYTES + 4096;
    char* const tick = myws + WS_TICK + c * 128;
    const unsigned tbase = 16 - (call - 1) * (np / 2);
    auto slot_of = [&](unsigned pc) -> unsigned {
      const int obj = pc >> 5;
      return obj < nW ? 128 + wslot0 + obj : rank * 16 + xu0 + obj - nW;
    };
    auto store = [&](Piece& s, unsigned pc) {
      piece_store(s, ddata + (size_t)slot_of(pc) * OBJ + (pc & 31) * 8192, lane16);
    };
    const unsigned zero = 0, one = 1;
    auto flag = [&](unsigned pc) {
      char* fp = dws + WS_FLAG + slot_of(pc) * 128 + (((pc >> 1) & 3) * 8 + ((pc >> 3) & 3) * 2 + (pc & 1)) * 4;
      if (lane == 0) asm volatile("global_store_dword %0, %1, %2 sc0 sc1" :: "v"(zero), "v"(call), "s"(fp) : "memory");
    };
    auto ticket = [&](unsigned& t) {
      if (lane == 0) asm volatile("global_atomic_add %0, %1, %2, %3 sc0" : "=v"(t) : "v"(zero), "v"(one), "s"(tick) : "memory");
    };
    STAMP(0);
    auto piece_of = [&](unsigned t) -> unsigned { return (t >> 3) * 16 + (t & 7); };
    unsigned tk = 0;
    unsigned cur = j, nxt = j + 16;
    store(A, cur);
    ticket(tk);
    load(A, nxt);
    piece_wait<25>(B);
    if (lane == 0) asm volatile("global_store_dword %0, %1, off sc1" :: "v"(cntp), "v"(call) : "memory");
    store(B, cur + 8);
    load(B, nxt + 8);
    do {
      piece_wait<25>(A);
      asm volatile("" : "+v"(tk));
      flag(cur);
      unsigned t = rfl(tk) + tbase;
      const unsigned nn = t < np / 2 ? piece_of(t) : np;
      if (c < 3) asm volatile("s_waitcnt vmcnt(%0)" :: "i"(17 + KA) : "memory");
      else if (c < 7) asm volatile("s_waitcnt vmcnt(%0)" :: "i"(17 + KC) : "memory");
      store(A, nxt);
      ticket(tk);
      load(A, nn);
      piece_wait<26>(B);
      flag(cur + 8);
      if ((nxt >> 5) != (cur >> 5)) STAMP(48 + (cur >> 5));
      if (c < 3) asm volatile("s_waitcnt vmcnt(%0)" :: "i"(18 + KA) : "memory");
      else if (c < 7) asm volatile("s_waitcnt vmcnt(%0)" :: "i"(18 + KC) : "memory");
      store(B, nxt + 8);
      load(B, nn + 8);
      cur = nxt;
      nxt = nn;
    } while (nxt < np);
    if (b < 64) __hip_atomic_store(lfree, 1u, __ATOMIC_RELAXED, __HIP_MEMORY_SCOPE_WORKGROUP);
    piece_wait<25>(A);
    asm volatile("" : "+v"(tk));
    flag(cur);
    piece_wait<0>(B);
    flag(cur + 8);
    STAMP(1);
    return;
  }

  const bool fwd = b < 64;
  if (fwd && (b < 24 || b >= 56)) return;
  const u64 t0 = __builtin_amdgcn_s_memrealtime();
  const unsigned call = rfl(__hip_atomic_load(cntp, __ATOMIC_RELAXED, __HIP_MEMORY_SCOPE_AGENT)) + 1;
  if (lane == 0) __hip_atomic_store(cntp, call, __ATOMIC_RELAXED, __HIP_MEMORY_SCOPE_AGENT);
  unsigned* const tmo = (unsigned*)(myws + WS_TMO);
  const int rr = lane >> 3, chn = lane & 7;
  const unsigned ovoff = rr * (COLS * 4) + chn * 16;
  bool bad = false;
  if (wave == 1) {
    STAMPV(5, t0);
    if (lane < 8) lcnt[lane] = 0;
    asm volatile("s_waitcnt lgkmcnt(0)" ::: "memory");
  }
  __builtin_amdgcn_s_barrier();

  if (fwd) {
    const int j = b & 7, p = (q + (b >> 3) - 3) & 3;
    const int owner = ((side ^ 1) * 4 + p) & (NR - 1);
    char* const obase = P.out[owner];
    unsigned* const lnext = (unsigned*)(smem + L_CTRL + 16);
    const unsigned tl = lane < 3 ? lane : 0;
    const unsigned* const stgf = (const unsigned*)(myws + WS_STGF + ((p * 4 + ((j + tl) & 3)) * 8 + 2 * tl + (j >> 2)) * 128);
    const unsigned n0 = P.units[j >> 2][1 + p][3], n1 = P.units[2 + (j >> 2)][1 + p][3], n2 = P.units[4 + (j >> 2)][1 + p][3];
    const unsigned nl = (tl == 0 ? n0 : (tl == 1 ? n1 : n2)) - 1;
    while (__builtin_amdgcn_s_memrealtime() - t0 < FWD_T0) __builtin_amdgcn_s_sleep(16);
    unsigned fv = 0;
    bool open = false, quick = false;
    u64 tl0 = 0;
    while (wave <= FW) {
      const unsigned nx = __hip_atomic_load(lnext + tl, __ATOMIC_RELAXED, __HIP_MEMORY_SCOPE_WORKGROUP);
      const unsigned left = (unsigned)__builtin_amdgcn_ballot_w64(nx < nl) & 7u;
      if (left == 0) break;
      const unsigned rdy = bad ? left : (unsigned)__builtin_amdgcn_ballot_w64(fv > call * 16 + nx) & left;
      if (rdy == 0) {
        fv = __hip_atomic_load(stgf, __ATOMIC_RELAXED, __HIP_MEMORY_SCOPE_SYSTEM);
        if (give_up(t0, tmo, call)) bad = true;
        __builtin_amdgcn_s_sleep(2);
        continue;
      }
      const unsigned rdy2 = quick ? (unsigned)__builtin_amdgcn_ballot_w64(fv > call * 16 + nx + 1 && nx + 1 < nl) & 7u : 0u;
      const unsigned t = __builtin_ctz(rdy2 ? rdy2 : rdy);
      const unsigned k = __builtin_amdgcn_readlane(nx, t);
      unsigned got = k;
      if (lane == 0) __hip_atomic_compare_exchange_strong(lnext + t, &got, k + (rdy2 ? 2 : 1), __ATOMIC_RELAXED, __ATOMIC_RELAXED, __HIP_MEMORY_SCOPE_WORKGROUP);
      if (rfl(got) != k) continue;
      const unsigned g = 2 * t + (j >> 2), i = (j + t) & 3;
      const unsigned unit = (P.units[g][1 + p][k >> 2] >> ((k & 3) * 8)) & 127u;
      const unsigned unit2 = (P.units[g][1 + p][(k + 1) >> 2] >> (((k + 1) & 3) * 8)) & 127u;
      const char* const src = myws + WS_STG + (size_t)(((p * 4 + i) * 8 + g) * 12 + k) * 4096;
      f32x4 y0, y1, y2, y3, z0, z1, z2, z3;
      asm volatile("global_load_dwordx4 %0, %9, %10 sc0 sc1\n\t"
                   "global_load_dwordx4 %1, %9, %10 offset:1024 sc0 sc1\n\t"
                   "global_load_dwordx4 %2, %9, %10 offset:2048 sc0 sc1\n\t"
                   "global_load_dwordx4 %3, %9, %10 offset:3072 sc0 sc1\n\t"
                   "global_load_dwordx4 %4, %9, %11 sc0 sc1\n\t"
                   "global_load_dwordx4 %5, %9, %11 offset:1024 sc0 sc1\n\t"
                   "global_load_dwordx4 %6, %9, %11 offset:2048 sc0 sc1\n\t"
                   "global_load_dwordx4 %7, %9, %11 offset:3072 sc0 sc1\n\t"
                   "global_load_dword %8, %12, off sc0 sc1\n\t"
                   "s_waitcnt vmcnt(0)"
                   : "=&v"(y0), "=&v"(y1), "=&v"(y2), "=&v"(y3), "=&v"(z0), "=&v"(z1), "=&v"(z2), "=&v"(z3), "=&v"(fv)
                   : "v"(lane * 16), "s"(src), "s"(src + (rdy2 ? 4096 : 0)), "v"(stgf) : "memory");
      if (!open) {
        while (rfl(__hip_atomic_load(lfree, __ATOMIC_RELAXED, __HIP_MEMORY_SCOPE_WORKGROUP)) == 0) {
          if (__builtin_amdgcn_s_memrealtime() - t0 > 2 * DEADLINE) break;
          __builtin_amdgcn_s_sleep(2);
        }
        open = true;
        if (wave < 5) STAMP(6 + wave);
      }
      const u64 now = __builtin_amdgcn_s_memrealtime();
      quick = now - tl0 < FWD_QUICK;
      tl0 = now;
      STAMP(16 + t * 10 + k);
      if (bad) y0.x = __builtin_nanf("");
      auto put = [&](unsigned u, f32x4 v0, f32x4 v1, f32x4 v2, f32x4 v3) {
        const unsigned vo = ovoff + u * (32 * COLS * 4) + lent_strip(p, q, i) * 128;
        asm volatile("global_store_dwordx4 %0, %4, %8 sc0 sc1\n\t"
                     "global_store_dwordx4 %1, %5, %8 sc0 sc1\n\t"
                     "global_store_dwordx4 %2, %6, %8 sc0 sc1\n\t"
                     "global_store_dwordx4 %3, %7, %8 sc0 sc1\n\ts_nop 1"
                     :: "v"(vo), "v"(vo + 8 * COLS * 4), "v"(vo + 16 * COLS * 4), "v"(vo + 24 * COLS * 4),
                        "v"(v0), "v"(v1), "v"(v2), "v"(v3), "s"(obase) : "memory");
      };
      put(unit, y0, y1, y2, y3);
      if (rdy2) {
        STAMP(16 + t * 10 + k + 1);
        put(unit2, z0, z1, z2, z3);
      }
    }
    asm volatile("s_waitcnt vmcnt(0)" ::: "memory");
    unsigned* const dp = (unsigned*)(P.ws[owner] + WS_DONE + q * 256 + (j * 5 + wave - 1) * 4);
    if (lane == 0) __hip_atomic_store(dp, call, __ATOMIC_RELAXED, __HIP_MEMORY_SCOPE_SYSTEM);
    STAMP(10 + wave);
    return;
  }

  const int nq = (b & 3) * 24 + (b >> 3) - 8;
  const int g = ((b >> 2) & 1) * 3 + (nq >> 5), s = nq & 31;
  const bool foreign = s >= 16;
  const int owner = (foreign ? (side ^ 1) * 4 + ((s - 16) >> 2) : rank) & (NR - 1);
  const int cls = foreign ? 1 + (owner & 3) : 0;
  const unsigned ul0 = P.units[g][cls][0], ul1 = P.units[g][cls][1], ul2 = P.units[g][cls][2];
  const unsigned nu = P.units[g][cls][3] < 12u ? P.units[g][cls][3] : 12u;
  auto unit_at = [&](unsigned k) -> unsigned {
    unsigned wv = k < 4 ? ul0 : (k < 8 ? ul1 : ul2);
    return (wv >> ((k & 3) * 8)) & 127u;
  };

  if (wave == 1) {
    const int col0 = (foreign ? lent_strip(owner & 3, q, s & 3) : s) * 32;
    char* const obase = P.out[owner] + col0 * 4;
    const unsigned prod = (s & 15) * 8 + g;
    char* const stg = myws + WS_STG + (size_t)prod * (12 * 4096);
    unsigned* const stgf = (unsigned*)(myws + WS_STGF + prod * 128);
    unsigned* const donep = (unsigned*)(P.ws[owner] + WS_DONE + q * 256 + 160 + ((s & 3) * 6 + g) * 4);
    const char* const pl = smem + L_PART + rr * 128 + ((chn ^ rr) * 16);
    f32x4 keep[KEEP][4];
    unsigned kd = 0, ks = 0;
    while (ks < nu) {
      const u64 st = __hip_atomic_load((u64*)lcnt, __ATOMIC_RELAXED, __HIP_MEMORY_SCOPE_WORKGROUP);
      const unsigned have = rfl((unsigned)st), idle = rfl((unsigned)(st >> 32));
      if (__builtin_amdgcn_s_memrealtime() - t0 > 2 * DEADLINE) bad = true;
      if (have >= (kd + 1) * 4 || bad) {
        asm volatile("" ::: "memory");
        f32x4 y[4];
#pragma unroll
        for (int it = 0; it < 4; ++it) {
          f32x4 p0 = *(const f32x4*)(pl + it * 1024);
          f32x4 p1 = *(const f32x4*)(pl + it * 1024 + 4096);
          f32x4 p2 = *(const f32x4*)(pl + it * 1024 + 8192);
          f32x4 p3 = *(const f32x4*)(pl + it * 1024 + 12288);
          y[it] = (p0 + p1) + (p2 + p3);
        }
#pragma unroll
        for (int it = 0; it < 4; ++it) asm volatile("" : "+v"(y[it]));
        asm volatile("s_waitcnt lgkmcnt(0)" ::: "memory");
        if (lane == 0) __hip_atomic_store(lfree, kd + 1, __ATOMIC_RELAXED, __HIP_MEMORY_SCOPE_WORKGROUP);
#pragma unroll
        for (int it = 0; it < 4; ++it) {
          y[it].x = gelu(y[it].x); y[it].y = gelu(y[it].y); y[it].z = gelu(y[it].z); y[it].w = gelu(y[it].w);
          if (bad) y[it].x = __builtin_nanf("");
        }
#pragma unroll
        for (unsigned k = 0; k < KEEP; ++k)
          if (kd - ks == k) {
#pragma unroll
            for (int it = 0; it < 4; ++it) { keep[k][it] = y[it]; asm volatile("" : "+v"(keep[k][it])); }
          }
        ++kd;
      } else if (ks == kd || idle != kd + 1) {
        __builtin_amdgcn_s_sleep(1);
        continue;
      }
      if (kd - ks == KEEP || kd + 1 >= nu || idle == kd + 1) {
        const bool stage = foreign && kd < nu;
        char* const base = stage ? stg : obase;
        const unsigned step = stage ? 1024 : 8 * COLS * 4;
#pragma unroll
        for (unsigned k = 0; k < KEEP; ++k)
          if (ks + k < kd) {
            const unsigned vo = stage ? lane * 16 + (ks + k) * 4096 : ovoff + unit_at(ks + k) * (32 * COLS * 4);
#pragma unroll
            for (int it = 0; it < 4; ++it)
              asm volatile("global_store_dwordx4 %0, %1, %2 sc0 sc1\n\ts_nop 1"
                           :: "v"(vo + it * step), "v"(keep[k][it]), "s"(base) : "memory");
            STAMP(32 + ks + k);
          }
        ks = kd;
        if (stage) {
          asm volatile("s_waitcnt vmcnt(0)" ::: "memory");
          if (lane == 0) __hip_atomic_store(stgf, call * 16 + kd, __ATOMIC_RELAXED, __HIP_MEMORY_SCOPE_SYSTEM);
        }
      }
    }
    char* const lastp = obase + (size_t)unit_at(0) * (32 * COLS * 4);
    asm volatile("s_waitcnt vmcnt(0)" ::: "memory");
    STAMP(2);
    if (foreign) {
      if (lane == 0) __hip_atomic_store(donep, call, __ATOMIC_RELAXED, __HIP_MEMORY_SCOPE_SYSTEM);
    } else if (b == 64) {
      const char* dp = myws + WS_DONE + lane * 16;
      while (true) {
        u32x4 v;
        asm volatile("global_load_dwordx4 %0, %1, off sc0 sc1\n\ts_waitcnt vmcnt(0)" : "=&v"(v) : "v"(dp) : "memory");
        bool ok = v.x >= call && v.y >= call && v.z >= call && v.w >= call;
        if (__all(ok)) break;
        if (give_up(t0, tmo, call)) {
          if (lane == 0) __hip_atomic_store((float*)lastp, __builtin_nanf(""), __ATOMIC_RELAXED, __HIP_MEMORY_SCOPE_SYSTEM);
          break;
        }
        __builtin_amdgcn_s_sleep(1);
      }
      STAMP(3);
    }
    return;
  }

  const int w = wave - 2;
  char* const data = myws + WS_DATA + (size_t)(call & 1) * PAR_BYTES;
  const unsigned* const flags = (const unsigned*)(myws + WS_FLAG) + w * 8 + (lane & 7);
  lds_char* const lw = (lds_char*)smem + w * 32768;
  lds_char* const ltr = lw + ((lane >> 5) * 8 + ((lane & 15) >> 2)) * 64 + ((lane >> 4) & 1) * 32 + (lane & 3) * 8;
  h8 wr[32];
  {
    const unsigned* pp = flags + (128 + s) * 32;
    const char* src = data + (size_t)(128 + s) * OBJ + w * 16384;
    const unsigned svoff = lane * 16;
    const unsigned lwa = (unsigned)(size_t)lw;
    unsigned seen = 0;
#pragma unroll
    for (int r = 0; r < 4; ++r) {
      wait_round(pp, r, seen, bad, t0, tmo, call, lidle, 0);
#pragma unroll
      for (int i = 0; i < 16; ++i)
        glds16(src + r * 65536 + i * 1024, svoff, lwa + (r & 1) * 16384 + i * 1024);
      if (r & 1) asm volatile("s_waitcnt vmcnt(0)" ::: "memory");
      if (r == 1) {
#pragma unroll
        for (int i = 0; i < 32; ++i) wr[i] = tr_frag(ltr + i * 1024);
#pragma unroll
        for (int i = 0; i < 32; ++i) asm volatile("" : "+v"(wr[i]));
        asm volatile("s_waitcnt lgkmcnt(0)" ::: "memory");
      }
    }
  }
  if (w == 0) STAMP(4);
  const __amdgpu_buffer_rsrc_t rsrc =
      __builtin_amdgcn_make_buffer_rsrc(data, 0, 128 * OBJ, 0x00020000);
  const unsigned xvoff = w * 16384 + lane * 16;
  char* const pw = smem + L_PART + w * 4096 + (lane & 31) * 128;
  unsigned nextf = 0;
  for (unsigned k = 0; k < nu; ++k) {
    const unsigned e = unit_at(k);
    const unsigned* pp = flags + e * 32;
    const unsigned* pn = flags + unit_at(k + 1 < nu ? k + 1 : k) * 32;
    const unsigned soff = e * OBJ;
    unsigned seen = flag_mask(nextf, call);
    f32x16 acc = {0.f, 0.f, 0.f, 0.f, 0.f, 0.f, 0.f, 0.f, 0.f, 0.f, 0.f, 0.f, 0.f, 0.f, 0.f, 0.f};
#pragma unroll
    for (int r = 0; r < 4; ++r) {
      wait_round(pp, r, seen, bad, t0, tmo, call, lidle, k + 1);
      u32x4 xa[16];
#pragma unroll
      for (int i = 0; i < 16; ++i)
        xa[i] = __builtin_amdgcn_raw_buffer_load_b128(rsrc, xvoff + (i & 3) * 1024, soff + r * 65536 + (i >> 2) * 4096, 17);
      __builtin_amdgcn_sched_barrier(0);
      if (r == 3) { nextf = flag_load(pn); __builtin_amdgcn_sched_barrier(0); }
#pragma unroll
      for (int i = 0; i < 16; ++i) {
        h8 wf = r < 2 ? wr[r * 16 + i] : tr_frag(ltr + ((r - 2) * 16 + i) * 1024);
        acc = __builtin_amdgcn_mfma_f32_32x32x16_f16(wf, __builtin_bit_cast(h8, xa[i]), acc, 0, 0, 0);
      }
    }
    if (bad) {
#pragma unroll
      for (int i = 0; i < 16; ++i) acc[i] = __builtin_nanf("");
    }
    while (rfl(__hip_atomic_load(lfree, __ATOMIC_RELAXED, __HIP_MEMORY_SCOPE_WORKGROUP)) < k) {
      if (__builtin_amdgcn_s_memrealtime() - t0 > 2 * DEADLINE) break;
      __builtin_amdgcn_s_sleep(1);
    }
    asm volatile("" ::: "memory");
#pragma unroll
    for (int gq = 0; gq < 4; ++gq) {
      f32x4 v = {acc[gq * 4], acc[gq * 4 + 1], acc[gq * 4 + 2], acc[gq * 4 + 3]};
      *(f32x4*)(pw + (((gq * 2 + (lane >> 5)) ^ (lane & 7)) * 16)) = v;
    }
    asm volatile("s_waitcnt lgkmcnt(0)" ::: "memory");
    if (lane == 0) __hip_atomic_fetch_add(lcnt, 1u, __ATOMIC_RELAXED, __HIP_MEMORY_SCOPE_WORKGROUP);
    if (w == 0) STAMP(16 + k);
  }
}

unsigned link_map(int rank) {
  const unsigned dflt = 0x76543210u;
  int dev = -1, n = 0, node[NR], port[NR + 1];
  if (hipGetDevice(&dev) != hipSuccess || hipGetDeviceCount(&n) != hipSuccess || dev != rank || n != NR) return dflt;
  for (int i = 0; i <= NR; ++i) port[i] = -1;
  for (int d = 0; d < NR; ++d) {
    char id[32], path[96];
    if (hipDeviceGetPCIBusId(id, sizeof id, d) != hipSuccess) return dflt;
    for (char* c = id; *c; ++c) if (*c >= 'A' && *c <= 'F') *c += 'a' - 'A';
    snprintf(path, sizeof path, "/sys/bus/pci/devices/%s/xgmi_port_num", id);
    FILE* f = fopen(path, "r");
    if (!f) return dflt;
    unsigned a, pa, z, pz;
    node[d] = -1;
    while (fscanf(f, " %x:%x -> %x:%x", &a, &pa, &z, &pz) == 4) {
      node[d] = (int)a;
      if (d == rank && z <= (unsigned)NR && pa < 8u) port[z] = (int)pa;
    }
    fclose(f);
    if (node[d] < 0 || node[d] > NR) return dflt;
  }
  unsigned map = 0, used = 0;
  const int side = rank >> 2, q = rank & 3;
  for (int c = 0; c < 7; ++c) {
    const int x = port[node[c < 3 ? side * 4 + ((q + 1 + c) & 3) : (side ^ 1) * 4 + ((q + c - 3) & 3)]];
    if (x < 0 || ((used >> x) & 1)) return dflt;
    used |= 1u << x;
    map |= (unsigned)c << (4 * x);
  }
  return map | 7u << (4 * __builtin_ctz(~used));
}

}

static volatile unsigned long long* g_diag; static int g_rank;
__attribute__((destructor)) static void dump() {
  if (!g_diag) return;
  static unsigned long long h[256 * 64];
  for (int i = 0; i < 256 * 64; ++i) h[i] = g_diag[i];
  unsigned long long t0 = ~0ull;
  for (int b = 0; b < 256; ++b) if (h[b * 64 + 5]) t0 = std::min(t0, h[b * 64 + 5]);
  auto T = [&](int b, int i) { return h[b * 64 + i] >= t0 ? (h[b * 64 + i] - t0) * 0.01 : -1.0; };
  char buf[4096]; int n = 0;
  double e = 0, lg = 0; for (int b = 64; b < 256; ++b) { e = std::max(e, T(b, 2)); for (int k = 0; k < 12; ++k) lg = std::max(lg, T(b, 16 + k)); }
  n += sprintf(buf + n, "DIAGDBG rank %d t0 %llu: xcd starts", g_rank, t0 % 10000000);
  for (int x = 0; x < 8; ++x) n += sprintf(buf + n, " %.1f", T(64 + x, 5));
  n += sprintf(buf + n, " | push");
  for (int c = 0; c < 7; ++c) { double a = 0, en = 0; for (int j = 0; j < 8; ++j) { a = std::max(a, T(c * 8 + j, 0)); en = std::max(en, T(c * 8 + j, 1)); } n += sprintf(buf + n, " [%.1f %.1f]", a, en); }
  { double en = 0; for (int b = 64; b < 192; ++b) en = std::max(en, T(b, 1)); n += sprintf(buf + n, " loc %.1f", en); }
  n += sprintf(buf + n, " | fwd");
  for (int c = 3; c < 7; ++c) { double en = 0; for (int j = 0; j < 8; ++j) for (int w = 1; w < 5; ++w) en = std::max(en, T(c * 8 + j, 10 + w)); n += sprintf(buf + n, " %.1f", en); }
  n += sprintf(buf + n, " | W first:");
  for (int s = 16; s < 32; s += 4) { int nq = s; int b = ((nq % 24) + 8) * 8 + nq / 24; n += sprintf(buf + n, " %.1f", T(b, 4)); }
  n += sprintf(buf + n, " | gemm %.1f epi %.1f final %.1f\n", lg, e, T(64, 3));
  for (int c = 0; c < 7; c += 3) {
    n += sprintf(buf + n, "DIAGDBG r%d push c%d unit ends:", g_rank, c);
    for (int k = 0; k < 11; ++k) { double m = -1; for (int j = 0; j < 8; ++j) m = std::max(m, T(c * 8 + j, 48 + k)); n += sprintf(buf + n, " %.1f", m); }
    n += sprintf(buf + n, "\n");
  }
  fputs(buf, stderr);
}
extern "C" void kernel_launch(int rank, int nranks, ncclComm_t comm,
                              hipStream_t stream,
                              void* const* d_in, const int* in_sizes, int n_in,
                              void* d_out, int out_size,
                              const int* in_dtypes, int out_dtype,
                              void* const* d_out_all,
                              void* const* d_ws_all, size_t ws_bytes) {
  (void)comm; (void)in_dtypes; (void)out_dtype; (void)d_out;
  if (nranks != NR || (unsigned)rank >= (unsigned)NR || n_in < 2 || ws_bytes < WS_NEED ||
      in_sizes[0] != ROWS * KDIM || in_sizes[1] != KDIM * COLS || out_size != NR * ROWS * COLS) {
    fprintf(stderr, "kernel_launch: unexpected problem shape\n");
    return;
  }
  Params P;
  P.x = (const char*)d_in[0];
  P.w = (const char*)d_in[1];
  for (int p = 0; p < NR; ++p) { P.ws[p] = (char*)d_ws_all[p]; P.out[p] = (char*)d_out_all[p]; }
  P.rank = rank;
  g_diag = (volatile unsigned long long*)(P.ws[rank] + WS_DIAG); g_rank = rank;
  P.xmap = link_map(rank);
  const int side = rank >> 2, q = rank & 3;
  static const unsigned char order[NG][12] = {
    {60, 61, 0, 6, 12, 40, 46, 18, 24, 52, 30, 99},
    {62, 63, 1, 7, 13, 41, 47, 19, 25, 53, 31, 99},
    {64, 65, 72, 2, 8, 14, 42, 20, 48, 26, 54, 32},
    {66, 67, 73, 3, 9, 43, 15, 21, 49, 55, 27, 33},
    {68, 69, 74, 4, 10, 16, 44, 22, 50, 28, 34, 99},
    {70, 71, 75, 5, 11, 17, 45, 23, 51, 29, 35, 99}};
  for (int g = 0; g < NG; ++g)
    for (int cls = 0; cls < 5; ++cls) {
      unsigned u[12] = {0}, n = 0;
      for (int k = 0; k < 12; ++k) {
        int v = order[g][k];
        if (v == 99) break;
        if (cls > 0 && (v >= 60 + NA || (v >= 40 && v < 60 && ((v - 40) & 3) == cls - 1))) continue;
        if (v < 40) u[n++] = (side * 4 + ((q + 1 + v % 3) & 3)) * 16 + v / 3;
        else if (v < 60) u[n++] = ((side ^ 1) * 4 + ((v - 40) & 3)) * 16 + NA + ((v - 40) >> 2);
        else u[n++] = rank * 16 + v - 60;
      }
      for (int k = 0; k < 3; ++k)
        P.units[g][cls][k] = u[4 * k] | (u[4 * k + 1] << 8) | (u[4 * k + 2] << 16) | (u[4 * k + 3] << 24);
      P.units[g][cls][3] = n;
    }
  hipLaunchKernelGGL(ag_gemm_gelu, dim3(256), dim3(384), 0, stream, P);
  HIP_CHECK(hipGetLastError());
}
```

```cpp
#include <hip/hip_runtime.h>
#include <rccl/rccl.h>
#include <cstdio>

#define HIP_CHECK(expr) do { hipError_t _e = (expr); if (_e != hipSuccess) { \
    fprintf(stderr, "HIP error %s at %s:%d\n", hipGetErrorString(_e), __FILE__, __LINE__); \
    return; } } while (0)

namespace {

typedef unsigned u32x4 __attribute__((ext_vector_type(4)));
typedef float f32x4 __attribute__((ext_vector_type(4)));
typedef float f32x16 __attribute__((ext_vector_type(16)));
typedef _Float16 h2 __attribute__((ext_vector_type(2)));
typedef _Float16 h8 __attribute__((ext_vector_type(8)));
typedef short s4 __attribute__((ext_vector_type(4)));
typedef short s8 __attribute__((ext_vector_type(8)));
typedef unsigned long long u64;

constexpr int NR = 8;
constexpr int KDIM = 4096;
constexpr int ROWS = 512;
constexpr int COLS = 512;
constexpr unsigned OBJ = 256u * 1024u;
constexpr int NG = 6;
constexpr int NA = 12;
constexpr size_t WS_CNT = 0;
constexpr size_t WS_TMO = 16 * 1024;
constexpr size_t WS_TICK = 20 * 1024;
constexpr size_t WS_FLAG = 32 * 1024;
constexpr size_t WS_DONE = 64 * 1024;
constexpr size_t WS_STGF = 80 * 1024;
constexpr size_t WS_DIAG = 128 * 1024;
constexpr size_t WS_DATA = 1024 * 1024;
constexpr unsigned NSLOT = 160;
constexpr size_t PAR_BYTES = (size_t)NSLOT * OBJ;
constexpr size_t WS_STG = WS_DATA + 2 * PAR_BYTES;
constexpr size_t WS_NEED = WS_STG + 128 * 12 * 4096;
constexpr u64 DEADLINE = 100000000ull;
#define FWD_T0 2500
#define SMODE 1
__host__ __device__ constexpr int lent_strip(int o, int h, int i) {
  return SMODE == 0 ? h * 4 + i : (SMODE == 1 ? ((o + h) & 3) * 4 + i : i * 4 + ((o + h) & 3));
}
#define FW 5
#define KA 4
#define KC 4
#define FWD_QUICK 0
#define RELAY 1
#define TOUCH 3
#define KEEP 4
constexpr unsigned L_PART = 128 * 1024;
constexpr unsigned L_CTRL = L_PART + 16 * 1024;
constexpr unsigned L_TOTAL = L_CTRL + 64;

struct Params {
  const char* x;
  const char* w;
  char* ws[NR];
  char* out[NR];
  unsigned units[NG][5][4];
  int rank;
  unsigned xmap;
};

__device__ __forceinline__ unsigned rfl(unsigned v) { return __builtin_amdgcn_readfirstlane(v); }
__device__ __forceinline__ unsigned pk(float a, float b) {
  h2 v; v.x = (_Float16)a; v.y = (_Float16)b; return __builtin_bit_cast(unsigned, v);
}
__device__ __forceinline__ u32x4 cvt8(u32x4 a, u32x4 b) {
  f32x4 fa = __builtin_bit_cast(f32x4, a), fb = __builtin_bit_cast(f32x4, b);
  u32x4 r; r.x = pk(fa.x, fa.y); r.y = pk(fa.z, fa.w); r.z = pk(fb.x, fb.y); r.w = pk(fb.z, fb.w);
  return r;
}
__device__ __forceinline__ float gelu(float y) {
  const float c2 = -2.0f * 0.7978845608028654f * 1.4426950408889634f;
  float u = y + 0.044715f * y * y * y;
  float e = __builtin_amdgcn_exp2f(c2 * u);
  return y * __builtin_amdgcn_rcpf(1.0f + e);
}

#define STAMPV(idx, val) do {} while (0)
#define STAMP(idx) STAMPV(idx, __builtin_amdgcn_s_memrealtime())

__device__ __forceinline__ bool give_up(u64 t0, unsigned* tmo, unsigned call) {
  u64 el = __builtin_amdgcn_s_memrealtime() - t0;
  if (el < 100000ull) return false;
  if (el > DEADLINE ||
      rfl(__hip_atomic_load(tmo, __ATOMIC_RELAXED, __HIP_MEMORY_SCOPE_AGENT)) == call) {
    __hip_atomic_store(tmo, call, __ATOMIC_RELAXED, __HIP_MEMORY_SCOPE_AGENT);
    return true;
  }
  return false;
}

__device__ __forceinline__ unsigned flag_load(const unsigned* p) {
  return __hip_atomic_load(p, __ATOMIC_RELAXED, __HIP_MEMORY_SCOPE_SYSTEM);
}
__device__ __forceinline__ unsigned flag_mask(unsigned v, unsigned call) {
  return (unsigned)__builtin_amdgcn_ballot_w64(v >= call) & 0xFFu;
}
__device__ __forceinline__ void wait_round(const unsigned* p, int r, unsigned& seen, bool& bad, u64 t0,
                                           unsigned* tmo, unsigned call, unsigned* lidle, unsigned mark) {
  while (((seen >> (2 * r)) & 3u) != 3u) {
    seen = flag_mask(flag_load(p), call);
    if (((seen >> (2 * r)) & 3u) == 3u) break;
    if (r < 3) __hip_atomic_store(lidle, mark, __ATOMIC_RELAXED, __HIP_MEMORY_SCOPE_WORKGROUP);
    if (give_up(t0, tmo, call)) { bad = true; break; }
    __builtin_amdgcn_s_sleep(2);
  }
  asm volatile("" ::: "memory");
}

struct Piece { u32x4 r[16]; };

__device__ __forceinline__ void piece_load(Piece& s, const char* sb, unsigned fstride, unsigned voff) {
#pragma unroll
  for (int u = 0; u < 8; ++u) {
    const char* p = sb + (size_t)u * fstride;
    asm volatile("global_load_dwordx4 %0, %1, %2" : "=v"(s.r[2 * u]) : "v"(voff), "s"(p) : "memory");
    asm volatile("global_load_dwordx4 %0, %1, %2 offset:16" : "=v"(s.r[2 * u + 1]) : "v"(voff), "s"(p) : "memory");
  }
}
template <int N>
__device__ __forceinline__ void piece_wait(Piece& s) {
  asm volatile("s_waitcnt vmcnt(%0)" :: "i"(N) : "memory");
#pragma unroll
  for (int u = 0; u < 16; ++u) asm volatile("" : "+v"(s.r[u]));
}
template <int U>
__device__ __forceinline__ void piece_store1(Piece& s, char* db, unsigned voff) {
  u32x4 v = cvt8(s.r[2 * U], s.r[2 * U + 1]);
  asm volatile("global_store_dwordx4 %0, %1, %2 offset:%3 sc0 sc1\n\ts_nop 1"
               :: "v"(voff), "v"(v), "s"(db), "i"((U - 4) * 1024) : "memory");
}
__device__ __forceinline__ void piece_store(Piece& s, char* db  , unsigned voff) {
  piece_store1<0>(s, db, voff); piece_store1<1>(s, db, voff); piece_store1<2>(s, db, voff);
  piece_store1<3>(s, db, voff); piece_store1<4>(s, db, voff); piece_store1<5>(s, db, voff);
  piece_store1<6>(s, db, voff); piece_store1<7>(s, db, voff);
}

typedef __attribute__((address_space(3))) char lds_char;

__device__ __forceinline__ void glds16(const char* src, unsigned voff, unsigned lds) {
  unsigned keep;
  asm volatile("s_mov_b32 %0, m0\n\ts_mov_b32 m0, %3\n\ts_nop 0\n\t"
               "global_load_lds_dwordx4 %1, %2 sc0 sc1\n\ts_mov_b32 m0, %0"
               : "=&s"(keep) : "v"(voff), "s"(src), "s"(lds) : "memory");
}

__device__ __forceinline__ h8 tr_frag(lds_char* p) {
  s4 a = __builtin_amdgcn_ds_read_tr16_b64_v4i16((__attribute__((address_space(3))) s4*)p);
  s4 c = __builtin_amdgcn_ds_read_tr16_b64_v4i16((__attribute__((address_space(3))) s4*)(p + 256));
  s8 r = __builtin_shufflevector(a, c, 0, 1, 2, 3, 4, 5, 6, 7);
  return __builtin_bit_cast(h8, r);
}

__global__ void __launch_bounds__(384) ag_gemm_gelu(Params P) {
  __shared__ __attribute__((aligned(1024))) char smem[L_TOTAL];
  const int lane = threadIdx.x & 63;
  const int wave = rfl(threadIdx.x >> 6);
  const int xch = (P.xmap >> ((__builtin_amdgcn_s_getreg(20 | (3 << 11)) & 7) * 4)) & 7;
  const int b = blockIdx.x < 64 ? xch * 8 + (blockIdx.x >> 3) : blockIdx.x;
  const int rank = P.rank & (NR - 1);
  const int side = rank >> 2, q = rank & 3;
  char* const myws = P.ws[rank];
  unsigned* const cntp = (unsigned*)(myws + WS_CNT + b * 32 + wave * 4);
  unsigned* const lcnt = (unsigned*)(smem + L_CTRL);
  unsigned* const lidle = (unsigned*)(smem + L_CTRL + 4);
  unsigned* const lfree = (unsigned*)(smem + L_CTRL + 8);

  if (wave == 0) {
    if (b >= 64 + 128 || (b >> 3) == 7) {
      const int c = xch;
      if ((TOUCH & 1) && (b >> 3) == 24 && c < 7) {
        const int dest = (c < 3 ? side * 4 + ((q + 1 + c) & 3) : (side ^ 1) * 4 + ((q + c - 3) & 3)) & (NR - 1);
        const char* const dd = P.ws[dest] + WS_DATA + (lane & 1) * PAR_BYTES;
        const char* a1 = lane < 2 ? dd + (size_t)(c < 3 ? rank * 16 : 144 + q * 4) * OBJ : P.ws[dest] + WS_FLAG;
        const char* a2 = lane < 32 ? dd + (size_t)(rank * 16 + (lane >> 1)) * OBJ : P.out[dest] + (size_t)(lane - 32) * OBJ;
        unsigned v1, v2;
        asm volatile("global_load_dword %0, %2, off sc0 sc1\n\tglobal_load_dword %1, %3, off sc0 sc1\n\ts_waitcnt vmcnt(0)"
                     : "=&v"(v1), "=&v"(v2) : "v"(a1), "v"(a2) : "memory");
      }
      return;
    }
    __builtin_amdgcn_s_setprio(3);
    __builtin_amdgcn_s_waitcnt(0);
    const int j = b & 7, c = b >> 3;
    int dest, nW, strip0, wslot0, xu0, nobj;
    unsigned nt, skip;
    if (c < 3)      { dest = side * 4 + ((q + 1 + c) & 3); nW = 0; strip0 = 0; wslot0 = 0; xu0 = 0; nobj = NA;
                      nt = NA * 16 - (RELAY ? 8 : 0); skip = RELAY ? 16 : 0; }
    else if (c < 7) { dest = (side ^ 1) * 4 + ((q + c - 3) & 3); nW = 4; strip0 = lent_strip(q, dest & 3, 0); wslot0 = 16 + q * 4;
                      xu0 = NA; nobj = 4 + 16 - NA + 1; nt = (nobj - 1) * 16 + (RELAY ? 2 : 0); skip = (dest & 3) * 2; }
    else            { dest = rank; nW = 1; strip0 = c - 8; wslot0 = c - 8; xu0 = c - 8; nobj = 2; nt = 32; skip = 0; }
    dest &= NR - 1;
    const unsigned np = nobj * 32;
    auto xunit = [&](int obj) -> int { const int u = xu0 + obj - nW; return u < 16 ? u : NA - 1; };
    const unsigned laneX = (lane & 31) * (KDIM * 4) + (lane >> 5) * 32;
    const unsigned laneW = (lane >> 2) * (COLS * 4) + (lane & 3) * 32;
    const unsigned lane16 = lane * 16;
    const char* px = P.x;
    const char* pw = P.w;
    asm volatile("" : "+s"(px), "+s"(pw));
    auto load = [&](Piece& s, unsigned pc) {
      pc = pc < np ? pc : np - 1;
      const int obj = pc >> 5, f0 = (pc & 31) * 8;
      const bool isW = obj < nW;
      const char* sb = isW ? pw + (size_t)(strip0 + obj * (lent_strip(0, 0, 1) - lent_strip(0, 0, 0))) * 128 + (size_t)f0 * (16 * COLS * 4)
                           : px + (size_t)xunit(obj) * (32 * KDIM * 4) + (size_t)f0 * 64;
      piece_load(s, sb, isW ? 16 * COLS * 4 : 64, isW ? laneW : laneX);
    };
    unsigned cnt;
    asm volatile("global_load_dword %0, %1, off sc1" : "=v"(cnt) : "v"(cntp) : "memory");
    Piece A, B;
    load(A, j);
    load(B, j + 8);
    __builtin_amdgcn_s_barrier();
    piece_wait<16>(A);
    asm volatile("" : "+v"(cnt));
    const unsigned call = rfl(cnt) + 1;
    char* const dws = P.ws[dest];
    char* const ddata = dws + WS_DATA + (size_t)(call & 1) * PAR_BYTES + 4096;
    char* const tick = myws + WS_TICK + c * 128;
    const unsigned tbase = 16 - (call - 1) * nt;
    auto slot_of = [&](unsigned pc) -> unsigned {
      const int obj = pc >> 5;
      return obj < nW ? 128 + wslot0 + obj : rank * 16 + xunit(obj);
    };
    auto store = [&](Piece& s, unsigned pc) {
      piece_store(s, ddata + (size_t)slot_of(pc) * OBJ + (pc & 31) * 8192, lane16);
    };
    const unsigned zero = 0, one = 1;
    auto flag = [&](unsigned pc) {
      char* fp = dws + WS_FLAG + slot_of(pc) * 128 + (((pc >> 1) & 3) * 8 + ((pc >> 3) & 3) * 2 + (pc & 1)) * 4;
      if (lane == 0) asm volatile("global_store_dword %0, %1, %2 sc0 sc1" :: "v"(zero), "v"(call), "s"(fp) : "memory");
    };
    auto ticket = [&](unsigned& t) {
      if (lane == 0) asm volatile("global_atomic_add %0, %1, %2, %3 sc0" : "=v"(t) : "v"(zero), "v"(one), "s"(tick) : "memory");
    };
    STAMP(0);
    auto piece_of = [&](unsigned t) -> unsigned { return (t >> 3) * 16 + (t & 7) + (t < np / 2 - 16 ? 0 : skip); };
    unsigned tk = 0;
    unsigned cur = j, nxt = j + 16;
    store(A, cur);
    ticket(tk);
    load(A, nxt);
    piece_wait<25>(B);
    if (lane == 0) asm volatile("global_store_dword %0, %1, off sc1" :: "v"(cntp), "v"(call) : "memory");
    store(B, cur + 8);
    load(B, nxt + 8);
    do {
      piece_wait<25>(A);
      asm volatile("" : "+v"(tk));
      flag(cur);
      unsigned t = rfl(tk) + tbase;
      const unsigned nn = t < nt ? piece_of(t) : np;
      if (c < 3) asm volatile("s_waitcnt vmcnt(%0)" :: "i"(17 + KA) : "memory");
      else if (c < 7) asm volatile("s_waitcnt vmcnt(%0)" :: "i"(17 + KC) : "memory");
      store(A, nxt);
      ticket(tk);
      load(A, nn);
      piece_wait<26>(B);
      flag(cur + 8);
      if (c < 3) asm volatile("s_waitcnt vmcnt(%0)" :: "i"(18 + KA) : "memory");
      else if (c < 7) asm volatile("s_waitcnt vmcnt(%0)" :: "i"(18 + KC) : "memory");
      store(B, nxt + 8);
      load(B, nn + 8);
      cur = nxt;
      nxt = nn;
    } while (nxt < np);
    if (b < 64) __hip_atomic_store(lfree, 1u, __ATOMIC_RELAXED, __HIP_MEMORY_SCOPE_WORKGROUP);
    piece_wait<25>(A);
    asm volatile("" : "+v"(tk));
    flag(cur);
    piece_wait<0>(B);
    flag(cur + 8);
    STAMP(1);
    return;
  }

  const bool fwd = b < 64;
  if (fwd && (b < 24 || b >= 56)) return;
  const u64 t0 = __builtin_amdgcn_s_memrealtime();
  const unsigned call = rfl(__hip_atomic_load(cntp, __ATOMIC_RELAXED, __HIP_MEMORY_SCOPE_AGENT)) + 1;
  if (lane == 0) __hip_atomic_store(cntp, call, __ATOMIC_RELAXED, __HIP_MEMORY_SCOPE_AGENT);
  unsigned* const tmo = (unsigned*)(myws + WS_TMO);
  const int rr = lane >> 3, chn = lane & 7;
  const unsigned ovoff = rr * (COLS * 4) + chn * 16;
  bool bad = false;
  if (wave == 1) {
    STAMPV(5, t0);
    if (lane < 8) lcnt[lane] = 0;
    asm volatile("s_waitcnt lgkmcnt(0)" ::: "memory");
  }
  __builtin_amdgcn_s_barrier();

  if (fwd) {
    const int j = b & 7, p = (q + (b >> 3) - 3) & 3;
    const int owner = ((side ^ 1) * 4 + p) & (NR - 1);
    char* const obase = P.out[owner];
    unsigned* const lnext = (unsigned*)(smem + L_CTRL + 16);
    const unsigned tl = lane < 3 ? lane : 0;
    const unsigned* const stgf = (const unsigned*)(myws + WS_STGF + ((p * 4 + ((j + tl) & 3)) * 8 + 2 * tl + (j >> 2)) * 128);
    const unsigned n0 = P.units[j >> 2][1 + p][3], n1 = P.units[2 + (j >> 2)][1 + p][3], n2 = P.units[4 + (j >> 2)][1 + p][3];
    const unsigned nl = (tl == 0 ? n0 : (tl == 1 ? n1 : n2)) - 1;
    while (__builtin_amdgcn_s_memrealtime() - t0 < FWD_T0) __builtin_amdgcn_s_sleep(16);
    if (RELAY && wave == 5) {
      for (int n = j; n < 12; n += 8) {
        const unsigned slot = ((side ^ 1) * 4 + ((p + 1 + (n >> 2)) & 3)) * 16 + NA - 1;
        const size_t fo = WS_FLAG + slot * 128 + (q * 8 + (n & 3)) * 4;
        const size_t go = WS_DATA + (size_t)(call & 1) * PAR_BYTES + (size_t)slot * OBJ + (q * 2 + (n & 1) + (n & 2) * 4) * 8192 + 4096;
        while (rfl(flag_load((const unsigned*)(myws + fo))) < call) {
          if (give_up(t0, tmo, call)) { bad = true; break; }
          __builtin_amdgcn_s_sleep(8);
        }
        if (bad) break;
        u32x4 y0, y1, y2, y3, y4, y5, y6, y7;
        asm volatile("global_load_dwordx4 %0, %8, %9 offset:-4096 sc0 sc1\n\t"
                     "global_load_dwordx4 %1, %8, %9 offset:-3072 sc0 sc1\n\t"
                     "global_load_dwordx4 %2, %8, %9 offset:-2048 sc0 sc1\n\t"
                     "global_load_dwordx4 %3, %8, %9 offset:-1024 sc0 sc1\n\t"
                     "global_load_dwordx4 %4, %8, %9 sc0 sc1\n\t"
                     "global_load_dwordx4 %5, %8, %9 offset:1024 sc0 sc1\n\t"
                     "global_load_dwordx4 %6, %8, %9 offset:2048 sc0 sc1\n\t"
                     "global_load_dwordx4 %7, %8, %9 offset:3072 sc0 sc1\n\t"
                     "s_waitcnt vmcnt(0)\n\t"
                     "global_store_dwordx4 %8, %0, %10 offset:-4096 sc0 sc1\n\t"
                     "global_store_dwordx4 %8, %1, %10 offset:-3072 sc0 sc1\n\t"
                     "global_store_dwordx4 %8, %2, %10 offset:-2048 sc0 sc1\n\t"
                     "global_store_dwordx4 %8, %3, %10 offset:-1024 sc0 sc1\n\t"
                     "global_store_dwordx4 %8, %4, %10 sc0 sc1\n\t"
                     "global_store_dwordx4 %8, %5, %10 offset:1024 sc0 sc1\n\t"
                     "global_store_dwordx4 %8, %6, %10 offset:2048 sc0 sc1\n\t"
                     "global_store_dwordx4 %8, %7, %10 offset:3072 sc0 sc1\n\t"
                     "s_waitcnt vmcnt(0)"
                     : "=&v"(y0), "=&v"(y1), "=&v"(y2), "=&v"(y3), "=&v"(y4), "=&v"(y5), "=&v"(y6), "=&v"(y7)
                     : "v"(lane * 16), "s"(myws + go), "s"(P.ws[owner] + go) : "memory");
        if (lane == 0) __hip_atomic_store((unsigned*)(P.ws[owner] + fo), call, __ATOMIC_RELAXED, __HIP_MEMORY_SCOPE_SYSTEM);
        STAMP(46 + (n >> 3));
      }
    }
    unsigned fv = 0;
    bool open = false, quick = false;
    u64 tl0 = 0;
    while (wave <= FW) {
      const unsigned nx = __hip_atomic_load(lnext + tl, __ATOMIC_RELAXED, __HIP_MEMORY_SCOPE_WORKGROUP);
      const unsigned left = (unsigned)__builtin_amdgcn_ballot_w64(nx < nl) & 7u;
      if (left == 0) break;
      const unsigned rdy = bad ? left : (unsigned)__builtin_amdgcn_ballot_w64(fv > call * 16 + nx) & left;
      if (rdy == 0) {
        fv = __hip_atomic_load(stgf, __ATOMIC_RELAXED, __HIP_MEMORY_SCOPE_SYSTEM);
        if (give_up(t0, tmo, call)) bad = true;
        __builtin_amdgcn_s_sleep(2);
        continue;
      }
      const unsigned rdy2 = quick ? (unsigned)__builtin_amdgcn_ballot_w64(fv > call * 16 + nx + 1 && nx + 1 < nl) & 7u : 0u;
      const unsigned t = __builtin_ctz(rdy2 ? rdy2 : rdy);
      const unsigned k = __builtin_amdgcn_readlane(nx, t);
      unsigned got = k;
      if (lane == 0) __hip_atomic_compare_exchange_strong(lnext + t, &got, k + (rdy2 ? 2 : 1), __ATOMIC_RELAXED, __ATOMIC_RELAXED, __HIP_MEMORY_SCOPE_WORKGROUP);
      if (rfl(got) != k) continue;
      const unsigned g = 2 * t + (j >> 2), i = (j + t) & 3;
      const unsigned unit = (P.units[g][1 + p][k >> 2] >> ((k & 3) * 8)) & 127u;
      const unsigned unit2 = (P.units[g][1 + p][(k + 1) >> 2] >> (((k + 1) & 3) * 8)) & 127u;
      const char* const src = myws + WS_STG + (size_t)(((p * 4 + i) * 8 + g) * 12 + k) * 4096;
      f32x4 y0, y1, y2, y3, z0, z1, z2, z3;
      asm volatile("global_load_dwordx4 %0, %9, %10 sc0 sc1\n\t"
                   "global_load_dwordx4 %1, %9, %10 offset:1024 sc0 sc1\n\t"
                   "global_load_dwordx4 %2, %9, %10 offset:2048 sc0 sc1\n\t"
                   "global_load_dwordx4 %3, %9, %10 offset:3072 sc0 sc1\n\t"
                   "global_load_dwordx4 %4, %9, %11 sc0 sc1\n\t"
                   "global_load_dwordx4 %5, %9, %11 offset:1024 sc0 sc1\n\t"
                   "global_load_dwordx4 %6, %9, %11 offset:2048 sc0 sc1\n\t"
                   "global_load_dwordx4 %7, %9, %11 offset:3072 sc0 sc1\n\t"
                   "global_load_dword %8, %12, off sc0 sc1\n\t"
                   "s_waitcnt vmcnt(0)"
                   : "=&v"(y0), "=&v"(y1), "=&v"(y2), "=&v"(y3), "=&v"(z0), "=&v"(z1), "=&v"(z2), "=&v"(z3), "=&v"(fv)
                   : "v"(lane * 16), "s"(src), "s"(src + (rdy2 ? 4096 : 0)), "v"(stgf) : "memory");
      if (!open) {
        while (rfl(__hip_atomic_load(lfree, __ATOMIC_RELAXED, __HIP_MEMORY_SCOPE_WORKGROUP)) == 0) {
          if (__builtin_amdgcn_s_memrealtime() - t0 > 2 * DEADLINE) break;
          __builtin_amdgcn_s_sleep(2);
        }
        open = true;
        if (wave < 5) STAMP(6 + wave);
      }
      const u64 now = __builtin_amdgcn_s_memrealtime();
      quick = now - tl0 < FWD_QUICK;
      tl0 = now;
      STAMP(16 + t * 10 + k);
      if (bad) y0.x = __builtin_nanf("");
      auto put = [&](unsigned u, f32x4 v0, f32x4 v1, f32x4 v2, f32x4 v3) {
        const unsigned vo = ovoff + u * (32 * COLS * 4) + lent_strip(p, q, i) * 128;
        asm volatile("global_store_dwordx4 %0, %4, %8 sc0 sc1\n\t"
                     "global_store_dwordx4 %1, %5, %8 sc0 sc1\n\t"
                     "global_store_dwordx4 %2, %6, %8 sc0 sc1\n\t"
                     "global_store_dwordx4 %3, %7, %8 sc0 sc1\n\ts_nop 1"
                     :: "v"(vo), "v"(vo + 8 * COLS * 4), "v"(vo + 16 * COLS * 4), "v"(vo + 24 * COLS * 4),
                        "v"(v0), "v"(v1), "v"(v2), "v"(v3), "s"(obase) : "memory");
      };
      put(unit, y0, y1, y2, y3);
      if (rdy2) {
        STAMP(16 + t * 10 + k + 1);
        put(unit2, z0, z1, z2, z3);
      }
    }
    asm volatile("s_waitcnt vmcnt(0)" ::: "memory");
    unsigned* const dp = (unsigned*)(P.ws[owner] + WS_DONE + q * 256 + (j * 5 + wave - 1) * 4);
    if (lane == 0) __hip_atomic_store(dp, call, __ATOMIC_RELAXED, __HIP_MEMORY_SCOPE_SYSTEM);
    STAMP(10 + wave);
    return;
  }

  const int nq = (b & 3) * 24 + (b >> 3) - 8;
  const int g = ((b >> 2) & 1) * 3 + (nq >> 5), s = nq & 31;
  const bool foreign = s >= 16;
  const int owner = (foreign ? (side ^ 1) * 4 + ((s - 16) >> 2) : rank) & (NR - 1);
  const int cls = foreign ? 1 + (owner & 3) : 0;
  const unsigned ul0 = P.units[g][cls][0], ul1 = P.units[g][cls][1], ul2 = P.units[g][cls][2];
  const unsigned nu = P.units[g][cls][3] < 12u ? P.units[g][cls][3] : 12u;
  auto unit_at = [&](unsigned k) -> unsigned {
    unsigned wv = k < 4 ? ul0 : (k < 8 ? ul1 : ul2);
    return (wv >> ((k & 3) * 8)) & 127u;
  };

  if (wave == 1) {
    const int col0 = (foreign ? lent_strip(owner & 3, q, s & 3) : s) * 32;
    char* const obase = P.out[owner] + col0 * 4;
    const unsigned prod = (s & 15) * 8 + g;
    char* const stg = myws + WS_STG + (size_t)prod * (12 * 4096);
    unsigned* const stgf = (unsigned*)(myws + WS_STGF + prod * 128);
    unsigned* const donep = (unsigned*)(P.ws[owner] + WS_DONE + q * 256 + 160 + ((s & 3) * 6 + g) * 4);
    if ((TOUCH & 2) && foreign) {
      const char* a1 = lane & 1 ? (const char*)donep : obase + (size_t)unit_at(nu - 1) * (32 * COLS * 4);
      unsigned v1;
      __builtin_amdgcn_s_sleep(100);
      asm volatile("global_load_dword %0, %1, off sc0 sc1\n\ts_waitcnt vmcnt(0)" : "=&v"(v1) : "v"(a1) : "memory");
    }
    const char* const pl = smem + L_PART + rr * 128 + ((chn ^ rr) * 16);
    f32x4 keep[KEEP][4];
    unsigned kd = 0, ks = 0;
    while (ks < nu) {
      const u64 st = __hip_atomic_load((u64*)lcnt, __ATOMIC_RELAXED, __HIP_MEMORY_SCOPE_WORKGROUP);
      const unsigned have = rfl((unsigned)st), idle = rfl((unsigned)(st >> 32));
      if (__builtin_amdgcn_s_memrealtime() - t0 > 2 * DEADLINE) bad = true;
      if (have >= (kd + 1) * 4 || bad) {
        asm volatile("" ::: "memory");
        f32x4 y[4];
#pragma unroll
        for (int it = 0; it < 4; ++it) {
          f32x4 p0 = *(const f32x4*)(pl + it * 1024);
          f32x4 p1 = *(const f32x4*)(pl + it * 1024 + 4096);
          f32x4 p2 = *(const f32x4*)(pl + it * 1024 + 8192);
          f32x4 p3 = *(const f32x4*)(pl + it * 1024 + 12288);
          y[it] = (p0 + p1) + (p2 + p3);
        }
#pragma unroll
        for (int it = 0; it < 4; ++it) asm volatile("" : "+v"(y[it]));
        asm volatile("s_waitcnt lgkmcnt(0)" ::: "memory");
        if (lane == 0) __hip_atomic_store(lfree, kd + 1, __ATOMIC_RELAXED, __HIP_MEMORY_SCOPE_WORKGROUP);
#pragma unroll
        for (int it = 0; it < 4; ++it) {
          y[it].x = gelu(y[it].x); y[it].y = gelu(y[it].y); y[it].z = gelu(y[it].z); y[it].w = gelu(y[it].w);
          if (bad) y[it].x = __builtin_nanf("");
        }
#pragma unroll
        for (unsigned k = 0; k < KEEP; ++k)
          if (kd - ks == k) {
#pragma unroll
            for (int it = 0; it < 4; ++it) { keep[k][it] = y[it]; asm volatile("" : "+v"(keep[k][it])); }
          }
        ++kd;
      } else if (ks == kd || idle != kd + 1) {
        __builtin_amdgcn_s_sleep(1);
        continue;
      }
      if (kd - ks == KEEP || kd + 1 >= nu || idle == kd + 1) {
        const bool stage = foreign && kd < nu;
        char* const base = stage ? stg : obase;
        const unsigned step = stage ? 1024 : 8 * COLS * 4;
#pragma unroll
        for (unsigned k = 0; k < KEEP; ++k)
          if (ks + k < kd) {
            const unsigned vo = stage ? lane * 16 + (ks + k) * 4096 : ovoff + unit_at(ks + k) * (32 * COLS * 4);
#pragma unroll
            for (int it = 0; it < 4; ++it)
              asm volatile("global_store_dwordx4 %0, %1, %2 sc0 sc1\n\ts_nop 1"
                           :: "v"(vo + it * step), "v"(keep[k][it]), "s"(base) : "memory");
            STAMP(32 + ks + k);
          }
        ks = kd;
        if (stage) {
          asm volatile("s_waitcnt vmcnt(0)" ::: "memory");
          if (lane == 0) __hip_atomic_store(stgf, call * 16 + kd, __ATOMIC_RELAXED, __HIP_MEMORY_SCOPE_SYSTEM);
        }
      }
    }
    char* const lastp = obase + (size_t)unit_at(0) * (32 * COLS * 4);
    asm volatile("s_waitcnt vmcnt(0)" ::: "memory");
    STAMP(2);
    if (foreign) {
      if (lane == 0) __hip_atomic_store(donep, call, __ATOMIC_RELAXED, __HIP_MEMORY_SCOPE_SYSTEM);
    } else if (b == 64) {
      const char* dp = myws + WS_DONE + lane * 16;
      while (true) {
        u32x4 v;
        asm volatile("global_load_dwordx4 %0, %1, off sc0 sc1\n\ts_waitcnt vmcnt(0)" : "=&v"(v) : "v"(dp) : "memory");
        bool ok = v.x >= call && v.y >= call && v.z >= call && v.w >= call;
        if (__all(ok)) break;
        if (give_up(t0, tmo, call)) {
          if (lane == 0) __hip_atomic_store((float*)lastp, __builtin_nanf(""), __ATOMIC_RELAXED, __HIP_MEMORY_SCOPE_SYSTEM);
          break;
        }
        __builtin_amdgcn_s_sleep(1);
      }
      STAMP(3);
    }
    return;
  }

  const int w = wave - 2;
  char* const data = myws + WS_DATA + (size_t)(call & 1) * PAR_BYTES;
  const unsigned* const flags = (const unsigned*)(myws + WS_FLAG) + w * 8 + (lane & 7);
  lds_char* const lw = (lds_char*)smem + w * 32768;
  lds_char* const ltr = lw + ((lane >> 5) * 8 + ((lane & 15) >> 2)) * 64 + ((lane >> 4) & 1) * 32 + (lane & 3) * 8;
  h8 wr[32];
  {
    const unsigned* pp = flags + (128 + s) * 32;
    const char* src = data + (size_t)(128 + s) * OBJ + w * 16384;
    const unsigned svoff = lane * 16;
    const unsigned lwa = (unsigned)(size_t)lw;
    unsigned seen = 0;
#pragma unroll
    for (int r = 0; r < 4; ++r) {
      wait_round(pp, r, seen, bad, t0, tmo, call, lidle, 0);
#pragma unroll
      for (int i = 0; i < 16; ++i)
        glds16(src + r * 65536 + i * 1024, svoff, lwa + (r & 1) * 16384 + i * 1024);
      if (r & 1) asm volatile("s_waitcnt vmcnt(0)" ::: "memory");
      if (r == 1) {
#pragma unroll
        for (int i = 0; i < 32; ++i) wr[i] = tr_frag(ltr + i * 1024);
#pragma unroll
        for (int i = 0; i < 32; ++i) asm volatile("" : "+v"(wr[i]));
        asm volatile("s_waitcnt lgkmcnt(0)" ::: "memory");
      }
    }
  }
  if (w == 0) STAMP(4);
  const __amdgpu_buffer_rsrc_t rsrc =
      __builtin_amdgcn_make_buffer_rsrc(data, 0, 128 * OBJ, 0x00020000);
  const unsigned xvoff = w * 16384 + lane * 16;
  char* const pw = smem + L_PART + w * 4096 + (lane & 31) * 128;
  unsigned nextf = 0;
  for (unsigned k = 0; k < nu; ++k) {
    const unsigned e = unit_at(k);
    const unsigned* pp = flags + e * 32;
    const unsigned* pn = flags + unit_at(k + 1 < nu ? k + 1 : k) * 32;
    const unsigned soff = e * OBJ;
    unsigned seen = flag_mask(nextf, call);
    f32x16 acc = {0.f, 0.f, 0.f, 0.f, 0.f, 0.f, 0.f, 0.f, 0.f, 0.f, 0.f, 0.f, 0.f, 0.f, 0.f, 0.f};
#pragma unroll
    for (int r = 0; r < 4; ++r) {
      wait_round(pp, r, seen, bad, t0, tmo, call, lidle, k + 1);
      u32x4 xa[16];
#pragma unroll
      for (int i = 0; i < 16; ++i)
        xa[i] = __builtin_amdgcn_raw_buffer_load_b128(rsrc, xvoff + (i & 3) * 1024, soff + r * 65536 + (i >> 2) * 4096, 17);
      __builtin_amdgcn_sched_barrier(0);
      if (r == 3) { nextf = flag_load(pn); __builtin_amdgcn_sched_barrier(0); }
#pragma unroll
      for (int i = 0; i < 16; ++i) {
        h8 wf = r < 2 ? wr[r * 16 + i] : tr_frag(ltr + ((r - 2) * 16 + i) * 1024);
        acc = __builtin_amdgcn_mfma_f32_32x32x16_f16(wf, __builtin_bit_cast(h8, xa[i]), acc, 0, 0, 0);
      }
    }
    if (bad) {
#pragma unroll
      for (int i = 0; i < 16; ++i) acc[i] = __builtin_nanf("");
    }
    while (rfl(__hip_atomic_load(lfree, __ATOMIC_RELAXED, __HIP_MEMORY_SCOPE_WORKGROUP)) < k) {
      if (__builtin_amdgcn_s_memrealtime() - t0 > 2 * DEADLINE) break;
      __builtin_amdgcn_s_sleep(1);
    }
    asm volatile("" ::: "memory");
#pragma unroll
    for (int gq = 0; gq < 4; ++gq) {
      f32x4 v = {acc[gq * 4], acc[gq * 4 + 1], acc[gq * 4 + 2], acc[gq * 4 + 3]};
      *(f32x4*)(pw + (((gq * 2 + (lane >> 5)) ^ (lane & 7)) * 16)) = v;
    }
    asm volatile("s_waitcnt lgkmcnt(0)" ::: "memory");
    if (lane == 0) __hip_atomic_fetch_add(lcnt, 1u, __ATOMIC_RELAXED, __HIP_MEMORY_SCOPE_WORKGROUP);
    if (w == 0) STAMP(16 + k);
  }
}

unsigned link_map(int rank) {
  const unsigned dflt = 0x76543210u;
  int dev = -1, n = 0, node[NR], port[NR + 1];
  if (hipGetDevice(&dev) != hipSuccess || hipGetDeviceCount(&n) != hipSuccess || dev != rank || n != NR) return dflt;
  for (int i = 0; i <= NR; ++i) port[i] = -1;
  for (int d = 0; d < NR; ++d) {
    char id[32], path[96];
    if (hipDeviceGetPCIBusId(id, sizeof id, d) != hipSuccess) return dflt;
    for (char* c = id; *c; ++c) if (*c >= 'A' && *c <= 'F') *c += 'a' - 'A';
    snprintf(path, sizeof path, "/sys/bus/pci/devices/%s/xgmi_port_num", id);
    FILE* f = fopen(path, "r");
    if (!f) return dflt;
    unsigned a, pa, z, pz;
    node[d] = -1;
    while (fscanf(f, " %x:%x -> %x:%x", &a, &pa, &z, &pz) == 4) {
      node[d] = (int)a;
      if (d == rank && z <= (unsigned)NR && pa < 8u) port[z] = (int)pa;
    }
    fclose(f);
    if (node[d] < 0 || node[d] > NR) return dflt;
  }
  unsigned map = 0, used = 0;
  const int side = rank >> 2, q = rank & 3;
  for (int c = 0; c < 7; ++c) {
    const int x = port[node[c < 3 ? side * 4 + ((q + 1 + c) & 3) : (side ^ 1) * 4 + ((q + c - 3) & 3)]];
    if (x < 0 || ((used >> x) & 1)) return dflt;
    used |= 1u << x;
    map |= (unsigned)c << (4 * x);
  }
  return map | 7u << (4 * __builtin_ctz(~used));
}

}

extern "C" void kernel_launch(int rank, int nranks, ncclComm_t comm,
                              hipStream_t stream,
                              void* const* d_in, const int* in_sizes, int n_in,
                              void* d_out, int out_size,
                              const int* in_dtypes, int out_dtype,
                              void* const* d_out_all,
                              void* const* d_ws_all, size_t ws_bytes) {
  (void)comm; (void)in_dtypes; (void)out_dtype; (void)d_out;
  if (nranks != NR || (unsigned)rank >= (unsigned)NR || n_in < 2 || ws_bytes < WS_NEED ||
      in_sizes[0] != ROWS * KDIM || in_sizes[1] != KDIM * COLS || out_size != NR * ROWS * COLS) {
    fprintf(stderr, "kernel_launch: unexpected problem shape\n");
    return;
  }
  Params P;
  P.x = (const char*)d_in[0];
  P.w = (const char*)d_in[1];
  for (int p = 0; p < NR; ++p) { P.ws[p] = (char*)d_ws_all[p]; P.out[p] = (char*)d_out_all[p]; }
  P.rank = rank;
  P.xmap = link_map(rank);
  const int side = rank >> 2, q = rank & 3;
  static const unsigned char order[NG][12] = {
    {60, 61, 0, 6, 12, 40, 46, 18, 24, 52, 30, 99},
    {62, 63, 1, 7, 13, 41, 47, 19, 25, 53, 31, 99},
    {64, 65, 72, 2, 8, 14, 42, 20, 48, 26, 54, 32},
    {66, 67, 73, 3, 9, 43, 15, 21, 49, 55, 27, 33},
    {68, 69, 74, 4, 10, 16, 44, 22, 50, 28, 34, 99},
    {70, 71, 75, 5, 11, 17, 45, 23, 51, 29, 35, 99}};
  for (int g = 0; g < NG; ++g)
    for (int cls = 0; cls < 5; ++cls) {
      unsigned u[12] = {0}, n = 0;
      for (int k = 0; k < 12; ++k) {
        int v = order[g][k];
        if (v == 99) break;
        if (cls > 0 && (v >= 60 + NA || (v >= 40 && v < 60 && ((v - 40) & 3) == cls - 1))) continue;
        if (v < 40) u[n++] = (side * 4 + ((q + 1 + v % 3) & 3)) * 16 + v / 3;
        else if (v < 60) u[n++] = ((side ^ 1) * 4 + ((v - 40) & 3)) * 16 + NA + ((v - 40) >> 2);
        else u[n++] = rank * 16 + v - 60;
      }
      for (int k = 0; k < 3; ++k)
        P.units[g][cls][k] = u[4 * k] | (u[4 * k + 1] << 8) | (u[4 * k + 2] << 16) | (u[4 * k + 3] << 24);
      P.units[g][cls][3] = n;
    }
  hipLaunchKernelGGL(ag_gemm_gelu, dim3(256), dim3(384), 0, stream, P);
  HIP_CHECK(hipGetLastError());
}
```

```cpp
#include <hip/hip_runtime.h>
#include <rccl/rccl.h>
#include <cstdio>

#define HIP_CHECK(expr) do { hipError_t _e = (expr); if (_e != hipSuccess) { \
    fprintf(stderr, "HIP error %s at %s:%d\n", hipGetErrorString(_e), __FILE__, __LINE__); \
    return; } } while (0)

namespace {

typedef unsigned u32x4 __attribute__((ext_vector_type(4)));
typedef float f32x4 __attribute__((ext_vector_type(4)));
typedef float f32x16 __attribute__((ext_vector_type(16)));
typedef _Float16 h2 __attribute__((ext_vector_type(2)));
typedef _Float16 h8 __attribute__((ext_vector_type(8)));
typedef short s4 __attribute__((ext_vector_type(4)));
typedef short s8 __attribute__((ext_vector_type(8)));
typedef unsigned long long u64;

constexpr int NR = 8;
constexpr int KDIM = 4096;
constexpr int ROWS = 512;
constexpr int COLS = 512;
constexpr unsigned OBJ = 256u * 1024u;
constexpr int NG = 6;
constexpr int NA = 12;
constexpr size_t WS_CNT = 0;
constexpr size_t WS_TMO = 16 * 1024;
constexpr size_t WS_TICK = 20 * 1024;
constexpr size_t WS_FLAG = 32 * 1024;
constexpr size_t WS_DONE = 64 * 1024;
constexpr size_t WS_STGF = 80 * 1024;
constexpr size_t WS_DIAG = 128 * 1024;
constexpr size_t WS_DATA = 1024 * 1024;
constexpr unsigned NSLOT = 160;
constexpr size_t PAR_BYTES = (size_t)NSLOT * OBJ;
constexpr size_t WS_STG = WS_DATA + 2 * PAR_BYTES;
constexpr size_t WS_NEED = WS_STG + 128 * 12 * 4096;
constexpr u64 DEADLINE = 100000000ull;
#define FWD_T0 2500
#define SMODE 1
__host__ __device__ constexpr int lent_strip(int o, int h, int i) {
  return SMODE == 0 ? h * 4 + i : (SMODE == 1 ? ((o + h) & 3) * 4 + i : i * 4 + ((o + h) & 3));
}
#define FW 5
#define KA 4
#define KC 4
#define FWD_QUICK 0
#define RP 3
#define TOUCH 3
#define LAG 30
#define KEEP 4
constexpr unsigned L_PART = 128 * 1024;
constexpr unsigned L_CTRL = L_PART + 16 * 1024;
constexpr unsigned L_TOTAL = L_CTRL + 64;

struct alignas(64) Head {
  const char* x;
  const char* w;
  char* myws;
  char* dws;
  int rank;
  int ch;
};
struct Params {
  Head head[8];
  char* ws[NR];
  char* out[NR];
  unsigned units[NG][5][4];
};

__device__ __forceinline__ unsigned rfl(unsigned v) { return __builtin_amdgcn_readfirstlane(v); }
__device__ __forceinline__ unsigned pk(float a, float b) {
  h2 v; v.x = (_Float16)a; v.y = (_Float16)b; return __builtin_bit_cast(unsigned, v);
}
__device__ __forceinline__ u32x4 cvt8(u32x4 a, u32x4 b) {
  f32x4 fa = __builtin_bit_cast(f32x4, a), fb = __builtin_bit_cast(f32x4, b);
  u32x4 r; r.x = pk(fa.x, fa.y); r.y = pk(fa.z, fa.w); r.z = pk(fb.x, fb.y); r.w = pk(fb.z, fb.w);
  return r;
}
__device__ __forceinline__ float gelu(float y) {
  const float c2 = -2.0f * 0.7978845608028654f * 1.4426950408889634f;
  float u = y + 0.044715f * y * y * y;
  float e = __builtin_amdgcn_exp2f(c2 * u);
  return y * __builtin_amdgcn_rcpf(1.0f + e);
}

#define STAMPV(idx, val) do {} while (0)
#define STAMP(idx) STAMPV(idx, __builtin_amdgcn_s_memrealtime())

__device__ __forceinline__ bool give_up(u64 t0, unsigned* tmo, unsigned call) {
  u64 el = __builtin_amdgcn_s_memrealtime() - t0;
  if (el < 100000ull) return false;
  if (el > DEADLINE ||
      rfl(__hip_atomic_load(tmo, __ATOMIC_RELAXED, __HIP_MEMORY_SCOPE_AGENT)) == call) {
    __hip_atomic_store(tmo, call, __ATOMIC_RELAXED, __HIP_MEMORY_SCOPE_AGENT);
    return true;
  }
  return false;
}

__device__ __forceinline__ unsigned flag_load(const unsigned* p) {
  return __hip_atomic_load(p, __ATOMIC_RELAXED, __HIP_MEMORY_SCOPE_SYSTEM);
}
__device__ __forceinline__ unsigned flag_mask(unsigned v, unsigned call) {
  return (unsigned)__builtin_amdgcn_ballot_w64(v >= call) & 0xFFu;
}
__device__ __forceinline__ void wait_round(const unsigned* p, int r, unsigned& seen, bool& bad, u64 t0,
                                           unsigned* tmo, unsigned call, unsigned* lidle, unsigned mark) {
  while (((seen >> (2 * r)) & 3u) != 3u) {
    seen = flag_mask(flag_load(p), call);
    if (((seen >> (2 * r)) & 3u) == 3u) break;
    if (r < 3) __hip_atomic_store(lidle, mark, __ATOMIC_RELAXED, __HIP_MEMORY_SCOPE_WORKGROUP);
    if (give_up(t0, tmo, call)) { bad = true; break; }
    __builtin_amdgcn_s_sleep(2);
  }
  asm volatile("" ::: "memory");
}

struct Piece { u32x4 r[16]; };

__device__ __forceinline__ void piece_load(Piece& s, const char* sb, unsigned fstride, unsigned voff) {
#pragma unroll
  for (int u = 0; u < 8; ++u) {
    const char* p = sb + (size_t)u * fstride;
    asm volatile("global_load_dwordx4 %0, %1, %2" : "=v"(s.r[2 * u]) : "v"(voff), "s"(p) : "memory");
    asm volatile("global_load_dwordx4 %0, %1, %2 offset:16" : "=v"(s.r[2 * u + 1]) : "v"(voff), "s"(p) : "memory");
  }
}
template <int N>
__device__ __forceinline__ void piece_wait(Piece& s) {
  asm volatile("s_waitcnt vmcnt(%0)" :: "i"(N) : "memory");
#pragma unroll
  for (int u = 0; u < 16; ++u) asm volatile("" : "+v"(s.r[u]));
}
template <int U>
__device__ __forceinline__ void piece_store1(Piece& s, char* db, unsigned voff) {
  u32x4 v = cvt8(s.r[2 * U], s.r[2 * U + 1]);
  asm volatile("global_store_dwordx4 %0, %1, %2 offset:%3 sc0 sc1\n\ts_nop 1"
               :: "v"(voff), "v"(v), "s"(db), "i"((U - 4) * 1024) : "memory");
}
__device__ __forceinline__ void piece_store(Piece& s, char* db  , unsigned voff) {
  piece_store1<0>(s, db, voff); piece_store1<1>(s, db, voff); piece_store1<2>(s, db, voff);
  piece_store1<3>(s, db, voff); piece_store1<4>(s, db, voff); piece_store1<5>(s, db, voff);
  piece_store1<6>(s, db, voff); piece_store1<7>(s, db, voff);
}

typedef __attribute__((address_space(3))) char lds_char;

__device__ __forceinline__ void glds16(const char* src, unsigned voff, unsigned lds) {
  unsigned keep;
  asm volatile("s_mov_b32 %0, m0\n\ts_mov_b32 m0, %3\n\ts_nop 0\n\t"
               "global_load_lds_dwordx4 %1, %2 sc0 sc1\n\ts_mov_b32 m0, %0"
               : "=&s"(keep) : "v"(voff), "s"(src), "s"(lds) : "memory");
}

__device__ __forceinline__ h8 tr_frag(lds_char* p) {
  s4 a = __builtin_amdgcn_ds_read_tr16_b64_v4i16((__attribute__((address_space(3))) s4*)p);
  s4 c = __builtin_amdgcn_ds_read_tr16_b64_v4i16((__attribute__((address_space(3))) s4*)(p + 256));
  s8 r = __builtin_shufflevector(a, c, 0, 1, 2, 3, 4, 5, 6, 7);
  return __builtin_bit_cast(h8, r);
}

__global__ void __launch_bounds__(384) ag_gemm_gelu(Params P) {
  __shared__ __attribute__((aligned(1024))) char smem[L_TOTAL];
  const int lane = threadIdx.x & 63;
  const int wave = rfl(threadIdx.x >> 6);
  Head H = P.head[__builtin_amdgcn_s_getreg(20 | (3 << 11)) & 7];
  asm volatile("" : "+s"(H.x), "+s"(H.w), "+s"(H.myws), "+s"(H.dws), "+s"(H.rank), "+s"(H.ch));
  const int xch = H.ch & 7;
  const int b = blockIdx.x < 64 ? xch * 8 + (blockIdx.x >> 3) : blockIdx.x;
  const int rank = H.rank & (NR - 1);
  const int side = rank >> 2, q = rank & 3;
  char* const myws = (char*)(__attribute__((address_space(1))) char*)(u64)H.myws;
  unsigned* const cntp = (unsigned*)(myws + WS_CNT + b * 32 + wave * 4);
  unsigned* const lcnt = (unsigned*)(smem + L_CTRL);
  unsigned* const lidle = (unsigned*)(smem + L_CTRL + 4);
  unsigned* const lfree = (unsigned*)(smem + L_CTRL + 8);

  if (__builtin_expect(wave == 0, 1)) {
    if (__builtin_expect(b >= 64 + 128 || (b >> 3) == 7, 0)) {
      const int c = xch;
      if ((TOUCH & 1) && (b >> 3) == 24 && c < 7) {
        const int dest = (c < 3 ? side * 4 + ((q + 1 + c) & 3) : (side ^ 1) * 4 + ((q + c - 3) & 3)) & (NR - 1);
        const char* const dd = H.dws + WS_DATA + (lane & 1) * PAR_BYTES;
        const char* a1 = lane < 2 ? dd + (size_t)(c < 3 ? rank * 16 : 144 + q * 4) * OBJ : H.dws + WS_FLAG;
        const char* a2 = lane < 32 ? dd + (size_t)(rank * 16 + (lane >> 1)) * OBJ : P.out[dest] + (size_t)(lane - 32) * OBJ;
        unsigned v1, v2;
        asm volatile("global_load_dword %0, %2, off sc0 sc1\n\tglobal_load_dword %1, %3, off sc0 sc1\n\ts_waitcnt vmcnt(0)"
                     : "=&v"(v1), "=&v"(v2) : "v"(a1), "v"(a2) : "memory");
      }
      return;
    }
    __builtin_amdgcn_s_setprio(3);
    __builtin_amdgcn_s_waitcnt(0);
    const int j = b & 7, c = b >> 3;
    int dest, nW, strip0, wslot0, xu0, nobj;
    if (c < 3)      { dest = side * 4 + ((q + 1 + c) & 3); nW = 0; strip0 = 0; wslot0 = 0; xu0 = 0; nobj = NA; }
    else if (c < 7) { dest = (side ^ 1) * 4 + ((q + c - 3) & 3); nW = 4; strip0 = lent_strip(q, dest & 3, 0); wslot0 = 16 + q * 4;
                      xu0 = NA; nobj = 4 + 16 - NA; }
    else            { dest = rank; nW = 1; strip0 = c - 8; wslot0 = c - 8; xu0 = c - 8; nobj = 2; }
    const unsigned np = nobj * 32;
    const unsigned nt = np / 2 - (c < 3 ? 2 * RP : 0);
    const unsigned laneX = (lane & 31) * (KDIM * 4) + (lane >> 5) * 32;
    const unsigned laneW = (lane >> 2) * (COLS * 4) + (lane & 3) * 32;
    const unsigned lane16 = lane * 16;
    const char* const px = H.x;
    const char* const pw = H.w;
    if (LAG && b >= 64) __builtin_amdgcn_s_sleep(LAG);
    auto load = [&](Piece& s, unsigned pc) {
      pc = pc < np ? pc : np - 1;
      const int obj = pc >> 5, f0 = (pc & 31) * 8;
      const bool isW = obj < nW;
      const char* sb = isW ? pw + (size_t)(strip0 + obj * (lent_strip(0, 0, 1) - lent_strip(0, 0, 0))) * 128 + (size_t)f0 * (16 * COLS * 4)
                           : px + (size_t)(xu0 + obj - nW) * (32 * KDIM * 4) + (size_t)f0 * 64;
      piece_load(s, sb, isW ? 16 * COLS * 4 : 64, isW ? laneW : laneX);
    };
    unsigned cnt;
    asm volatile("global_load_dword %0, %1, off sc1" : "=v"(cnt) : "v"(cntp) : "memory");
    Piece A, B;
    load(A, j);
    load(B, j + 8);
    piece_wait<16>(A);
    asm volatile("" : "+v"(cnt));
    const unsigned call = rfl(cnt) + 1;
    char* const dws = c < 8 ? H.dws : myws;
    char* const ddata = dws + WS_DATA + (size_t)(call & 1) * PAR_BYTES + 4096;
    char* const tick = myws + WS_TICK + c * 128;
    const unsigned tbase = 16 - (call - 1) * nt;
    auto slot_of = [&](unsigned pc) -> unsigned {
      const int obj = pc >> 5;
      return obj < nW ? 128 + wslot0 + obj : rank * 16 + xu0 + obj - nW;
    };
    auto store = [&](Piece& s, unsigned pc) {
      piece_store(s, ddata + (size_t)slot_of(pc) * OBJ + (pc & 31) * 8192, lane16);
    };
    const unsigned zero = 0, one = 1;
    auto flag = [&](unsigned pc) {
      char* fp = dws + WS_FLAG + slot_of(pc) * 128 + (((pc >> 1) & 3) * 8 + ((pc >> 3) & 3) * 2 + (pc & 1)) * 4;
      if (lane == 0) asm volatile("global_store_dword %0, %1, %2 sc0 sc1" :: "v"(zero), "v"(call), "s"(fp) : "memory");
    };
    auto ticket = [&](unsigned& t) {
      if (lane == 0) asm volatile("global_atomic_add %0, %1, %2, %3 sc0" : "=v"(t) : "v"(zero), "v"(one), "s"(tick) : "memory");
    };
    STAMP(0);
    auto piece_of = [&](unsigned t) -> unsigned {
      if (t >= (NA - 1) * 16) t += 2 * RP;
      return (t >> 3) * 16 + (t & 7);
    };
    unsigned tk = 0;
    unsigned cur = j, nxt = j + 16;
    store(A, cur);
    ticket(tk);
    load(A, nxt);
    piece_wait<25>(B);
    if (lane == 0) asm volatile("global_store_dword %0, %1, off sc1" :: "v"(cntp), "v"(call) : "memory");
    store(B, cur + 8);
    load(B, nxt + 8);
    __builtin_amdgcn_s_barrier();
    do {
      piece_wait<25>(A);
      asm volatile("" : "+v"(tk));
      flag(cur);
      unsigned t = rfl(tk) + tbase;
      const unsigned nn = t < nt ? piece_of(t) : np;
      if (c < 3) asm volatile("s_waitcnt vmcnt(%0)" :: "i"(17 + KA) : "memory");
      else if (c < 7) asm volatile("s_waitcnt vmcnt(%0)" :: "i"(17 + KC) : "memory");
      store(A, nxt);
      ticket(tk);
      load(A, nn);
      piece_wait<26>(B);
      flag(cur + 8);
      if (c < 3) asm volatile("s_waitcnt vmcnt(%0)" :: "i"(18 + KA) : "memory");
      else if (c < 7) asm volatile("s_waitcnt vmcnt(%0)" :: "i"(18 + KC) : "memory");
      store(B, nxt + 8);
      load(B, nn + 8);
      cur = nxt;
      nxt = nn;
    } while (nxt < np);
    if (b < 64) __hip_atomic_store(lfree, 1u, __ATOMIC_RELAXED, __HIP_MEMORY_SCOPE_WORKGROUP);
    piece_wait<25>(A);
    asm volatile("" : "+v"(tk));
    flag(cur);
    piece_wait<0>(B);
    flag(cur + 8);
    STAMP(1);
    return;
  }

  const bool fwd = b < 64;
  if (fwd && (b < 24 || b >= 56)) return;
  const u64 t0 = __builtin_amdgcn_s_memrealtime();
  if (LAG && !fwd) __builtin_amdgcn_s_sleep(LAG);
  const unsigned call = rfl(__hip_atomic_load(cntp, __ATOMIC_RELAXED, __HIP_MEMORY_SCOPE_AGENT)) + 1;
  if (lane == 0) __hip_atomic_store(cntp, call, __ATOMIC_RELAXED, __HIP_MEMORY_SCOPE_AGENT);
  unsigned* const tmo = (unsigned*)(myws + WS_TMO);
  const int rr = lane >> 3, chn = lane & 7;
  const unsigned ovoff = rr * (COLS * 4) + chn * 16;
  bool bad = false;
  if (wave == 1) {
    STAMPV(5, t0);
    if (lane < 8) lcnt[lane] = 0;
    asm volatile("s_waitcnt lgkmcnt(0)" ::: "memory");
  }
  __builtin_amdgcn_s_barrier();

  if (fwd) {
    const int j = b & 7, p = (q + (b >> 3) - 3) & 3;
    const int owner = ((side ^ 1) * 4 + p) & (NR - 1);
    char* const obase = P.out[owner];
    unsigned* const lnext = (unsigned*)(smem + L_CTRL + 16);
    const unsigned tl = lane < 3 ? lane : 0;
    const unsigned* const stgf = (const unsigned*)(myws + WS_STGF + ((p * 4 + ((j + tl) & 3)) * 8 + 2 * tl + (j >> 2)) * 128);
    const unsigned n0 = P.units[j >> 2][1 + p][3], n1 = P.units[2 + (j >> 2)][1 + p][3], n2 = P.units[4 + (j >> 2)][1 + p][3];
    const unsigned nl = (tl == 0 ? n0 : (tl == 1 ? n1 : n2)) - 1;
    while (__builtin_amdgcn_s_memrealtime() - t0 < FWD_T0) __builtin_amdgcn_s_sleep(16);
    if (RP && wave == 5) {
      const size_t dbase = WS_DATA + (size_t)(call & 1) * PAR_BYTES + 4096;
      auto rpiece = [](unsigned n) -> unsigned { return n % (2 * RP) + n / (2 * RP) * 8; };
      auto rflag = [](unsigned pc) -> unsigned { return (((pc >> 1) & 3) * 8 + (pc >> 3) * 2 + (pc & 1)) * 4; };
      if (j >= 8 - RP) {
        const unsigned slot = rank * 16 + NA - 1, pc = rpiece(p * RP + 7 - j);
        Piece A;
        piece_load(A, H.x + (size_t)(NA - 1) * (32 * KDIM * 4) + pc * 512, 64, (lane & 31) * (KDIM * 4) + (lane >> 5) * 32);
        piece_wait<0>(A);
        while (rfl(__hip_atomic_load(lfree, __ATOMIC_RELAXED, __HIP_MEMORY_SCOPE_WORKGROUP)) == 0) {
          if (__builtin_amdgcn_s_memrealtime() - t0 > 2 * DEADLINE) break;
          __builtin_amdgcn_s_sleep(2);
        }
        piece_store(A, P.ws[owner] + dbase + (size_t)slot * OBJ + pc * 8192, lane * 16);
        asm volatile("s_waitcnt vmcnt(0)" ::: "memory");
        if (lane == 0) __hip_atomic_store((unsigned*)(P.ws[owner] + WS_FLAG + slot * 128 + rflag(pc)), call, __ATOMIC_RELAXED, __HIP_MEMORY_SCOPE_SYSTEM);
      }
      for (int n = j; n < 3 * RP; n += 8) {
        const unsigned slot = ((side ^ 1) * 4 + ((p + 1 + n / RP) & 3)) * 16 + NA - 1, pc = rpiece(q * RP + n % RP);
        const size_t fo = WS_FLAG + slot * 128 + rflag(pc);
        const size_t go = dbase + (size_t)slot * OBJ + pc * 8192;
        while (rfl(flag_load((const unsigned*)(myws + fo))) < call) {
          if (give_up(t0, tmo, call)) { bad = true; break; }
          __builtin_amdgcn_s_sleep(8);
        }
        if (bad) break;
        u32x4 y0, y1, y2, y3, y4, y5, y6, y7;
        asm volatile("global_load_dwordx4 %0, %8, %9 offset:-4096 sc0 sc1\n\t"
                     "global_load_dwordx4 %1, %8, %9 offset:-3072 sc0 sc1\n\t"
                     "global_load_dwordx4 %2, %8, %9 offset:-2048 sc0 sc1\n\t"
                     "global_load_dwordx4 %3, %8, %9 offset:-1024 sc0 sc1\n\t"
                     "global_load_dwordx4 %4, %8, %9 sc0 sc1\n\t"
                     "global_load_dwordx4 %5, %8, %9 offset:1024 sc0 sc1\n\t"
                     "global_load_dwordx4 %6, %8, %9 offset:2048 sc0 sc1\n\t"
                     "global_load_dwordx4 %7, %8, %9 offset:3072 sc0 sc1\n\t"
                     "s_waitcnt vmcnt(0)\n\t"
                     "global_store_dwordx4 %8, %0, %10 offset:-4096 sc0 sc1\n\t"
                     "global_store_dwordx4 %8, %1, %10 offset:-3072 sc0 sc1\n\t"
                     "global_store_dwordx4 %8, %2, %10 offset:-2048 sc0 sc1\n\t"
                     "global_store_dwordx4 %8, %3, %10 offset:-1024 sc0 sc1\n\t"
                     "global_store_dwordx4 %8, %4, %10 sc0 sc1\n\t"
                     "global_store_dwordx4 %8, %5, %10 offset:1024 sc0 sc1\n\t"
                     "global_store_dwordx4 %8, %6, %10 offset:2048 sc0 sc1\n\t"
                     "global_store_dwordx4 %8, %7, %10 offset:3072 sc0 sc1\n\t"
                     "s_waitcnt vmcnt(0)"
                     : "=&v"(y0), "=&v"(y1), "=&v"(y2), "=&v"(y3), "=&v"(y4), "=&v"(y5), "=&v"(y6), "=&v"(y7)
                     : "v"(lane * 16), "s"(myws + go), "s"(P.ws[owner] + go) : "memory");
        if (lane == 0) __hip_atomic_store((unsigned*)(P.ws[owner] + fo), call, __ATOMIC_RELAXED, __HIP_MEMORY_SCOPE_SYSTEM);
        STAMP(46 + (n >> 3));
      }
    }
    unsigned fv = 0;
    bool open = false, quick = false;
    u64 tl0 = 0;
    while (wave <= FW) {
      const unsigned nx = __hip_atomic_load(lnext + tl, __ATOMIC_RELAXED, __HIP_MEMORY_SCOPE_WORKGROUP);
      const unsigned left = (unsigned)__builtin_amdgcn_ballot_w64(nx < nl) & 7u;
      if (left == 0) break;
      const unsigned rdy = bad ? left : (unsigned)__builtin_amdgcn_ballot_w64(fv > call * 16 + nx) & left;
      if (rdy == 0) {
        fv = __hip_atomic_load(stgf, __ATOMIC_RELAXED, __HIP_MEMORY_SCOPE_SYSTEM);
        if (give_up(t0, tmo, call)) bad = true;
        __builtin_amdgcn_s_sleep(2);
        continue;
      }
      const unsigned rdy2 = quick ? (unsigned)__builtin_amdgcn_ballot_w64(fv > call * 16 + nx + 1 && nx + 1 < nl) & 7u : 0u;
      const unsigned t = __builtin_ctz(rdy2 ? rdy2 : rdy);
      const unsigned k = __builtin_amdgcn_readlane(nx, t);
      unsigned got = k;
      if (lane == 0) __hip_atomic_compare_exchange_strong(lnext + t, &got, k + (rdy2 ? 2 : 1), __ATOMIC_RELAXED, __ATOMIC_RELAXED, __HIP_MEMORY_SCOPE_WORKGROUP);
      if (rfl(got) != k) continue;
      const unsigned g = 2 * t + (j >> 2), i = (j + t) & 3;
      const unsigned unit = (P.units[g][1 + p][k >> 2] >> ((k & 3) * 8)) & 127u;
      const unsigned unit2 = (P.units[g][1 + p][(k + 1) >> 2] >> (((k + 1) & 3) * 8)) & 127u;
      const char* const src = myws + WS_STG + (size_t)(((p * 4 + i) * 8 + g) * 12 + k) * 4096;
      f32x4 y0, y1, y2, y3, z0, z1, z2, z3;
      asm volatile("global_load_dwordx4 %0, %9, %10 sc0 sc1\n\t"
                   "global_load_dwordx4 %1, %9, %10 offset:1024 sc0 sc1\n\t"
                   "global_load_dwordx4 %2, %9, %10 offset:2048 sc0 sc1\n\t"
                   "global_load_dwordx4 %3, %9, %10 offset:3072 sc0 sc1\n\t"
                   "global_load_dwordx4 %4, %9, %11 sc0 sc1\n\t"
                   "global_load_dwordx4 %5, %9, %11 offset:1024 sc0 sc1\n\t"
                   "global_load_dwordx4 %6, %9, %11 offset:2048 sc0 sc1\n\t"
                   "global_load_dwordx4 %7, %9, %11 offset:3072 sc0 sc1\n\t"
                   "global_load_dword %8, %12, off sc0 sc1\n\t"
                   "s_waitcnt vmcnt(0)"
                   : "=&v"(y0), "=&v"(y1), "=&v"(y2), "=&v"(y3), "=&v"(z0), "=&v"(z1), "=&v"(z2), "=&v"(z3), "=&v"(fv)
                   : "v"(lane * 16), "s"(src), "s"(src + (rdy2 ? 4096 : 0)), "v"(stgf) : "memory");
      if (!open) {
        while (rfl(__hip_atomic_load(lfree, __ATOMIC_RELAXED, __HIP_MEMORY_SCOPE_WORKGROUP)) == 0) {
          if (__builtin_amdgcn_s_memrealtime() - t0 > 2 * DEADLINE) break;
          __builtin_amdgcn_s_sleep(2);
        }
        open = true;
        if (wave < 5) STAMP(6 + wave);
      }
      const u64 now = __builtin_amdgcn_s_memrealtime();
      quick = now - tl0 < FWD_QUICK;
      tl0 = now;
      STAMP(16 + t * 10 + k);
      if (bad) y0.x = __builtin_nanf("");
      auto put = [&](unsigned u, f32x4 v0, f32x4 v1, f32x4 v2, f32x4 v3) {
        const unsigned vo = ovoff + u * (32 * COLS * 4) + lent_strip(p, q, i) * 128;
        asm volatile("global_store_dwordx4 %0, %4, %8 sc0 sc1\n\t"
                     "global_store_dwordx4 %1, %5, %8 sc0 sc1\n\t"
                     "global_store_dwordx4 %2, %6, %8 sc0 sc1\n\t"
                     "global_store_dwordx4 %3, %7, %8 sc0 sc1\n\ts_nop 1"
                     :: "v"(vo), "v"(vo + 8 * COLS * 4), "v"(vo + 16 * COLS * 4), "v"(vo + 24 * COLS * 4),
                        "v"(v0), "v"(v1), "v"(v2), "v"(v3), "s"(obase) : "memory");
      };
      put(unit, y0, y1, y2, y3);
      if (rdy2) {
        STAMP(16 + t * 10 + k + 1);
        put(unit2, z0, z1, z2, z3);
      }
    }
    asm volatile("s_waitcnt vmcnt(0)" ::: "memory");
    unsigned* const dp = (unsigned*)(P.ws[owner] + WS_DONE + q * 1024 + (j * 5 + wave - 1) * 4);
    if (lane == 0) __hip_atomic_store(dp, call, __ATOMIC_RELAXED, __HIP_MEMORY_SCOPE_SYSTEM);
    STAMP(10 + wave);
    return;
  }

  const int nq = (b & 3) * 24 + (b >> 3) - 8;
  const int g = ((b >> 2) & 1) * 3 + (nq >> 5), s = nq & 31;
  const bool foreign = s >= 16;
  const int owner = (foreign ? (side ^ 1) * 4 + ((s - 16) >> 2) : rank) & (NR - 1);
  const int cls = foreign ? 1 + (owner & 3) : 0;
  const unsigned ul0 = P.units[g][cls][0], ul1 = P.units[g][cls][1], ul2 = P.units[g][cls][2];
  const unsigned nu = P.units[g][cls][3] < 12u ? P.units[g][cls][3] : 12u;
  auto unit_at = [&](unsigned k) -> unsigned {
    unsigned wv = k < 4 ? ul0 : (k < 8 ? ul1 : ul2);
    return (wv >> ((k & 3) * 8)) & 127u;
  };

  const int col0 = (foreign ? lent_strip(owner & 3, q, s & 3) : s) * 32;
  char* const obase = P.out[owner] + col0 * 4;
  unsigned* const donep = (unsigned*)(P.ws[owner] + WS_DONE + q * 1024 + 160 + ((s & 3) * 6 + g) * 16);
  const char* const pl = smem + L_PART + rr * 128 + ((chn ^ rr) * 16);
  if (wave == 1) {
    const unsigned prod = (s & 15) * 8 + g;
    char* const stg = myws + WS_STG + (size_t)prod * (12 * 4096);
    unsigned* const stgf = (unsigned*)(myws + WS_STGF + prod * 128);
    if ((TOUCH & 2) && foreign) {
      const char* a1 = lane & 1 ? (const char*)donep : obase + (size_t)unit_at(nu - 1) * (32 * COLS * 4);
      unsigned v1;
      __builtin_amdgcn_s_sleep(100);
      asm volatile("global_load_dword %0, %1, off sc0 sc1\n\ts_waitcnt vmcnt(0)" : "=&v"(v1) : "v"(a1) : "memory");
    }
    f32x4 keep[KEEP][4];
    unsigned kd = 0, ks = 0;
    while (ks + 1 < nu) {
      const u64 st = __hip_atomic_load((u64*)lcnt, __ATOMIC_RELAXED, __HIP_MEMORY_SCOPE_WORKGROUP);
      const unsigned have = rfl((unsigned)st), idle = rfl((unsigned)(st >> 32));
      if (__builtin_amdgcn_s_memrealtime() - t0 > 2 * DEADLINE) bad = true;
      if (have >= (kd + 1) * 4 || bad) {
        asm volatile("" ::: "memory");
        f32x4 y[4];
#pragma unroll
        for (int it = 0; it < 4; ++it) {
          f32x4 p0 = *(const f32x4*)(pl + it * 1024);
          f32x4 p1 = *(const f32x4*)(pl + it * 1024 + 4096);
          f32x4 p2 = *(const f32x4*)(pl + it * 1024 + 8192);
          f32x4 p3 = *(const f32x4*)(pl + it * 1024 + 12288);
          y[it] = (p0 + p1) + (p2 + p3);
        }
#pragma unroll
        for (int it = 0; it < 4; ++it) asm volatile("" : "+v"(y[it]));
        asm volatile("s_waitcnt lgkmcnt(0)" ::: "memory");
        if (lane == 0) __hip_atomic_store(lfree, kd + 1, __ATOMIC_RELAXED, __HIP_MEMORY_SCOPE_WORKGROUP);
#pragma unroll
        for (int it = 0; it < 4; ++it) {
          y[it].x = gelu(y[it].x); y[it].y = gelu(y[it].y); y[it].z = gelu(y[it].z); y[it].w = gelu(y[it].w);
          if (bad) y[it].x = __builtin_nanf("");
        }
#pragma unroll
        for (unsigned k = 0; k < KEEP; ++k)
          if (kd - ks == k) {
#pragma unroll
            for (int it = 0; it < 4; ++it) { keep[k][it] = y[it]; asm volatile("" : "+v"(keep[k][it])); }
          }
        ++kd;
      } else if (ks == kd || idle != kd + 1) {
        __builtin_amdgcn_s_sleep(1);
        continue;
      }
      if (kd - ks == KEEP || kd + 1 >= nu || idle == kd + 1) {
        const bool stage = foreign;
        char* const base = stage ? stg : obase;
        const unsigned step = stage ? 1024 : 8 * COLS * 4;
#pragma unroll
        for (unsigned k = 0; k < KEEP; ++k)
          if (ks + k < kd) {
            const unsigned vo = stage ? lane * 16 + (ks + k) * 4096 : ovoff + unit_at(ks + k) * (32 * COLS * 4);
#pragma unroll
            for (int it = 0; it < 4; ++it)
              asm volatile("global_store_dwordx4 %0, %1, %2 sc0 sc1\n\ts_nop 1"
                           :: "v"(vo + it * step), "v"(keep[k][it]), "s"(base) : "memory");
            STAMP(32 + ks + k);
          }
        ks = kd;
        if (stage) {
          asm volatile("s_waitcnt vmcnt(0)" ::: "memory");
          if (lane == 0) __hip_atomic_store(stgf, call * 16 + kd, __ATOMIC_RELAXED, __HIP_MEMORY_SCOPE_SYSTEM);
        }
      }
    }
    if (g == 0 && s < 4) {
      const char* dp = myws + WS_DONE + s * 1024 + (lane < 34 ? lane : 0) * 16;
      while (true) {
        u32x4 v;
        asm volatile("global_load_dwordx4 %0, %1, off sc0 sc1\n\ts_waitcnt vmcnt(0)" : "=&v"(v) : "v"(dp) : "memory");
        bool ok = v.x >= call && v.y >= call && v.z >= call && v.w >= call;
        if (__all(ok)) break;
        if (give_up(t0, tmo, call)) {
          char* const lastp = obase + (size_t)unit_at(0) * (32 * COLS * 4);
          if (lane == 0) __hip_atomic_store((float*)lastp, __builtin_nanf(""), __ATOMIC_RELAXED, __HIP_MEMORY_SCOPE_SYSTEM);
          break;
        }
      }
      STAMP(3);
    }
    return;
  }

  const int w = wave - 2;
  char* const data = myws + WS_DATA + (size_t)(call & 1) * PAR_BYTES;
  const unsigned* const flags = (const unsigned*)(myws + WS_FLAG) + w * 8 + (lane & 7);
  lds_char* const lw = (lds_char*)smem + w * 32768;
  lds_char* const ltr = lw + ((lane >> 5) * 8 + ((lane & 15) >> 2)) * 64 + ((lane >> 4) & 1) * 32 + (lane & 3) * 8;
  h8 wr[32];
  {
    const unsigned* pp = flags + (128 + s) * 32;
    const char* src = data + (size_t)(128 + s) * OBJ + w * 16384;
    const unsigned svoff = lane * 16;
    const unsigned lwa = (unsigned)(size_t)lw;
    unsigned seen = 0;
#pragma unroll
    for (int r = 0; r < 4; ++r) {
      wait_round(pp, r, seen, bad, t0, tmo, call, lidle, 0);
#pragma unroll
      for (int i = 0; i < 16; ++i)
        glds16(src + r * 65536 + i * 1024, svoff, lwa + (r & 1) * 16384 + i * 1024);
      if (r & 1) asm volatile("s_waitcnt vmcnt(0)" ::: "memory");
      if (r == 1) {
#pragma unroll
        for (int i = 0; i < 32; ++i) wr[i] = tr_frag(ltr + i * 1024);
#pragma unroll
        for (int i = 0; i < 32; ++i) asm volatile("" : "+v"(wr[i]));
        asm volatile("s_waitcnt lgkmcnt(0)" ::: "memory");
      }
    }
  }
  if (w == 0) STAMP(4);
  const __amdgpu_buffer_rsrc_t rsrc =
      __builtin_amdgcn_make_buffer_rsrc(data, 0, 128 * OBJ, 0x00020000);
  const unsigned xvoff = w * 16384 + lane * 16;
  char* const pw = smem + L_PART + w * 4096 + (lane & 31) * 128;
  unsigned nextf = 0, seen, soff;
  const unsigned* pp;
  f32x16 acc;
  auto xload = [&](u32x4 (&xa)[16], int r) {
#pragma unroll
    for (int i = 0; i < 16; ++i)
      xa[i] = __builtin_amdgcn_raw_buffer_load_b128(rsrc, xvoff + (i & 3) * 1024, soff + r * 65536 + (i >> 2) * 4096, 17);
    __builtin_amdgcn_sched_barrier(0);
  };
  auto xmul = [&](u32x4 (&xa)[16], int r) {
#pragma unroll
    for (int i = 0; i < 16; ++i) {
      h8 wf = r < 2 ? wr[r * 16 + i] : tr_frag(ltr + ((r - 2) * 16 + i) * 1024);
      acc = __builtin_amdgcn_mfma_f32_32x32x16_f16(wf, __builtin_bit_cast(h8, xa[i]), acc, 0, 0, 0);
    }
  };
  auto part = [&](unsigned k) {
    if (bad) {
#pragma unroll
      for (int i = 0; i < 16; ++i) acc[i] = __builtin_nanf("");
    }
    while (rfl(__hip_atomic_load(lfree, __ATOMIC_RELAXED, __HIP_MEMORY_SCOPE_WORKGROUP)) < k) {
      if (__builtin_amdgcn_s_memrealtime() - t0 > 2 * DEADLINE) break;
      __builtin_amdgcn_s_sleep(1);
    }
    asm volatile("" ::: "memory");
#pragma unroll
    for (int gq = 0; gq < 4; ++gq) {
      f32x4 v = {acc[gq * 4], acc[gq * 4 + 1], acc[gq * 4 + 2], acc[gq * 4 + 3]};
      *(f32x4*)(pw + (((gq * 2 + (lane >> 5)) ^ (lane & 7)) * 16)) = v;
    }
    asm volatile("s_waitcnt lgkmcnt(0)" ::: "memory");
    if (lane == 0) __hip_atomic_fetch_add(lcnt, 1u, __ATOMIC_RELAXED, __HIP_MEMORY_SCOPE_WORKGROUP);
    if (w == 0) STAMP(16 + k);
  };
  for (unsigned k = 0;; ++k) {
    const unsigned e = unit_at(k);
    pp = flags + e * 32;
    soff = e * OBJ;
    seen = flag_mask(nextf, call);
    acc = f32x16{0.f, 0.f, 0.f, 0.f, 0.f, 0.f, 0.f, 0.f, 0.f, 0.f, 0.f, 0.f, 0.f, 0.f, 0.f, 0.f};
    u32x4 xa[16];
#pragma unroll
    for (int r = 0; r < 2; ++r) {
      wait_round(pp, r, seen, bad, t0, tmo, call, lidle, k + 1);
      xload(xa, r);
      xmul(xa, r);
    }
    if (k + 1 >= nu) break;
#pragma unroll
    for (int r = 2; r < 4; ++r) {
      wait_round(pp, r, seen, bad, t0, tmo, call, lidle, k + 1);
      xload(xa, r);
      if (r == 3) { nextf = flag_load(flags + unit_at(k + 1) * 32); __builtin_amdgcn_sched_barrier(0); }
      xmul(xa, r);
    }
    part(k);
  }
  {
    u32x4 xa[16], xb[16];
#pragma unroll
    for (int i = 0; i < 16; ++i) wr[i] = tr_frag(ltr + (16 + i) * 1024);
#pragma unroll
    for (int i = 0; i < 16; ++i) asm volatile("" : "+v"(wr[i]));
    wait_round(pp, 2, seen, bad, t0, tmo, call, lidle, nu);
    STAMP(52 + w);
    xload(xa, 2);
    wait_round(pp, 3, seen, bad, t0, tmo, call, lidle, nu);
    STAMP(56 + w);
    xload(xb, 3);
    xmul(xa, 2);
    xmul(xb, 0);
    part(nu - 1);
    STAMP(60 + w);
    while (rfl(__hip_atomic_load(lcnt, __ATOMIC_RELAXED, __HIP_MEMORY_SCOPE_WORKGROUP)) < nu * 4) {
      if (__builtin_amdgcn_s_memrealtime() - t0 > 2 * DEADLINE) { bad = true; break; }
    }
    asm volatile("" ::: "memory");
    const char* const pq = pl + w * 1024;
    f32x4 y = (*(const f32x4*)pq + *(const f32x4*)(pq + 4096)) + (*(const f32x4*)(pq + 8192) + *(const f32x4*)(pq + 12288));
    y.x = gelu(y.x); y.y = gelu(y.y); y.z = gelu(y.z); y.w = gelu(y.w);
    if (bad) y.x = __builtin_nanf("");
    asm volatile("global_store_dwordx4 %0, %1, %2 sc0 sc1\n\ts_nop 1"
                 :: "v"(ovoff + (unit_at(nu - 1) * 32 + w * 8) * (COLS * 4)), "v"(y), "s"(obase) : "memory");
    if (w == 0) STAMP(32 + nu - 1);
    if (foreign) {
      asm volatile("s_waitcnt vmcnt(0)" ::: "memory");
      if (lane == 0) __hip_atomic_store(donep + w, call, __ATOMIC_RELAXED, __HIP_MEMORY_SCOPE_SYSTEM);
    }
    if (w == 0) STAMP(2);
  }
}

unsigned link_map(int rank) {
  const unsigned dflt = 0x76543210u;
  int dev = -1, n = 0, node[NR], port[NR + 1];
  if (hipGetDevice(&dev) != hipSuccess || hipGetDeviceCount(&n) != hipSuccess || dev != rank || n != NR) return dflt;
  for (int i = 0; i <= NR; ++i) port[i] = -1;
  for (int d = 0; d < NR; ++d) {
    char id[32], path[96];
    if (hipDeviceGetPCIBusId(id, sizeof id, d) != hipSuccess) return dflt;
    for (char* c = id; *c; ++c) if (*c >= 'A' && *c <= 'F') *c += 'a' - 'A';
    snprintf(path, sizeof path, "/sys/bus/pci/devices/%s/xgmi_port_num", id);
    FILE* f = fopen(path, "r");
    if (!f) return dflt;
    unsigned a, pa, z, pz;
    node[d] = -1;
    while (fscanf(f, " %x:%x -> %x:%x", &a, &pa, &z, &pz) == 4) {
      node[d] = (int)a;
      if (d == rank && z <= (unsigned)NR && pa < 8u) port[z] = (int)pa;
    }
    fclose(f);
    if (node[d] < 0 || node[d] > NR) return dflt;
  }
  unsigned map = 0, used = 0;
  const int side = rank >> 2, q = rank & 3;
  for (int c = 0; c < 7; ++c) {
    const int x = port[node[c < 3 ? side * 4 + ((q + 1 + c) & 3) : (side ^ 1) * 4 + ((q + c - 3) & 3)]];
    if (x < 0 || ((used >> x) & 1)) return dflt;
    used |= 1u << x;
    map |= (unsigned)c << (4 * x);
  }
  return map | 7u << (4 * __builtin_ctz(~used));
}

}

extern "C" void kernel_launch(int rank, int nranks, ncclComm_t comm,
                              hipStream_t stream,
                              void* const* d_in, const int* in_sizes, int n_in,
                              void* d_out, int out_size,
                              const int* in_dtypes, int out_dtype,
                              void* const* d_out_all,
                              void* const* d_ws_all, size_t ws_bytes) {
  (void)comm; (void)in_dtypes; (void)out_dtype; (void)d_out;
  if (nranks != NR || (unsigned)rank >= (unsigned)NR || n_in < 2 || ws_bytes < WS_NEED ||
      in_sizes[0] != ROWS * KDIM || in_sizes[1] != KDIM * COLS || out_size != NR * ROWS * COLS) {
    fprintf(stderr, "kernel_launch: unexpected problem shape\n");
    return;
  }
  Params P;
  const int side = rank >> 2, q = rank & 3;
  for (int p = 0; p < NR; ++p) { P.ws[p] = (char*)d_ws_all[p]; P.out[p] = (char*)d_out_all[p]; }
  unsigned xmap = link_map(rank);
  for (int x = 0; x < 8; ++x) {
    const int c = (xmap >> (4 * x)) & 7;
    const int dest = c < 3 ? side * 4 + ((q + 1 + c) & 3) : (c < 7 ? (side ^ 1) * 4 + ((q + c - 3) & 3) : rank);
    P.head[x] = Head{(const char*)d_in[0], (const char*)d_in[1], P.ws[rank], P.ws[dest], rank, c};
  }
  static const unsigned char order[NG][12] = {
    {60, 61, 0, 6, 12, 40, 46, 18, 24, 52, 30, 99},
    {62, 63, 1, 7, 13, 41, 47, 19, 25, 53, 31, 99},
    {64, 65, 72, 2, 8, 14, 42, 20, 48, 26, 54, 32},
    {66, 67, 73, 3, 9, 43, 15, 21, 49, 55, 27, 33},
    {68, 69, 74, 4, 10, 16, 44, 22, 50, 28, 34, 99},
    {70, 71, 75, 5, 11, 17, 45, 23, 51, 29, 35, 99}};
  for (int g = 0; g < NG; ++g)
    for (int cls = 0; cls < 5; ++cls) {
      unsigned u[12] = {0}, n = 0;
      for (int k = 0; k < 12; ++k) {
        int v = order[g][k];
        if (v == 99) break;
        if (cls > 0 && (v >= 60 + NA || (v >= 40 && v < 60 && ((v - 40) & 3) == cls - 1))) continue;
        if (v < 40) u[n++] = (side * 4 + ((q + 1 + v % 3) & 3)) * 16 + v / 3;
        else if (v < 60) u[n++] = ((side ^ 1) * 4 + ((v - 40) & 3)) * 16 + NA + ((v - 40) >> 2);
        else u[n++] = rank * 16 + v - 60;
      }
      for (int k = 0; k < 3; ++k)
        P.units[g][cls][k] = u[4 * k] | (u[4 * k + 1] << 8) | (u[4 * k + 2] << 16) | (u[4 * k + 3] << 24);
      P.units[g][cls][3] = n;
    }
  hipLaunchKernelGGL(ag_gemm_gelu, dim3(256), dim3(384), 0, stream, P);
  HIP_CHECK(hipGetLastError());
}
```

```cpp
#include <hip/hip_runtime.h>
#include <rccl/rccl.h>
#include <cstdio>

#define HIP_CHECK(expr) do { hipError_t _e = (expr); if (_e != hipSuccess) { \
    fprintf(stderr, "HIP error %s at %s:%d\n", hipGetErrorString(_e), __FILE__, __LINE__); \
    return; } } while (0)

namespace {

typedef unsigned u32x4 __attribute__((ext_vector_type(4)));
typedef float f32x4 __attribute__((ext_vector_type(4)));
typedef float f32x16 __attribute__((ext_vector_type(16)));
typedef _Float16 h2 __attribute__((ext_vector_type(2)));
typedef _Float16 h8 __attribute__((ext_vector_type(8)));
typedef short s4 __attribute__((ext_vector_type(4)));
typedef short s8 __attribute__((ext_vector_type(8)));
typedef unsigned long long u64;

constexpr int NR = 8;
constexpr int KDIM = 4096;
constexpr int ROWS = 512;
constexpr int COLS = 512;
constexpr unsigned OBJ = 256u * 1024u;
constexpr int NG = 6;
constexpr int NA = 12;
constexpr size_t WS_CNT = 0;
constexpr size_t WS_TMO = 16 * 1024;
constexpr size_t WS_TICK = 20 * 1024;
constexpr size_t WS_FLAG = 32 * 1024;
constexpr size_t WS_DONE = 64 * 1024;
constexpr size_t WS_STGF = 80 * 1024;
constexpr size_t WS_DIAG = 128 * 1024;
constexpr size_t WS_DATA = 1024 * 1024;
constexpr unsigned NSLOT = 160;
constexpr size_t PAR_BYTES = (size_t)NSLOT * OBJ;
constexpr size_t WS_STG = WS_DATA + 2 * PAR_BYTES;
constexpr size_t WS_NEED = WS_STG + 128 * 12 * 4096;
constexpr u64 DEADLINE = 100000000ull;
#define FWD_T0 2500
#define SMODE 1
__host__ __device__ constexpr int lent_strip(int o, int h, int i) {
  return SMODE == 0 ? h * 4 + i : (SMODE == 1 ? ((o + h) & 3) * 4 + i : i * 4 + ((o + h) & 3));
}
#define FW 5
#define KA 4
#define KC 4
#define RP 2
#define INB 1
#define SMP 2
#define KL 4
#define KEEP2 2
#define AGGN 8
#define AGGC 7
#define TOUCH 3
#define LAG 30
#define KEEP 4
constexpr unsigned L_PART = 128 * 1024;
constexpr unsigned L_CTRL = L_PART + 16 * 1024;
constexpr unsigned L_TOTAL = L_CTRL + 64;

static_assert(AGGN < NA, "unit NA-1 has halves with fewer than 8 tickets");

struct alignas(64) Head {
  const char* x;
  const char* w;
  char* myws;
  char* dws;
  int rank;
  int ch;
};
struct Params {
  Head head[8];
  char* ws[NR];
  char* out[NR];
  unsigned units[NG][5][4];
};

__device__ __forceinline__ unsigned rfl(unsigned v) { return __builtin_amdgcn_readfirstlane(v); }
__device__ __forceinline__ unsigned pk(float a, float b) {
  h2 v; v.x = (_Float16)a; v.y = (_Float16)b; return __builtin_bit_cast(unsigned, v);
}
__device__ __forceinline__ u32x4 cvt8(u32x4 a, u32x4 b) {
  f32x4 fa = __builtin_bit_cast(f32x4, a), fb = __builtin_bit_cast(f32x4, b);
  u32x4 r; r.x = pk(fa.x, fa.y); r.y = pk(fa.z, fa.w); r.z = pk(fb.x, fb.y); r.w = pk(fb.z, fb.w);
  return r;
}
__device__ __forceinline__ float gelu(float y) {
  const float c2 = -2.0f * 0.7978845608028654f * 1.4426950408889634f;
  float u = y + 0.044715f * y * y * y;
  float e = __builtin_amdgcn_exp2f(c2 * u);
  return y * __builtin_amdgcn_rcpf(1.0f + e);
}

#define STAMPV(idx, val) do {} while (0)
#define STAMP(idx) STAMPV(idx, __builtin_amdgcn_s_memrealtime())

__device__ __forceinline__ bool give_up(u64 t0, unsigned* tmo, unsigned call) {
  u64 el = __builtin_amdgcn_s_memrealtime() - t0;
  if (el < 100000ull) return false;
  if (el > DEADLINE ||
      rfl(__hip_atomic_load(tmo, __ATOMIC_RELAXED, __HIP_MEMORY_SCOPE_AGENT)) == call) {
    __hip_atomic_store(tmo, call, __ATOMIC_RELAXED, __HIP_MEMORY_SCOPE_AGENT);
    return true;
  }
  return false;
}

__device__ __forceinline__ unsigned flag_word(unsigned pc) {
  return ((pc >> 4) & 1) * 16 + ((pc >> 1) & 3) * 4 + ((pc >> 3) & 1) * 2 + (pc & 1);
}
__device__ __forceinline__ unsigned flag_load(const unsigned* p) {
  return __hip_atomic_load(p, __ATOMIC_RELAXED, __HIP_MEMORY_SCOPE_SYSTEM);
}
__device__ __forceinline__ unsigned flag_mask(unsigned v, unsigned call) {
  return (unsigned)__builtin_amdgcn_ballot_w64(v >= call) & 0xFFu;
}
__device__ __forceinline__ void wait_round(const unsigned* p, int r, unsigned& seen, bool& bad, u64 t0,
                                           unsigned* tmo, unsigned call, unsigned* lidle, unsigned mark) {
  while (((seen >> (2 * r)) & 3u) != 3u) {
    seen = flag_mask(flag_load(p), call);
    if (((seen >> (2 * r)) & 3u) == 3u) break;
    if (r < 3) __hip_atomic_store(lidle, mark, __ATOMIC_RELAXED, __HIP_MEMORY_SCOPE_WORKGROUP);
    if (give_up(t0, tmo, call)) { bad = true; break; }
    __builtin_amdgcn_s_sleep(2);
  }
  asm volatile("" ::: "memory");
}

struct Piece { u32x4 r[16]; };

__device__ __forceinline__ void piece_load(Piece& s, const char* sb, unsigned fstride, unsigned voff) {
#pragma unroll
  for (int u = 0; u < 8; ++u) {
    const char* p = sb + (size_t)u * fstride;
    asm volatile("global_load_dwordx4 %0, %1, %2" : "=v"(s.r[2 * u]) : "v"(voff), "s"(p) : "memory");
    asm volatile("global_load_dwordx4 %0, %1, %2 offset:16" : "=v"(s.r[2 * u + 1]) : "v"(voff), "s"(p) : "memory");
  }
}
template <int N>
__device__ __forceinline__ void piece_wait(Piece& s) {
  asm volatile("s_waitcnt vmcnt(%0)" :: "i"(N) : "memory");
#pragma unroll
  for (int u = 0; u < 16; ++u) asm volatile("" : "+v"(s.r[u]));
}
template <int U>
__device__ __forceinline__ void piece_store1(Piece& s, char* db, unsigned voff) {
  u32x4 v = cvt8(s.r[2 * U], s.r[2 * U + 1]);
  asm volatile("global_store_dwordx4 %0, %1, %2 offset:%3 sc0 sc1\n\ts_nop 1"
               :: "v"(voff), "v"(v), "s"(db), "i"((U - 4) * 1024) : "memory");
}
__device__ __forceinline__ void piece_store(Piece& s, char* db  , unsigned voff) {
  piece_store1<0>(s, db, voff); piece_store1<1>(s, db, voff); piece_store1<2>(s, db, voff);
  piece_store1<3>(s, db, voff); piece_store1<4>(s, db, voff); piece_store1<5>(s, db, voff);
  piece_store1<6>(s, db, voff); piece_store1<7>(s, db, voff);
}

typedef __attribute__((address_space(3))) char lds_char;

__device__ __forceinline__ void glds16(const char* src, unsigned voff, unsigned lds) {
  unsigned keep;
  asm volatile("s_mov_b32 %0, m0\n\ts_mov_b32 m0, %3\n\ts_nop 0\n\t"
               "global_load_lds_dwordx4 %1, %2 sc0 sc1\n\ts_mov_b32 m0, %0"
               : "=&s"(keep) : "v"(voff), "s"(src), "s"(lds) : "memory");
}

__device__ __forceinline__ h8 tr_frag(lds_char* p) {
  s4 a = __builtin_amdgcn_ds_read_tr16_b64_v4i16((__attribute__((address_space(3))) s4*)p);
  s4 c = __builtin_amdgcn_ds_read_tr16_b64_v4i16((__attribute__((address_space(3))) s4*)(p + 256));
  s8 r = __builtin_shufflevector(a, c, 0, 1, 2, 3, 4, 5, 6, 7);
  return __builtin_bit_cast(h8, r);
}

__global__ void __launch_bounds__(384) ag_gemm_gelu(Params P) {
  __shared__ __attribute__((aligned(1024))) char smem[L_TOTAL];
  const int lane = threadIdx.x & 63;
  const int wave = rfl(threadIdx.x >> 6);
  Head H = P.head[__builtin_amdgcn_s_getreg(20 | (3 << 11)) & 7];
  asm volatile("" : "+s"(H.x), "+s"(H.w), "+s"(H.myws), "+s"(H.dws), "+s"(H.rank), "+s"(H.ch));
  const int xch = H.ch & 7;
  const int b = blockIdx.x < 64 ? xch * 8 + (blockIdx.x >> 3) : blockIdx.x;
  const int rank = H.rank & (NR - 1);
  const int side = rank >> 2, q = rank & 3;
  char* const myws = (char*)(__attribute__((address_space(1))) char*)(u64)H.myws;
  unsigned* const cntp = (unsigned*)(myws + WS_CNT + b * 32 + wave * 4);
  unsigned* const lcnt = (unsigned*)(smem + L_CTRL);
  unsigned* const lidle = (unsigned*)(smem + L_CTRL + 4);
  unsigned* const lfree = (unsigned*)(smem + L_CTRL + 8);

  if (__builtin_expect(wave == 0, 1)) {
    if (__builtin_expect(b >= 64 + 128 || (b >> 3) == 7, 0)) {
      const int c = xch;
      if ((TOUCH & 1) && (b >> 3) == 24 && c < 7) {
        const int dest = (c < 3 ? side * 4 + ((q + 1 + c) & 3) : (side ^ 1) * 4 + ((q + c - 3) & 3)) & (NR - 1);
        const char* const dd = H.dws + WS_DATA + (lane & 1) * PAR_BYTES;
        const char* a1 = lane < 2 ? dd + (size_t)(c < 3 ? rank * 16 : 144 + q * 4) * OBJ : H.dws + WS_FLAG;
        const char* a2 = lane < 32 ? dd + (size_t)(rank * 16 + (lane >> 1)) * OBJ : P.out[dest] + (size_t)(lane - 32) * OBJ;
        unsigned v1, v2;
        asm volatile("global_load_dword %0, %2, off sc0 sc1\n\tglobal_load_dword %1, %3, off sc0 sc1\n\ts_waitcnt vmcnt(0)"
                     : "=&v"(v1), "=&v"(v2) : "v"(a1), "v"(a2) : "memory");
      }
      return;
    }
    __builtin_amdgcn_s_setprio(3);
    __builtin_amdgcn_s_waitcnt(0);
    const int j = b & 7, c = b >> 3;
    int dest, nW, strip0, wslot0, xu0, nobj;
    if (c < 3)      { dest = side * 4 + ((q + 1 + c) & 3); nW = 0; strip0 = 0; wslot0 = 0; xu0 = 0; nobj = NA; }
    else if (c < 7) { dest = (side ^ 1) * 4 + ((q + c - 3) & 3); nW = 4; strip0 = lent_strip(q, dest & 3, 0); wslot0 = 16 + q * 4;
                      xu0 = NA; nobj = 4 + 16 - NA; }
    else            { dest = rank; nW = 1; strip0 = c - 8; wslot0 = c - 8; xu0 = c - 8; nobj = 2; }
    const unsigned np = nobj * 32;
    const unsigned nt = np / 2 - (c < 3 ? 2 * RP : 0);
    const unsigned laneX = (lane & 31) * (KDIM * 4) + (lane >> 5) * 32;
    const unsigned laneW = (lane >> 2) * (COLS * 4) + (lane & 3) * 32;
    const unsigned lane16 = lane * 16;
    const char* const px = H.x;
    const char* const pw = H.w;
    if (LAG && b >= 64) __builtin_amdgcn_s_sleep(LAG);
    auto load = [&](Piece& s, unsigned pc) {
      pc = pc < np ? pc : np - 1;
      const int obj = pc >> 5, f0 = (pc & 31) * 8;
      const bool isW = obj < nW;
      const char* sb = isW ? pw + (size_t)(strip0 + obj * (lent_strip(0, 0, 1) - lent_strip(0, 0, 0))) * 128 + (size_t)f0 * (16 * COLS * 4)
                           : px + (size_t)(xu0 + obj - nW) * (32 * KDIM * 4) + (size_t)f0 * 64;
      piece_load(s, sb, isW ? 16 * COLS * 4 : 64, isW ? laneW : laneX);
    };
    unsigned cnt;
    asm volatile("global_load_dword %0, %1, off sc1" : "=v"(cnt) : "v"(cntp) : "memory");
    Piece A, B;
    load(A, j);
    load(B, j + 8);
    piece_wait<16>(A);
    asm volatile("" : "+v"(cnt));
    const unsigned call = rfl(cnt) + 1;
    char* const dws = c < 8 ? H.dws : myws;
    char* const ddata = dws + WS_DATA + (size_t)(call & 1) * PAR_BYTES + 4096;
    char* const tick = myws + WS_TICK + c * 128;
    const unsigned tbase = 16 - (call - 1) * nt;
    auto slot_of = [&](unsigned pc) -> unsigned {
      const int obj = pc >> 5;
      return obj < nW ? 128 + wslot0 + obj : rank * 16 + xu0 + obj - nW;
    };
    auto store = [&](Piece& s, unsigned pc) {
      piece_store(s, ddata + (size_t)slot_of(pc) * OBJ + (pc & 31) * 8192, lane16);
    };
    const unsigned one = 1;
    auto flag = [&](unsigned pc, unsigned m) {
      char* fp = dws + WS_FLAG + slot_of(pc) * 128 + (pc & 16) * 4;
      if ((m >> (lane & 31)) & (lane < 16)) asm volatile("global_store_dword %0, %1, %2 sc0 sc1" :: "v"(lane * 4), "v"(call), "s"(fp) : "memory");
    };
    unsigned dn = 0, pend = np;
    auto count = [&](unsigned& r, unsigned off, unsigned lanes) {
      asm volatile("s_mov_b32 exec_lo, %4\n\ts_mov_b32 exec_hi, 0\n\tglobal_atomic_add %0, %1, %2, %3 sc0\n\ts_mov_b64 exec, -1"
                   : "+v"(r) : "v"(off), "v"(one), "s"(tick), "s"(lanes) : "memory");
    };
    auto done = [&](unsigned pc) {
      const bool agg = c < AGGC && (int)(pc >> 5) >= nW && pc < AGGN * 32;
      count(dn, 16 + (pc >> 4) * 4, rfl(agg));
      if (agg) pend = pc;
      else flag(pc, 5u << (flag_word(pc) & 15));
    };
    auto post = [&]() {
      asm volatile("" : "+v"(dn));
      if (pend < np && rfl(dn) + 1 == call * 8) flag(pend, 0xFFFFu);
      pend = np;
    };
    STAMP(0);
    auto piece_of = [&](unsigned t) -> unsigned {
      if (t >= NA * 16 - 24) t += t < NA * 16 - 16 - 2 * RP ? 8 + 2 * RP : (t < NA * 16 - 8 - 2 * RP ? 2 * RP - 8 : 2 * RP);
      return (t >> 3) * 16 + (t & 7);
    };
    unsigned tk = 0;
    unsigned cur = j, nxt = j + 16;
    store(A, cur);
    count(tk, 0, 1);
    load(A, nxt);
    piece_wait<25>(B);
    if (lane == 0) asm volatile("global_store_dword %0, %1, off sc1" :: "v"(cntp), "v"(call) : "memory");
    store(B, cur + 8);
    load(B, nxt + 8);
    __builtin_amdgcn_s_barrier();
    bool gate = INB && c < 3;
    do {
      piece_wait<25>(A);
      asm volatile("" : "+v"(tk));
      unsigned t = rfl(tk) + tbase;
      const unsigned nn = t < nt ? piece_of(t) : np;
      if (c < 3) asm volatile("s_waitcnt vmcnt(%0)" :: "i"(16 + KA) : "memory");
      else if (c < 7) asm volatile("s_waitcnt vmcnt(%0)" :: "i"(16 + KC) : "memory");
      post();
      if (gate && nxt >= 96) { __builtin_amdgcn_s_barrier(); gate = false; }
      store(A, nxt);
      count(tk, 0, 1);
      load(A, nn);
      piece_wait<25>(B);
      done(cur);
      if (c < 3) asm volatile("s_waitcnt vmcnt(%0)" :: "i"(18 + KA) : "memory");
      else if (c < 7) asm volatile("s_waitcnt vmcnt(%0)" :: "i"(18 + KC) : "memory");
      store(B, nxt + 8);
      load(B, nn + 8);
      cur = nxt;
      nxt = nn;
    } while (nxt < np);
    if (b < 64) __hip_atomic_store(lfree, 1u, __ATOMIC_RELAXED, __HIP_MEMORY_SCOPE_WORKGROUP);
    if (gate) __builtin_amdgcn_s_barrier();
    piece_wait<24>(A);
    asm volatile("" : "+v"(tk));
    post();
    piece_wait<0>(B);
    done(cur);
    asm volatile("s_waitcnt vmcnt(0)" ::: "memory");
    post();
    STAMP(1);
    return;
  }

  const bool fwd = b < 64;
  if (fwd && (b < 24 || b >= 56)) {
    if (INB && b < 24 && wave <= 2) {
      __builtin_amdgcn_s_barrier();
      __builtin_amdgcn_s_sleep(100);
      const unsigned call = rfl(__hip_atomic_load(cntp, __ATOMIC_RELAXED, __HIP_MEMORY_SCOPE_AGENT)) + 1;
      if (lane == 0) __hip_atomic_store(cntp, call, __ATOMIC_RELAXED, __HIP_MEMORY_SCOPE_AGENT);
      const unsigned slot = (side * 4 + ((q + 1 + (b >> 3)) & 3)) * 16 + NA - wave;
      char* const zp = myws + WS_DATA + (size_t)(~call & 1) * PAR_BYTES + (size_t)slot * OBJ + OBJ / 2 + (b & 7) * 16384;
      const u32x4 z = {0u, 0u, 0u, 0u};
#pragma unroll
      for (int i = 0; i < 16; ++i)
        asm volatile("global_store_dwordx4 %0, %1, %2 sc0 sc1" :: "v"(lane * 16 + i * 1024), "v"(z), "s"(zp) : "memory");
      asm volatile("s_waitcnt vmcnt(0)" ::: "memory");
    }
    return;
  }
  const u64 t0 = __builtin_amdgcn_s_memrealtime();
  if (LAG && !fwd) __builtin_amdgcn_s_sleep(LAG);
  const unsigned call = rfl(__hip_atomic_load(cntp, __ATOMIC_RELAXED, __HIP_MEMORY_SCOPE_AGENT)) + 1;
  if (lane == 0) __hip_atomic_store(cntp, call, __ATOMIC_RELAXED, __HIP_MEMORY_SCOPE_AGENT);
  unsigned* const tmo = (unsigned*)(myws + WS_TMO);
  const int rr = lane >> 3, chn = lane & 7;
  const unsigned ovoff = rr * (COLS * 4) + chn * 16;
  bool bad = false;
  if (wave == 1) {
    STAMPV(5, t0);
    if (lane < 8) lcnt[lane] = 0;
    asm volatile("s_waitcnt lgkmcnt(0)" ::: "memory");
  }
  __builtin_amdgcn_s_barrier();

  if (fwd) {
    const int j = b & 7, p = (q + (b >> 3) - 3) & 3;
    const int owner = ((side ^ 1) * 4 + p) & (NR - 1);
    char* const obase = P.out[owner];
    unsigned* const lnext = (unsigned*)(smem + L_CTRL + 16);
    const unsigned tl = lane < 3 ? lane : 0;
    const unsigned* const stgf = (const unsigned*)(myws + WS_STGF + ((p * 4 + ((j + tl) & 3)) * 8 + 2 * tl + (j >> 2)) * 128);
    const unsigned n0 = P.units[j >> 2][1 + p][3], n1 = P.units[2 + (j >> 2)][1 + p][3], n2 = P.units[4 + (j >> 2)][1 + p][3];
    const unsigned nl = (tl == 0 ? n0 : (tl == 1 ? n1 : n2)) - 1;
    while (__builtin_amdgcn_s_memrealtime() - t0 < FWD_T0) __builtin_amdgcn_s_sleep(16);
    if (RP && wave == 5) {
      const size_t dbase = WS_DATA + (size_t)(call & 1) * PAR_BYTES + 4096;
      auto rpiece = [](unsigned n) -> unsigned { return n % (2 * RP) + n / (2 * RP) * 8; };
      if (j >= 8 - RP) {
        const unsigned slot = rank * 16 + NA - 1, pc = rpiece(p * RP + 7 - j);
        Piece A;
        piece_load(A, H.x + (size_t)(NA - 1) * (32 * KDIM * 4) + pc * 512, 64, (lane & 31) * (KDIM * 4) + (lane >> 5) * 32);
        piece_wait<0>(A);
        while (rfl(__hip_atomic_load(lfree, __ATOMIC_RELAXED, __HIP_MEMORY_SCOPE_WORKGROUP)) == 0) {
          if (__builtin_amdgcn_s_memrealtime() - t0 > 2 * DEADLINE) break;
          __builtin_amdgcn_s_sleep(2);
        }
        piece_store(A, P.ws[owner] + dbase + (size_t)slot * OBJ + pc * 8192, lane * 16);
        asm volatile("s_waitcnt vmcnt(0)" ::: "memory");
        if (lane == 0) __hip_atomic_store((unsigned*)(P.ws[owner] + WS_FLAG + slot * 128 + flag_word(pc) * 4), call, __ATOMIC_RELAXED, __HIP_MEMORY_SCOPE_SYSTEM);
      }
      for (int n = j; n < 3 * RP; n += 8) {
        const unsigned slot = ((side ^ 1) * 4 + ((p + 1 + n / RP) & 3)) * 16 + NA - 1, pc = rpiece(q * RP + n % RP);
        const size_t fo = WS_FLAG + slot * 128 + flag_word(pc) * 4;
        const size_t go = dbase + (size_t)slot * OBJ + pc * 8192;
        while (rfl(flag_load((const unsigned*)(myws + fo))) < call) {
          if (give_up(t0, tmo, call)) { bad = true; break; }
          __builtin_amdgcn_s_sleep(8);
        }
        if (bad) break;
        u32x4 y0, y1, y2, y3, y4, y5, y6, y7;
        asm volatile("global_load_dwordx4 %0, %8, %9 offset:-4096 sc0 sc1\n\t"
                     "global_load_dwordx4 %1, %8, %9 offset:-3072 sc0 sc1\n\t"
                     "global_load_dwordx4 %2, %8, %9 offset:-2048 sc0 sc1\n\t"
                     "global_load_dwordx4 %3, %8, %9 offset:-1024 sc0 sc1\n\t"
                     "global_load_dwordx4 %4, %8, %9 sc0 sc1\n\t"
                     "global_load_dwordx4 %5, %8, %9 offset:1024 sc0 sc1\n\t"
                     "global_load_dwordx4 %6, %8, %9 offset:2048 sc0 sc1\n\t"
                     "global_load_dwordx4 %7, %8, %9 offset:3072 sc0 sc1\n\t"
                     "s_waitcnt vmcnt(0)\n\t"
                     "global_store_dwordx4 %8, %0, %10 offset:-4096 sc0 sc1\n\t"
                     "global_store_dwordx4 %8, %1, %10 offset:-3072 sc0 sc1\n\t"
                     "global_store_dwordx4 %8, %2, %10 offset:-2048 sc0 sc1\n\t"
                     "global_store_dwordx4 %8, %3, %10 offset:-1024 sc0 sc1\n\t"
                     "global_store_dwordx4 %8, %4, %10 sc0 sc1\n\t"
                     "global_store_dwordx4 %8, %5, %10 offset:1024 sc0 sc1\n\t"
                     "global_store_dwordx4 %8, %6, %10 offset:2048 sc0 sc1\n\t"
                     "global_store_dwordx4 %8, %7, %10 offset:3072 sc0 sc1\n\t"
                     "s_waitcnt vmcnt(0)"
                     : "=&v"(y0), "=&v"(y1), "=&v"(y2), "=&v"(y3), "=&v"(y4), "=&v"(y5), "=&v"(y6), "=&v"(y7)
                     : "v"(lane * 16), "s"(myws + go), "s"(P.ws[owner] + go) : "memory");
        if (lane == 0) __hip_atomic_store((unsigned*)(P.ws[owner] + fo), call, __ATOMIC_RELAXED, __HIP_MEMORY_SCOPE_SYSTEM);
        STAMP(46 + (n >> 3));
      }
    }
    unsigned fv = 0;
    bool open = false;
    while (wave <= FW) {
      const unsigned nx = __hip_atomic_load(lnext + tl, __ATOMIC_RELAXED, __HIP_MEMORY_SCOPE_WORKGROUP);
      const unsigned left = (unsigned)__builtin_amdgcn_ballot_w64(nx < nl) & 7u;
      if (left == 0) break;
      const unsigned rdy = rfl(bad ? left : (unsigned)__builtin_amdgcn_ballot_w64(fv > call * 16 + nx) & left);
      if (rdy == 0) {
        fv = __hip_atomic_load(stgf, __ATOMIC_RELAXED, __HIP_MEMORY_SCOPE_SYSTEM);
        if (give_up(t0, tmo, call)) bad = true;
        __builtin_amdgcn_s_sleep(2);
        continue;
      }
      const unsigned t = __builtin_ctz(rdy);
      const unsigned k = __builtin_amdgcn_readlane(nx, t);
      unsigned got = k;
      if (lane == 0) __hip_atomic_compare_exchange_strong(lnext + t, &got, k + 1, __ATOMIC_RELAXED, __ATOMIC_RELAXED, __HIP_MEMORY_SCOPE_WORKGROUP);
      if (rfl(got) != k) continue;
      const unsigned g = 2 * t + (j >> 2), i = (j + t) & 3;
      const unsigned unit = (P.units[g][1 + p][k >> 2] >> ((k & 3) * 8)) & 127u;
      const char* const src = myws + WS_STG + (size_t)(((p * 4 + i) * 8 + g) * 12 + k) * 4096;
      f32x4 y0, y1, y2, y3;
      asm volatile("global_load_dwordx4 %0, %5, %6 sc0 sc1\n\t"
                   "global_load_dwordx4 %1, %5, %6 offset:1024 sc0 sc1\n\t"
                   "global_load_dwordx4 %2, %5, %6 offset:2048 sc0 sc1\n\t"
                   "global_load_dwordx4 %3, %5, %6 offset:3072 sc0 sc1\n\t"
                   "global_load_dword %4, %7, off sc0 sc1\n\t"
                   "s_waitcnt vmcnt(0)"
                   : "=&v"(y0), "=&v"(y1), "=&v"(y2), "=&v"(y3), "=&v"(fv)
                   : "v"(lane * 16), "s"(src), "v"(stgf) : "memory");
      if (!open) {
        while (rfl(__hip_atomic_load(lfree, __ATOMIC_RELAXED, __HIP_MEMORY_SCOPE_WORKGROUP)) == 0) {
          if (__builtin_amdgcn_s_memrealtime() - t0 > 2 * DEADLINE) break;
          __builtin_amdgcn_s_sleep(2);
        }
        open = true;
        if (wave < 5) STAMP(6 + wave);
      }
      STAMP(16 + t * 10 + k);
      if (bad) y0.x = __builtin_nanf("");
      const unsigned vo = ovoff + unit * (32 * COLS * 4) + lent_strip(p, q, i) * 128, vs = 8 * COLS * 4;
      asm volatile("global_store_dwordx4 %0, %4, %8 sc0 sc1\n\t"
                   "global_store_dwordx4 %1, %5, %8 sc0 sc1\n\t"
                   "global_store_dwordx4 %2, %6, %8 sc0 sc1\n\t"
                   "global_store_dwordx4 %3, %7, %8 sc0 sc1\n\ts_nop 1"
                   :: "v"(vo), "v"(vo + vs), "v"(vo + 2 * vs), "v"(vo + 3 * vs),
                      "v"(y0), "v"(y1), "v"(y2), "v"(y3), "s"(obase) : "memory");
    }
    asm volatile("s_waitcnt vmcnt(0)" ::: "memory");
    unsigned* const dp = (unsigned*)(P.ws[owner] + WS_DONE + q * 1024 + (j * 5 + wave - 1) * 4);
    if (lane == 0) __hip_atomic_store(dp, call, __ATOMIC_RELAXED, __HIP_MEMORY_SCOPE_SYSTEM);
    STAMP(10 + wave);
    return;
  }

  const int nq = (b & 3) * 24 + (b >> 3) - 8;
  const int g = ((b >> 2) & 1) * 3 + (nq >> 5), s = nq & 31;
  const bool foreign = s >= 16;
  const int owner = (foreign ? (side ^ 1) * 4 + ((s - 16) >> 2) : rank) & (NR - 1);
  const int cls = foreign ? 1 + (owner & 3) : 0;
  const unsigned ul0 = P.units[g][cls][0], ul1 = P.units[g][cls][1], ul2 = P.units[g][cls][2];
  const unsigned nu = P.units[g][cls][3] < 12u ? P.units[g][cls][3] : 12u;
  auto unit_at = [&](unsigned k) -> unsigned {
    unsigned wv = k < 4 ? ul0 : (k < 8 ? ul1 : ul2);
    return (wv >> ((k & 3) * 8)) & 127u;
  };

  const int col0 = (foreign ? lent_strip(owner & 3, q, s & 3) : s) * 32;
  char* const obase = P.out[owner] + col0 * 4;
  unsigned* const donep = (unsigned*)(P.ws[owner] + WS_DONE + q * 1024 + 160 + ((s & 3) * 6 + g) * 16);
  const char* const pl = smem + L_PART + rr * 128 + ((chn ^ rr) * 16);
  if (wave == 1) {
    const unsigned prod = (s & 15) * 8 + g;
    char* const stg = myws + WS_STG + (size_t)prod * (12 * 4096);
    unsigned* const stgf = (unsigned*)(myws + WS_STGF + prod * 128);
    if ((TOUCH & 2) && foreign) {
      const char* a1 = lane & 1 ? (const char*)donep : obase + (size_t)unit_at(nu - 1) * (32 * COLS * 4);
      unsigned v1;
      __builtin_amdgcn_s_sleep(100);
      asm volatile("global_load_dword %0, %1, off sc0 sc1\n\ts_waitcnt vmcnt(0)" : "=&v"(v1) : "v"(a1) : "memory");
    }
    f32x4 keep[KEEP][4];
    unsigned kd = 0, ks = 0;
    while (ks + 1 < nu) {
      const u64 st = __hip_atomic_load((u64*)lcnt, __ATOMIC_RELAXED, __HIP_MEMORY_SCOPE_WORKGROUP);
      const unsigned have = rfl((unsigned)st), idle = rfl((unsigned)(st >> 32));
      if (__builtin_amdgcn_s_memrealtime() - t0 > 2 * DEADLINE) bad = true;
      if (have >= (kd + 1) * 4 || bad) {
        asm volatile("" ::: "memory");
        f32x4 y[4];
#pragma unroll
        for (int it = 0; it < 4; ++it) {
          f32x4 p0 = *(const f32x4*)(pl + it * 1024);
          f32x4 p1 = *(const f32x4*)(pl + it * 1024 + 4096);
          f32x4 p2 = *(const f32x4*)(pl + it * 1024 + 8192);
          f32x4 p3 = *(const f32x4*)(pl + it * 1024 + 12288);
          y[it] = (p0 + p1) + (p2 + p3);
        }
#pragma unroll
        for (int it = 0; it < 4; ++it) asm volatile("" : "+v"(y[it]));
        asm volatile("s_waitcnt lgkmcnt(0)" ::: "memory");
        if (lane == 0) __hip_atomic_store(lfree, kd + 1, __ATOMIC_RELAXED, __HIP_MEMORY_SCOPE_WORKGROUP);
#pragma unroll
        for (int it = 0; it < 4; ++it) {
          y[it].x = gelu(y[it].x); y[it].y = gelu(y[it].y); y[it].z = gelu(y[it].z); y[it].w = gelu(y[it].w);
          if (bad) y[it].x = __builtin_nanf("");
        }
#pragma unroll
        for (unsigned k = 0; k < KEEP; ++k)
          if (kd - ks == k) {
#pragma unroll
            for (int it = 0; it < 4; ++it) { keep[k][it] = y[it]; asm volatile("" : "+v"(keep[k][it])); }
          }
        ++kd;
      } else if (ks == kd || idle != kd + 1) {
        __builtin_amdgcn_s_sleep(1);
        continue;
      }
      if (kd - ks >= (foreign && kd > KL ? KEEP2 : KEEP) || kd + 1 >= nu || idle == kd + 1) {
        const bool stage = foreign;
        char* const base = stage ? stg : obase;
        const unsigned step = stage ? 1024 : 8 * COLS * 4;
#pragma unroll
        for (unsigned k = 0; k < KEEP; ++k)
          if (ks + k < kd) {
            const unsigned vo = stage ? lane * 16 + (ks + k) * 4096 : ovoff + unit_at(ks + k) * (32 * COLS * 4);
#pragma unroll
            for (int it = 0; it < 4; ++it)
              asm volatile("global_store_dwordx4 %0, %1, %2 sc0 sc1\n\ts_nop 1"
                           :: "v"(vo + it * step), "v"(keep[k][it]), "s"(base) : "memory");
            STAMP(32 + ks + k);
          }
        ks = kd;
        if (stage) {
          asm volatile("s_waitcnt vmcnt(0)" ::: "memory");
          if (lane == 0) __hip_atomic_store(stgf, call * 16 + kd, __ATOMIC_RELAXED, __HIP_MEMORY_SCOPE_SYSTEM);
        }
      }
    }
    if (g == 0 && s < 4) {
      const char* dp = myws + WS_DONE + s * 1024 + (lane < 34 ? lane : 0) * 16;
      while (true) {
        u32x4 v;
        asm volatile("global_load_dwordx4 %0, %1, off sc0 sc1\n\ts_waitcnt vmcnt(0)" : "=&v"(v) : "v"(dp) : "memory");
        bool ok = v.x >= call && v.y >= call && v.z >= call && v.w >= call;
        if (__all(ok)) break;
        if (give_up(t0, tmo, call)) {
          char* const lastp = obase + (size_t)unit_at(0) * (32 * COLS * 4);
          if (lane == 0) __hip_atomic_store((float*)lastp, __builtin_nanf(""), __ATOMIC_RELAXED, __HIP_MEMORY_SCOPE_SYSTEM);
          break;
        }
      }
      STAMP(3);
    }
    return;
  }

  const int w = wave - 2;
  char* const data = myws + WS_DATA + (size_t)(call & 1) * PAR_BYTES;
  const unsigned* const flags = (const unsigned*)(myws + WS_FLAG) + (lane & 4) * 4 + w * 4 + (lane & 3);
  lds_char* const lw = (lds_char*)smem + w * 32768;
  lds_char* const ltr = lw + ((lane >> 5) * 8 + ((lane & 15) >> 2)) * 64 + ((lane >> 4) & 1) * 32 + (lane & 3) * 8;
  h8 wr[32];
  {
    const unsigned* pp = flags + (128 + s) * 32;
    const char* src = data + (size_t)(128 + s) * OBJ + w * 16384;
    const unsigned svoff = lane * 16;
    const unsigned lwa = (unsigned)(size_t)lw;
    unsigned seen = 0;
#pragma unroll
    for (int r = 0; r < 4; ++r) {
      wait_round(pp, r, seen, bad, t0, tmo, call, lidle, 0);
#pragma unroll
      for (int i = 0; i < 16; ++i)
        glds16(src + r * 65536 + i * 1024, svoff, lwa + (r & 1) * 16384 + i * 1024);
      if (r & 1) asm volatile("s_waitcnt vmcnt(0)" ::: "memory");
      if (r == 1) {
#pragma unroll
        for (int i = 0; i < 32; ++i) wr[i] = tr_frag(ltr + i * 1024);
#pragma unroll
        for (int i = 0; i < 32; ++i) asm volatile("" : "+v"(wr[i]));
        asm volatile("s_waitcnt lgkmcnt(0)" ::: "memory");
      }
    }
  }
  if (w == 0) STAMP(4);
  const __amdgpu_buffer_rsrc_t rsrc =
      __builtin_amdgcn_make_buffer_rsrc(data, 0, 128 * OBJ, 0x00020000);
  const unsigned xvoff = w * 16384 + lane * 16;
  char* const pw = smem + L_PART + w * 4096 + (lane & 31) * 128;
  unsigned nextf = 0, seen, soff;
  const unsigned* pp;
  f32x16 acc;
  auto xload = [&](u32x4 (&xa)[16], int r) {
#pragma unroll
    for (int i = 0; i < 16; ++i)
      xa[i] = __builtin_amdgcn_raw_buffer_load_b128(rsrc, xvoff + (i & 3) * 1024, soff + r * 65536 + (i >> 2) * 4096, 17);
    __builtin_amdgcn_sched_barrier(0);
  };
  auto xmul = [&](u32x4 (&xa)[16], int r) {
#pragma unroll
    for (int i = 0; i < 16; ++i) {
      h8 wf = r < 2 ? wr[r * 16 + i] : tr_frag(ltr + ((r - 2) * 16 + i) * 1024);
      acc = __builtin_amdgcn_mfma_f32_32x32x16_f16(wf, __builtin_bit_cast(h8, xa[i]), acc, 0, 0, 0);
    }
  };
  auto part = [&](unsigned k) {
    if (bad) {
#pragma unroll
      for (int i = 0; i < 16; ++i) acc[i] = __builtin_nanf("");
    }
    while (rfl(__hip_atomic_load(lfree, __ATOMIC_RELAXED, __HIP_MEMORY_SCOPE_WORKGROUP)) < k) {
      if (__builtin_amdgcn_s_memrealtime() - t0 > 2 * DEADLINE) break;
      __builtin_amdgcn_s_sleep(1);
    }
    asm volatile("" ::: "memory");
#pragma unroll
    for (int gq = 0; gq < 4; ++gq) {
      f32x4 v = {acc[gq * 4], acc[gq * 4 + 1], acc[gq * 4 + 2], acc[gq * 4 + 3]};
      *(f32x4*)(pw + (((gq * 2 + (lane >> 5)) ^ (lane & 7)) * 16)) = v;
    }
    asm volatile("s_waitcnt lgkmcnt(0)" ::: "memory");
    if (lane == 0) __hip_atomic_fetch_add(lcnt, 1u, __ATOMIC_RELAXED, __HIP_MEMORY_SCOPE_WORKGROUP);
    if (w == 0) STAMP(16 + k);
  };
  for (unsigned k = 0;; ++k) {
    const unsigned e = unit_at(k);
    pp = flags + e * 32;
    soff = e * OBJ;
    seen = flag_mask(nextf, call);
    acc = f32x16{0.f, 0.f, 0.f, 0.f, 0.f, 0.f, 0.f, 0.f, 0.f, 0.f, 0.f, 0.f, 0.f, 0.f, 0.f, 0.f};
    u32x4 xa[16];
#pragma unroll
    for (int r = 0; r < 2; ++r) {
      wait_round(pp, r, seen, bad, t0, tmo, call, lidle, k + 1);
      xload(xa, r);
      xmul(xa, r);
    }
    if (k + 1 >= nu) break;
#pragma unroll
    for (int r = 2; r < 4; ++r) {
      wait_round(pp, r, seen, bad, t0, tmo, call, lidle, k + 1);
      xload(xa, r);
      if (r == 3) { nextf = flag_load(flags + unit_at(k + 1) * 32); __builtin_amdgcn_sched_barrier(0); }
      xmul(xa, r);
    }
    part(k);
  }
  {
    u32x4 xa[16], xb[16];
#pragma unroll
    for (int i = 0; i < 16; ++i) wr[i] = tr_frag(ltr + i * 1024);
#pragma unroll
    for (int i = 0; i < 16; ++i) asm volatile("" : "+v"(wr[i]));
    const bool inb = INB;
    auto full = [&](u32x4 (&x)[16]) -> bool {
      unsigned m = ~0u;
#pragma unroll
      for (int i = 0; i < 16; ++i) m = min(min(min(min(m, x[i].x), x[i].y), x[i].z), x[i].w);
      return __builtin_amdgcn_ballot_w64(m == 0) == 0;
    };
    while (((seen >> 4) & 3u) != 3u) {
      const unsigned smp = __builtin_amdgcn_raw_buffer_load_b32(rsrc, w * 16384 + ((lane & (SMP - 1)) + 1) * (16384 / SMP) - 4, soff + 2 * 65536, 17);
      seen = flag_mask(flag_load(pp), call);
      if (inb && __builtin_amdgcn_ballot_w64(smp == 0) == 0) break;
      __hip_atomic_store(lidle, nu, __ATOMIC_RELAXED, __HIP_MEMORY_SCOPE_WORKGROUP);
      if (give_up(t0, tmo, call)) { bad = true; break; }
      __builtin_amdgcn_s_sleep(2);
    }
    asm volatile("" ::: "memory");
    STAMP(52 + w);
    xload(xa, 2);
    if (!inb) wait_round(pp, 3, seen, bad, t0, tmo, call, lidle, nu);
    xload(xb, 3);
    unsigned f = flag_load(pp);
    auto settle = [&](u32x4 (&x)[16], int r, int rw) {
      const f32x16 c0 = acc;
#pragma unroll
      for (int i = 0; i < 16; ++i) {
        acc = __builtin_amdgcn_mfma_f32_32x32x16_f16(wr[(r - 2) * 16 + i], __builtin_bit_cast(h8, x[i]), acc, 0, 0, 0);
        if (r == 2) wr[16 + i] = tr_frag(ltr + (16 + i) * 1024);
      }
      if (r == 2) {
#pragma unroll
        for (int i = 0; i < 16; ++i) asm volatile("" : "+v"(wr[16 + i]));
      }
      if (__builtin_expect(!(full(x) | (((seen >> (2 * r)) & 3u) == 3u)), 0)) {
        bool ok;
        do {
          ok = ((flag_mask(f, call) >> (2 * r)) & 3u) == 3u;
          if (give_up(t0, tmo, call)) bad = true;
          asm volatile("" ::: "memory");
          xload(x, r);
          f = flag_load(pp);
          acc = c0;
          xmul(x, rw);
          ok = full(x) | ok | bad;
        } while (!ok);
      }
    };
    settle(xa, 2, 2);
    settle(xb, 3, 1);
    STAMP(56 + w);
    part(nu - 1);
    STAMP(60 + w);
    while (rfl(__hip_atomic_load(lcnt, __ATOMIC_RELAXED, __HIP_MEMORY_SCOPE_WORKGROUP)) < nu * 4) {
      if (__builtin_amdgcn_s_memrealtime() - t0 > 2 * DEADLINE) { bad = true; break; }
    }
    asm volatile("" ::: "memory");
    const char* const pq = pl + w * 1024;
    f32x4 y = (*(const f32x4*)pq + *(const f32x4*)(pq + 4096)) + (*(const f32x4*)(pq + 8192) + *(const f32x4*)(pq + 12288));
    y.x = gelu(y.x); y.y = gelu(y.y); y.z = gelu(y.z); y.w = gelu(y.w);
    if (bad) y.x = __builtin_nanf("");
    asm volatile("global_store_dwordx4 %0, %1, %2 sc0 sc1\n\ts_nop 1"
                 :: "v"(ovoff + (unit_at(nu - 1) * 32 + w * 8) * (COLS * 4)), "v"(y), "s"(obase) : "memory");
    if (w == 0) STAMP(32 + nu - 1);
    if (foreign) {
      asm volatile("s_waitcnt vmcnt(0)" ::: "memory");
      if (lane == 0) __hip_atomic_store(donep + w, call, __ATOMIC_RELAXED, __HIP_MEMORY_SCOPE_SYSTEM);
    }
    if (w == 0) STAMP(2);
  }
}

unsigned link_map(int rank) {
  const unsigned dflt = 0x76543210u;
  int dev = -1, n = 0, node[NR], port[NR + 1];
  if (hipGetDevice(&dev) != hipSuccess || hipGetDeviceCount(&n) != hipSuccess || dev != rank || n != NR) return dflt;
  for (int i = 0; i <= NR; ++i) port[i] = -1;
  for (int d = 0; d < NR; ++d) {
    char id[32], path[96];
    if (hipDeviceGetPCIBusId(id, sizeof id, d) != hipSuccess) return dflt;
    for (char* c = id; *c; ++c) if (*c >= 'A' && *c <= 'F') *c += 'a' - 'A';
    snprintf(path, sizeof path, "/sys/bus/pci/devices/%s/xgmi_port_num", id);
    FILE* f = fopen(path, "r");
    if (!f) return dflt;
    unsigned a, pa, z, pz;
    node[d] = -1;
    while (fscanf(f, " %x:%x -> %x:%x", &a, &pa, &z, &pz) == 4) {
      node[d] = (int)a;
      if (d == rank && z <= (unsigned)NR && pa < 8u) port[z] = (int)pa;
    }
    fclose(f);
    if (node[d] < 0 || node[d] > NR) return dflt;
  }
  unsigned map = 0, used = 0;
  const int side = rank >> 2, q = rank & 3;
  for (int c = 0; c < 7; ++c) {
    const int x = port[node[c < 3 ? side * 4 + ((q + 1 + c) & 3) : (side ^ 1) * 4 + ((q + c - 3) & 3)]];
    if (x < 0 || ((used >> x) & 1)) return dflt;
    used |= 1u << x;
    map |= (unsigned)c << (4 * x);
  }
  return map | 7u << (4 * __builtin_ctz(~used));
}

}

extern "C" void kernel_launch(int rank, int nranks, ncclComm_t comm,
                              hipStream_t stream,
                              void* const* d_in, const int* in_sizes, int n_in,
                              void* d_out, int out_size,
                              const int* in_dtypes, int out_dtype,
                              void* const* d_out_all,
                              void* const* d_ws_all, size_t ws_bytes) {
  (void)comm; (void)in_dtypes; (void)out_dtype; (void)d_out;
  if (nranks != NR || (unsigned)rank >= (unsigned)NR || n_in < 2 || ws_bytes < WS_NEED ||
      in_sizes[0] != ROWS * KDIM || in_sizes[1] != KDIM * COLS || out_size != NR * ROWS * COLS) {
    fprintf(stderr, "kernel_launch: unexpected problem shape\n");
    return;
  }
  Params P;
  const int side = rank >> 2, q = rank & 3;
  for (int p = 0; p < NR; ++p) { P.ws[p] = (char*)d_ws_all[p]; P.out[p] = (char*)d_out_all[p]; }
  unsigned xmap = link_map(rank);
  for (int x = 0; x < 8; ++x) {
    const int c = (xmap >> (4 * x)) & 7;
    const int dest = c < 3 ? side * 4 + ((q + 1 + c) & 3) : (c < 7 ? (side ^ 1) * 4 + ((q + c - 3) & 3) : rank);
    P.head[x] = Head{(const char*)d_in[0], (const char*)d_in[1], P.ws[rank], P.ws[dest], rank, c};
  }
  static const unsigned char order[NG][12] = {
    {60, 72, 0, 6, 40, 12, 45, 50, 18, 55, 24, 30},
    {61, 73, 1, 7, 43, 13, 44, 49, 19, 54, 25, 31},
    {62, 2, 8, 42, 14, 47, 48, 20, 53, 26, 32, 99},
    {63, 3, 9, 41, 46, 15, 51, 21, 52, 27, 33, 99},
    {64, 65, 66, 67, 74, 4, 10, 16, 22, 28, 34, 99},
    {68, 69, 70, 71, 75, 5, 11, 17, 23, 29, 35, 99}};
  for (int g = 0; g < NG; ++g)
    for (int cls = 0; cls < 5; ++cls) {
      unsigned u[12] = {0}, n = 0;
      for (int k = 0; k < 12; ++k) {
        int v = order[g][k];
        if (v == 99) break;
        if (cls > 0 && (v >= 60 + NA || (v >= 40 && v < 60 && ((v - 40) & 3) == cls - 1))) continue;
        if (v < 40) u[n++] = (side * 4 + ((q + 1 + v % 3) & 3)) * 16 + v / 3;
        else if (v < 60) u[n++] = ((side ^ 1) * 4 + ((v - 40) & 3)) * 16 + NA + ((v - 40) >> 2);
        else u[n++] = rank * 16 + v - 60;
      }
      for (int k = 0; k < 3; ++k)
        P.units[g][cls][k] = u[4 * k] | (u[4 * k + 1] << 8) | (u[4 * k + 2] << 16) | (u[4 * k + 3] << 24);
      P.units[g][cls][3] = n;
    }
  hipLaunchKernelGGL(ag_gemm_gelu, dim3(256), dim3(384), 0, stream, P);
  HIP_CHECK(hipGetLastError());
}
```

```cpp
#include <hip/hip_runtime.h>
#include <rccl/rccl.h>
#include <cstdio>

#define HIP_CHECK(expr) do { hipError_t _e = (expr); if (_e != hipSuccess) { \
    fprintf(stderr, "HIP error %s at %s:%d\n", hipGetErrorString(_e), __FILE__, __LINE__); \
    return; } } while (0)

namespace {

typedef unsigned u32x4 __attribute__((ext_vector_type(4)));
typedef float f32x4 __attribute__((ext_vector_type(4)));
typedef float f32x16 __attribute__((ext_vector_type(16)));
typedef _Float16 h2 __attribute__((ext_vector_type(2)));
typedef _Float16 h8 __attribute__((ext_vector_type(8)));
typedef short s4 __attribute__((ext_vector_type(4)));
typedef short s8 __attribute__((ext_vector_type(8)));
typedef unsigned long long u64;

constexpr int NR = 8;
constexpr int KDIM = 4096;
constexpr int ROWS = 512;
constexpr int COLS = 512;
constexpr unsigned OBJ = 256u * 1024u;
constexpr int NG = 6;
constexpr int NA = 12;
constexpr size_t WS_CNT = 0;
constexpr size_t WS_TMO = 16 * 1024;
constexpr size_t WS_TICK = 20 * 1024;
constexpr size_t WS_FLAG = 32 * 1024;
constexpr size_t WS_DONE = 64 * 1024;
constexpr size_t WS_STGF = 80 * 1024;
constexpr size_t WS_DIAG = 128 * 1024;
constexpr size_t WS_DATA = 1024 * 1024;
constexpr unsigned NSLOT = 160;
constexpr size_t PAR_BYTES = (size_t)NSLOT * OBJ;
constexpr size_t WS_STG = WS_DATA + 2 * PAR_BYTES;
constexpr size_t WS_NEED = WS_STG + 128 * 12 * 4096;
constexpr u64 DEADLINE = 100000000ull;
#define FWD_T0 2500
#define SMODE 1
__host__ __device__ constexpr int lent_strip(int o, int h, int i) {
  return SMODE == 0 ? h * 4 + i : (SMODE == 1 ? ((o + h) & 3) * 4 + i : i * 4 + ((o + h) & 3));
}
#define FW 5
#define KA 4
#define KC 4
#define RP 2
#define INB 1
#define SSL 2
#define SOFF 8188
#define KL 4
#define KEEP2 2
#define AGGN 8
#define AGGC 7
#define TOUCH 3
#define LAG 30
#define KEEP 4
constexpr unsigned L_PART = 128 * 1024;
constexpr unsigned L_CTRL = L_PART + 16 * 1024;
constexpr unsigned L_TOTAL = L_CTRL + 64;

static_assert(AGGN < NA, "unit NA-1 has halves with fewer than 8 tickets");

struct alignas(64) Head {
  const char* x;
  const char* w;
  char* myws;
  char* dws;
  int rank;
  int ch;
};
struct Params {
  Head head[8];
  char* ws[NR];
  char* out[NR];
  unsigned units[NG][5][4];
};

__device__ __forceinline__ unsigned rfl(unsigned v) { return __builtin_amdgcn_readfirstlane(v); }
__device__ __forceinline__ unsigned pk(float a, float b) {
  h2 v; v.x = (_Float16)a; v.y = (_Float16)b; return __builtin_bit_cast(unsigned, v);
}
__device__ __forceinline__ u32x4 cvt8(u32x4 a, u32x4 b) {
  f32x4 fa = __builtin_bit_cast(f32x4, a), fb = __builtin_bit_cast(f32x4, b);
  u32x4 r; r.x = pk(fa.x, fa.y); r.y = pk(fa.z, fa.w); r.z = pk(fb.x, fb.y); r.w = pk(fb.z, fb.w);
  return r;
}
__device__ __forceinline__ float gelu(float y) {
  const float c2 = -2.0f * 0.7978845608028654f * 1.4426950408889634f;
  float u = y + 0.044715f * y * y * y;
  float e = __builtin_amdgcn_exp2f(c2 * u);
  return y * __builtin_amdgcn_rcpf(1.0f + e);
}

#define STAMPV(idx, val) do {} while (0)
#define STAMP(idx) STAMPV(idx, __builtin_amdgcn_s_memrealtime())

__device__ __forceinline__ bool give_up(u64 t0, unsigned* tmo, unsigned call) {
  u64 el = __builtin_amdgcn_s_memrealtime() - t0;
  if (el < 100000ull) return false;
  if (el > DEADLINE ||
      rfl(__hip_atomic_load(tmo, __ATOMIC_RELAXED, __HIP_MEMORY_SCOPE_AGENT)) == call) {
    __hip_atomic_store(tmo, call, __ATOMIC_RELAXED, __HIP_MEMORY_SCOPE_AGENT);
    return true;
  }
  return false;
}

__device__ __forceinline__ unsigned flag_word(unsigned pc) {
  return ((pc >> 4) & 1) * 16 + ((pc >> 1) & 3) * 4 + ((pc >> 3) & 1) * 2 + (pc & 1);
}
__device__ __forceinline__ unsigned flag_load(const unsigned* p) {
  return __hip_atomic_load(p, __ATOMIC_RELAXED, __HIP_MEMORY_SCOPE_SYSTEM);
}
__device__ __forceinline__ unsigned flag_mask(unsigned v, unsigned call) {
  return (unsigned)__builtin_amdgcn_ballot_w64(v >= call) & 0xFFu;
}
__device__ __forceinline__ void wait_round(const unsigned* p, int r, unsigned& seen, bool& bad, u64 t0,
                                           unsigned* tmo, unsigned call, unsigned* lidle, unsigned mark) {
  while (((seen >> (2 * r)) & 3u) != 3u) {
    seen = flag_mask(flag_load(p), call);
    if (((seen >> (2 * r)) & 3u) == 3u) break;
    if (r < 3) __hip_atomic_store(lidle, mark, __ATOMIC_RELAXED, __HIP_MEMORY_SCOPE_WORKGROUP);
    if (give_up(t0, tmo, call)) { bad = true; break; }
    __builtin_amdgcn_s_sleep(2);
  }
  asm volatile("" ::: "memory");
}

struct Piece { u32x4 r[16]; };

__device__ __forceinline__ void piece_load(Piece& s, const char* sb, unsigned fstride, unsigned voff) {
#pragma unroll
  for (int u = 0; u < 8; ++u) {
    const char* p = sb + (size_t)u * fstride;
    asm volatile("global_load_dwordx4 %0, %1, %2" : "=v"(s.r[2 * u]) : "v"(voff), "s"(p) : "memory");
    asm volatile("global_load_dwordx4 %0, %1, %2 offset:16" : "=v"(s.r[2 * u + 1]) : "v"(voff), "s"(p) : "memory");
  }
}
template <int N>
__device__ __forceinline__ void piece_wait(Piece& s) {
  asm volatile("s_waitcnt vmcnt(%0)" :: "i"(N) : "memory");
#pragma unroll
  for (int u = 0; u < 16; ++u) asm volatile("" : "+v"(s.r[u]));
}
template <int U>
__device__ __forceinline__ void piece_store1(Piece& s, char* db, unsigned voff) {
  u32x4 v = cvt8(s.r[2 * U], s.r[2 * U + 1]);
  asm volatile("global_store_dwordx4 %0, %1, %2 offset:%3 sc0 sc1\n\ts_nop 1"
               :: "v"(voff), "v"(v), "s"(db), "i"((U - 4) * 1024) : "memory");
}
__device__ __forceinline__ void piece_store(Piece& s, char* db  , unsigned voff) {
  piece_store1<0>(s, db, voff); piece_store1<1>(s, db, voff); piece_store1<2>(s, db, voff);
  piece_store1<3>(s, db, voff); piece_store1<4>(s, db, voff); piece_store1<5>(s, db, voff);
  piece_store1<6>(s, db, voff); piece_store1<7>(s, db, voff);
}

typedef __attribute__((address_space(3))) char lds_char;

__device__ __forceinline__ void glds16(const char* src, unsigned voff, unsigned lds) {
  unsigned keep;
  asm volatile("s_mov_b32 %0, m0\n\ts_mov_b32 m0, %3\n\ts_nop 0\n\t"
               "global_load_lds_dwordx4 %1, %2 sc0 sc1\n\ts_mov_b32 m0, %0"
               : "=&s"(keep) : "v"(voff), "s"(src), "s"(lds) : "memory");
}

__device__ __forceinline__ h8 tr_frag(lds_char* p) {
  s4 a = __builtin_amdgcn_ds_read_tr16_b64_v4i16((__attribute__((address_space(3))) s4*)p);
  s4 c = __builtin_amdgcn_ds_read_tr16_b64_v4i16((__attribute__((address_space(3))) s4*)(p + 256));
  s8 r = __builtin_shufflevector(a, c, 0, 1, 2, 3, 4, 5, 6, 7);
  return __builtin_bit_cast(h8, r);
}

__global__ void __launch_bounds__(384) ag_gemm_gelu(Params P) {
  __shared__ __attribute__((aligned(1024))) char smem[L_TOTAL];
  const int lane = threadIdx.x & 63;
  const int wave = rfl(threadIdx.x >> 6);
  Head H = P.head[__builtin_amdgcn_s_getreg(20 | (3 << 11)) & 7];
  asm volatile("" : "+s"(H.x), "+s"(H.w), "+s"(H.myws), "+s"(H.dws), "+s"(H.rank), "+s"(H.ch));
  const int xch = H.ch & 7;
  const int b = blockIdx.x < 64 ? xch * 8 + (blockIdx.x >> 3) : blockIdx.x;
  const int rank = H.rank & (NR - 1);
  const int side = rank >> 2, q = rank & 3;
  char* const myws = (char*)(__attribute__((address_space(1))) char*)(u64)H.myws;
  unsigned* const cntp = (unsigned*)(myws + WS_CNT + b * 32 + wave * 4);
  unsigned* const lcnt = (unsigned*)(smem + L_CTRL);
  unsigned* const lidle = (unsigned*)(smem + L_CTRL + 4);
  unsigned* const lfree = (unsigned*)(smem + L_CTRL + 8);

  if (__builtin_expect(wave == 0, 1)) {
    if (__builtin_expect(b >= 64 + 128 || (b >> 3) == 7, 0)) {
      const int c = xch;
      if ((TOUCH & 1) && (b >> 3) == 24 && c < 7) {
        const int dest = (c < 3 ? side * 4 + ((q + 1 + c) & 3) : (side ^ 1) * 4 + ((q + c - 3) & 3)) & (NR - 1);
        const char* const dd = H.dws + WS_DATA + (lane & 1) * PAR_BYTES;
        const char* a1 = lane < 2 ? dd + (size_t)(c < 3 ? rank * 16 : 144 + q * 4) * OBJ : H.dws + WS_FLAG;
        const char* a2 = lane < 32 ? dd + (size_t)(rank * 16 + (lane >> 1)) * OBJ : P.out[dest] + (size_t)(lane - 32) * OBJ;
        unsigned v1, v2;
        asm volatile("global_load_dword %0, %2, off sc0 sc1\n\tglobal_load_dword %1, %3, off sc0 sc1\n\ts_waitcnt vmcnt(0)"
                     : "=&v"(v1), "=&v"(v2) : "v"(a1), "v"(a2) : "memory");
      }
      return;
    }
    __builtin_amdgcn_s_setprio(3);
    __builtin_amdgcn_s_waitcnt(0);
    const int j = b & 7, c = b >> 3;
    int dest, nW, strip0, wslot0, xu0, nobj;
    if (c < 3)      { dest = side * 4 + ((q + 1 + c) & 3); nW = 0; strip0 = 0; wslot0 = 0; xu0 = 0; nobj = NA; }
    else if (c < 7) { dest = (side ^ 1) * 4 + ((q + c - 3) & 3); nW = 4; strip0 = lent_strip(q, dest & 3, 0); wslot0 = 16 + q * 4;
                      xu0 = NA; nobj = 4 + 16 - NA; }
    else            { dest = rank; nW = 1; strip0 = c - 8; wslot0 = c - 8; xu0 = c - 8; nobj = 2; }
    const unsigned np = nobj * 32;
    const unsigned nt = np / 2 - (c < 3 ? 2 * RP : 0);
    const unsigned laneX = (lane & 31) * (KDIM * 4) + (lane >> 5) * 32;
    const unsigned laneW = (lane >> 2) * (COLS * 4) + (lane & 3) * 32;
    const unsigned lane16 = lane * 16;
    const char* const px = H.x;
    const char* const pw = H.w;
    if (LAG && b >= 64) __builtin_amdgcn_s_sleep(LAG);
    auto load = [&](Piece& s, unsigned pc) {
      pc = pc < np ? pc : np - 1;
      const int obj = pc >> 5, f0 = (pc & 31) * 8;
      const bool isW = obj < nW;
      const char* sb = isW ? pw + (size_t)(strip0 + obj * (lent_strip(0, 0, 1) - lent_strip(0, 0, 0))) * 128 + (size_t)f0 * (16 * COLS * 4)
                           : px + (size_t)(xu0 + obj - nW) * (32 * KDIM * 4) + (size_t)f0 * 64;
      piece_load(s, sb, isW ? 16 * COLS * 4 : 64, isW ? laneW : laneX);
    };
    unsigned cnt;
    asm volatile("global_load_dword %0, %1, off sc1" : "=v"(cnt) : "v"(cntp) : "memory");
    Piece A, B;
    load(A, j);
    load(B, j + 8);
    piece_wait<16>(A);
    asm volatile("" : "+v"(cnt));
    const unsigned call = rfl(cnt) + 1;
    char* const dws = c < 8 ? H.dws : myws;
    char* const ddata = dws + WS_DATA + (size_t)(call & 1) * PAR_BYTES + 4096;
    char* const tick = myws + WS_TICK + c * 128;
    const unsigned tbase = 16 - (call - 1) * nt;
    auto slot_of = [&](unsigned pc) -> unsigned {
      const int obj = pc >> 5;
      return obj < nW ? 128 + wslot0 + obj : rank * 16 + xu0 + obj - nW;
    };
    auto store = [&](Piece& s, unsigned pc) {
      piece_store(s, ddata + (size_t)slot_of(pc) * OBJ + (pc & 31) * 8192, lane16);
    };
    const unsigned one = 1;
    auto flag = [&](unsigned pc, unsigned m) {
      char* fp = dws + WS_FLAG + slot_of(pc) * 128 + (pc & 16) * 4;
      if ((m >> (lane & 31)) & (lane < 16)) asm volatile("global_store_dword %0, %1, %2 sc0 sc1" :: "v"(lane * 4), "v"(call), "s"(fp) : "memory");
    };
    unsigned dn = 0, pend = np;
    auto count = [&](unsigned& r, unsigned off, unsigned lanes) {
      asm volatile("s_mov_b32 exec_lo, %4\n\ts_mov_b32 exec_hi, 0\n\tglobal_atomic_add %0, %1, %2, %3 sc0\n\ts_mov_b64 exec, -1"
                   : "+v"(r) : "v"(off), "v"(one), "s"(tick), "s"(lanes) : "memory");
    };
    auto done = [&](unsigned pc) {
      const bool agg = c < AGGC && (int)(pc >> 5) >= nW && pc < AGGN * 32;
      count(dn, 16 + (pc >> 4) * 4, rfl(agg));
      if (agg) pend = pc;
      else flag(pc, 5u << (flag_word(pc) & 15));
    };
    auto post = [&]() {
      asm volatile("" : "+v"(dn));
      if (pend < np && rfl(dn) + 1 == call * 8) flag(pend, 0xFFFFu);
      pend = np;
    };
    STAMP(0);
    auto piece_of = [&](unsigned t) -> unsigned {
      if (t >= NA * 16 - 24) t += t < NA * 16 - 16 - 2 * RP ? 8 + 2 * RP : (t < NA * 16 - 8 - 2 * RP ? 2 * RP - 8 : 2 * RP);
      return (t >> 3) * 16 + (t & 7);
    };
    unsigned tk = 0;
    unsigned cur = j, nxt = j + 16;
    store(A, cur);
    count(tk, 0, 1);
    load(A, nxt);
    piece_wait<25>(B);
    if (lane == 0) asm volatile("global_store_dword %0, %1, off sc1" :: "v"(cntp), "v"(call) : "memory");
    store(B, cur + 8);
    load(B, nxt + 8);
    __builtin_amdgcn_s_barrier();
    bool gate = INB && c < 3;
    do {
      piece_wait<25>(A);
      asm volatile("" : "+v"(tk));
      unsigned t = rfl(tk) + tbase;
      const unsigned nn = t < nt ? piece_of(t) : np;
      if (c < 3) asm volatile("s_waitcnt vmcnt(%0)" :: "i"(16 + KA) : "memory");
      else if (c < 7) asm volatile("s_waitcnt vmcnt(%0)" :: "i"(16 + KC) : "memory");
      post();
      if (gate && nxt >= 96) { __builtin_amdgcn_s_barrier(); gate = false; }
      store(A, nxt);
      count(tk, 0, 1);
      load(A, nn);
      piece_wait<25>(B);
      done(cur);
      if (c < 3) asm volatile("s_waitcnt vmcnt(%0)" :: "i"(18 + KA) : "memory");
      else if (c < 7) asm volatile("s_waitcnt vmcnt(%0)" :: "i"(18 + KC) : "memory");
      store(B, nxt + 8);
      load(B, nn + 8);
      cur = nxt;
      nxt = nn;
    } while (nxt < np);
    if (b < 64) __hip_atomic_store(lfree, 1u, __ATOMIC_RELAXED, __HIP_MEMORY_SCOPE_WORKGROUP);
    if (gate) __builtin_amdgcn_s_barrier();
    piece_wait<24>(A);
    asm volatile("" : "+v"(tk));
    post();
    piece_wait<0>(B);
    done(cur);
    asm volatile("s_waitcnt vmcnt(0)" ::: "memory");
    post();
    STAMP(1);
    return;
  }

  const bool fwd = b < 64;
  if (fwd && (b < 24 || b >= 56)) {
    if (INB && b < 24 && wave <= 2) {
      __builtin_amdgcn_s_barrier();
      __builtin_amdgcn_s_sleep(100);
      const unsigned call = rfl(__hip_atomic_load(cntp, __ATOMIC_RELAXED, __HIP_MEMORY_SCOPE_AGENT)) + 1;
      if (lane == 0) __hip_atomic_store(cntp, call, __ATOMIC_RELAXED, __HIP_MEMORY_SCOPE_AGENT);
      const unsigned slot = (side * 4 + ((q + 1 + (b >> 3)) & 3)) * 16 + NA - wave;
      char* const zp = myws + WS_DATA + (size_t)(~call & 1) * PAR_BYTES + (size_t)slot * OBJ + OBJ / 2 + (b & 7) * 16384;
      const u32x4 z = {0u, 0u, 0u, 0u};
#pragma unroll
      for (int i = 0; i < 16; ++i)
        asm volatile("global_store_dwordx4 %0, %1, %2 sc0 sc1" :: "v"(lane * 16 + i * 1024), "v"(z), "s"(zp) : "memory");
      asm volatile("s_waitcnt vmcnt(0)" ::: "memory");
    }
    return;
  }
  const u64 t0 = __builtin_amdgcn_s_memrealtime();
  if (LAG && !fwd) __builtin_amdgcn_s_sleep(LAG);
  const unsigned call = rfl(__hip_atomic_load(cntp, __ATOMIC_RELAXED, __HIP_MEMORY_SCOPE_AGENT)) + 1;
  if (lane == 0) __hip_atomic_store(cntp, call, __ATOMIC_RELAXED, __HIP_MEMORY_SCOPE_AGENT);
  unsigned* const tmo = (unsigned*)(myws + WS_TMO);
  const int rr = lane >> 3, chn = lane & 7;
  const unsigned ovoff = rr * (COLS * 4) + chn * 16;
  bool bad = false;
  if (wave == 1) {
    STAMPV(5, t0);
    if (lane < 8) lcnt[lane] = 0;
    asm volatile("s_waitcnt lgkmcnt(0)" ::: "memory");
  }
  __builtin_amdgcn_s_barrier();

  if (fwd) {
    const int j = b & 7, p = (q + (b >> 3) - 3) & 3;
    const int owner = ((side ^ 1) * 4 + p) & (NR - 1);
    char* const obase = P.out[owner];
    unsigned* const lnext = (unsigned*)(smem + L_CTRL + 16);
    const unsigned tl = lane < 3 ? lane : 0;
    const unsigned* const stgf = (const unsigned*)(myws + WS_STGF + ((p * 4 + ((j + tl) & 3)) * 8 + 2 * tl + (j >> 2)) * 128);
    const unsigned n0 = P.units[j >> 2][1 + p][3], n1 = P.units[2 + (j >> 2)][1 + p][3], n2 = P.units[4 + (j >> 2)][1 + p][3];
    const unsigned nl = (tl == 0 ? n0 : (tl == 1 ? n1 : n2)) - 1;
    while (__builtin_amdgcn_s_memrealtime() - t0 < FWD_T0) __builtin_amdgcn_s_sleep(16);
    if (RP && wave == 5) {
      const size_t dbase = WS_DATA + (size_t)(call & 1) * PAR_BYTES + 4096;
      auto rpiece = [](unsigned n) -> unsigned { return n % (2 * RP) + n / (2 * RP) * 8; };
      if (j >= 8 - RP) {
        const unsigned slot = rank * 16 + NA - 1, pc = rpiece(p * RP + 7 - j);
        Piece A;
        piece_load(A, H.x + (size_t)(NA - 1) * (32 * KDIM * 4) + pc * 512, 64, (lane & 31) * (KDIM * 4) + (lane >> 5) * 32);
        piece_wait<0>(A);
        while (rfl(__hip_atomic_load(lfree, __ATOMIC_RELAXED, __HIP_MEMORY_SCOPE_WORKGROUP)) == 0) {
          if (__builtin_amdgcn_s_memrealtime() - t0 > 2 * DEADLINE) break;
          __builtin_amdgcn_s_sleep(2);
        }
        piece_store(A, P.ws[owner] + dbase + (size_t)slot * OBJ + pc * 8192, lane * 16);
        asm volatile("s_waitcnt vmcnt(0)" ::: "memory");
        if (lane == 0) __hip_atomic_store((unsigned*)(P.ws[owner] + WS_FLAG + slot * 128 + flag_word(pc) * 4), call, __ATOMIC_RELAXED, __HIP_MEMORY_SCOPE_SYSTEM);
      }
      for (int n = j; n < 3 * RP; n += 8) {
        const unsigned slot = ((side ^ 1) * 4 + ((p + 1 + n / RP) & 3)) * 16 + NA - 1, pc = rpiece(q * RP + n % RP);
        const size_t fo = WS_FLAG + slot * 128 + flag_word(pc) * 4;
        const size_t go = dbase + (size_t)slot * OBJ + pc * 8192;
        while (rfl(flag_load((const unsigned*)(myws + fo))) < call) {
          if (give_up(t0, tmo, call)) { bad = true; break; }
          __builtin_amdgcn_s_sleep(8);
        }
        if (bad) break;
        u32x4 y0, y1, y2, y3, y4, y5, y6, y7;
        asm volatile("global_load_dwordx4 %0, %8, %9 offset:-4096 sc0 sc1\n\t"
                     "global_load_dwordx4 %1, %8, %9 offset:-3072 sc0 sc1\n\t"
                     "global_load_dwordx4 %2, %8, %9 offset:-2048 sc0 sc1\n\t"
                     "global_load_dwordx4 %3, %8, %9 offset:-1024 sc0 sc1\n\t"
                     "global_load_dwordx4 %4, %8, %9 sc0 sc1\n\t"
                     "global_load_dwordx4 %5, %8, %9 offset:1024 sc0 sc1\n\t"
                     "global_load_dwordx4 %6, %8, %9 offset:2048 sc0 sc1\n\t"
                     "global_load_dwordx4 %7, %8, %9 offset:3072 sc0 sc1\n\t"
                     "s_waitcnt vmcnt(0)\n\t"
                     "global_store_dwordx4 %8, %0, %10 offset:-4096 sc0 sc1\n\t"
                     "global_store_dwordx4 %8, %1, %10 offset:-3072 sc0 sc1\n\t"
                     "global_store_dwordx4 %8, %2, %10 offset:-2048 sc0 sc1\n\t"
                     "global_store_dwordx4 %8, %3, %10 offset:-1024 sc0 sc1\n\t"
                     "global_store_dwordx4 %8, %4, %10 sc0 sc1\n\t"
                     "global_store_dwordx4 %8, %5, %10 offset:1024 sc0 sc1\n\t"
                     "global_store_dwordx4 %8, %6, %10 offset:2048 sc0 sc1\n\t"
                     "global_store_dwordx4 %8, %7, %10 offset:3072 sc0 sc1\n\t"
                     "s_waitcnt vmcnt(0)"
                     : "=&v"(y0), "=&v"(y1), "=&v"(y2), "=&v"(y3), "=&v"(y4), "=&v"(y5), "=&v"(y6), "=&v"(y7)
                     : "v"(lane * 16), "s"(myws + go), "s"(P.ws[owner] + go) : "memory");
        if (lane == 0) __hip_atomic_store((unsigned*)(P.ws[owner] + fo), call, __ATOMIC_RELAXED, __HIP_MEMORY_SCOPE_SYSTEM);
        STAMP(46 + (n >> 3));
      }
    }
    unsigned fv = 0;
    bool open = false;
    while (wave <= FW) {
      const unsigned nx = __hip_atomic_load(lnext + tl, __ATOMIC_RELAXED, __HIP_MEMORY_SCOPE_WORKGROUP);
      const unsigned left = (unsigned)__builtin_amdgcn_ballot_w64(nx < nl) & 7u;
      if (left == 0) break;
      const unsigned rdy = rfl(bad ? left : (unsigned)__builtin_amdgcn_ballot_w64(fv > call * 16 + nx) & left);
      if (rdy == 0) {
        fv = __hip_atomic_load(stgf, __ATOMIC_RELAXED, __HIP_MEMORY_SCOPE_SYSTEM);
        if (give_up(t0, tmo, call)) bad = true;
        __builtin_amdgcn_s_sleep(2);
        continue;
      }
      const unsigned t = __builtin_ctz(rdy);
      const unsigned k = __builtin_amdgcn_readlane(nx, t);
      unsigned got = k;
      if (lane == 0) __hip_atomic_compare_exchange_strong(lnext + t, &got, k + 1, __ATOMIC_RELAXED, __ATOMIC_RELAXED, __HIP_MEMORY_SCOPE_WORKGROUP);
      if (rfl(got) != k) continue;
      const unsigned g = 2 * t + (j >> 2), i = (j + t) & 3;
      const unsigned unit = (P.units[g][1 + p][k >> 2] >> ((k & 3) * 8)) & 127u;
      const char* const src = myws + WS_STG + (size_t)(((p * 4 + i) * 8 + g) * 12 + k) * 4096;
      f32x4 y0, y1, y2, y3;
      asm volatile("global_load_dwordx4 %0, %5, %6 sc0 sc1\n\t"
                   "global_load_dwordx4 %1, %5, %6 offset:1024 sc0 sc1\n\t"
                   "global_load_dwordx4 %2, %5, %6 offset:2048 sc0 sc1\n\t"
                   "global_load_dwordx4 %3, %5, %6 offset:3072 sc0 sc1\n\t"
                   "global_load_dword %4, %7, off sc0 sc1\n\t"
                   "s_waitcnt vmcnt(0)"
                   : "=&v"(y0), "=&v"(y1), "=&v"(y2), "=&v"(y3), "=&v"(fv)
                   : "v"(lane * 16), "s"(src), "v"(stgf) : "memory");
      if (!open) {
        while (rfl(__hip_atomic_load(lfree, __ATOMIC_RELAXED, __HIP_MEMORY_SCOPE_WORKGROUP)) == 0) {
          if (__builtin_amdgcn_s_memrealtime() - t0 > 2 * DEADLINE) break;
          __builtin_amdgcn_s_sleep(2);
        }
        open = true;
        if (wave < 5) STAMP(6 + wave);
      }
      STAMP(16 + t * 10 + k);
      if (bad) y0.x = __builtin_nanf("");
      const unsigned vo = ovoff + unit * (32 * COLS * 4) + lent_strip(p, q, i) * 128, vs = 8 * COLS * 4;
      asm volatile("global_store_dwordx4 %0, %4, %8 sc0 sc1\n\t"
                   "global_store_dwordx4 %1, %5, %8 sc0 sc1\n\t"
                   "global_store_dwordx4 %2, %6, %8 sc0 sc1\n\t"
                   "global_store_dwordx4 %3, %7, %8 sc0 sc1\n\ts_nop 1"
                   :: "v"(vo), "v"(vo + vs), "v"(vo + 2 * vs), "v"(vo + 3 * vs),
                      "v"(y0), "v"(y1), "v"(y2), "v"(y3), "s"(obase) : "memory");
    }
    asm volatile("s_waitcnt vmcnt(0)" ::: "memory");
    unsigned* const dp = (unsigned*)(P.ws[owner] + WS_DONE + q * 1024 + (j * 5 + wave - 1) * 4);
    if (lane == 0) __hip_atomic_store(dp, call, __ATOMIC_RELAXED, __HIP_MEMORY_SCOPE_SYSTEM);
    STAMP(10 + wave);
    return;
  }

  const int nq = (b & 3) * 24 + (b >> 3) - 8;
  const int g = ((b >> 2) & 1) * 3 + (nq >> 5), s = nq & 31;
  const bool foreign = s >= 16;
  const int owner = (foreign ? (side ^ 1) * 4 + ((s - 16) >> 2) : rank) & (NR - 1);
  const int cls = foreign ? 1 + (owner & 3) : 0;
  const unsigned ul0 = P.units[g][cls][0], ul1 = P.units[g][cls][1], ul2 = P.units[g][cls][2];
  const unsigned nu = P.units[g][cls][3] < 12u ? P.units[g][cls][3] : 12u;
  auto unit_at = [&](unsigned k) -> unsigned {
    unsigned wv = k < 4 ? ul0 : (k < 8 ? ul1 : ul2);
    return (wv >> ((k & 3) * 8)) & 127u;
  };

  const int col0 = (foreign ? lent_strip(owner & 3, q, s & 3) : s) * 32;
  char* const obase = P.out[owner] + col0 * 4;
  unsigned* const donep = (unsigned*)(P.ws[owner] + WS_DONE + q * 1024 + 160 + ((s & 3) * 6 + g) * 16);
  const char* const pl = smem + L_PART + rr * 128 + ((chn ^ rr) * 16);
  if (wave == 1) {
    const unsigned prod = (s & 15) * 8 + g;
    char* const stg = myws + WS_STG + (size_t)prod * (12 * 4096);
    unsigned* const stgf = (unsigned*)(myws + WS_STGF + prod * 128);
    if ((TOUCH & 2) && foreign) {
      const char* a1 = lane & 1 ? (const char*)donep : obase + (size_t)unit_at(nu - 1) * (32 * COLS * 4);
      unsigned v1;
      __builtin_amdgcn_s_sleep(100);
      asm volatile("global_load_dword %0, %1, off sc0 sc1\n\ts_waitcnt vmcnt(0)" : "=&v"(v1) : "v"(a1) : "memory");
    }
    f32x4 keep[KEEP][4];
    unsigned kd = 0, ks = 0;
    while (ks + 1 < nu) {
      const u64 st = __hip_atomic_load((u64*)lcnt, __ATOMIC_RELAXED, __HIP_MEMORY_SCOPE_WORKGROUP);
      const unsigned have = rfl((unsigned)st), idle = rfl((unsigned)(st >> 32));
      if (__builtin_amdgcn_s_memrealtime() - t0 > 2 * DEADLINE) bad = true;
      if (have >= (kd + 1) * 4 || bad) {
        asm volatile("" ::: "memory");
        f32x4 y[4];
#pragma unroll
        for (int it = 0; it < 4; ++it) {
          f32x4 p0 = *(const f32x4*)(pl + it * 1024);
          f32x4 p1 = *(const f32x4*)(pl + it * 1024 + 4096);
          f32x4 p2 = *(const f32x4*)(pl + it * 1024 + 8192);
          f32x4 p3 = *(const f32x4*)(pl + it * 1024 + 12288);
          y[it] = (p0 + p1) + (p2 + p3);
        }
#pragma unroll
        for (int it = 0; it < 4; ++it) asm volatile("" : "+v"(y[it]));
        asm volatile("s_waitcnt lgkmcnt(0)" ::: "memory");
        if (lane == 0) __hip_atomic_store(lfree, kd + 1, __ATOMIC_RELAXED, __HIP_MEMORY_SCOPE_WORKGROUP);
#pragma unroll
        for (int it = 0; it < 4; ++it) {
          y[it].x = gelu(y[it].x); y[it].y = gelu(y[it].y); y[it].z = gelu(y[it].z); y[it].w = gelu(y[it].w);
          if (bad) y[it].x = __builtin_nanf("");
        }
#pragma unroll
        for (unsigned k = 0; k < KEEP; ++k)
          if (kd - ks == k) {
#pragma unroll
            for (int it = 0; it < 4; ++it) { keep[k][it] = y[it]; asm volatile("" : "+v"(keep[k][it])); }
          }
        ++kd;
      } else if (ks == kd || idle != kd + 1) {
        __builtin_amdgcn_s_sleep(1);
        continue;
      }
      if (kd - ks >= (foreign && kd > KL ? KEEP2 : KEEP) || kd + 1 >= nu || idle == kd + 1) {
        const bool stage = foreign;
        char* const base = stage ? stg : obase;
        const unsigned step = stage ? 1024 : 8 * COLS * 4;
#pragma unroll
        for (unsigned k = 0; k < KEEP; ++k)
          if (ks + k < kd) {
            const unsigned vo = stage ? lane * 16 + (ks + k) * 4096 : ovoff + unit_at(ks + k) * (32 * COLS * 4);
#pragma unroll
            for (int it = 0; it < 4; ++it)
              asm volatile("global_store_dwordx4 %0, %1, %2 sc0 sc1\n\ts_nop 1"
                           :: "v"(vo + it * step), "v"(keep[k][it]), "s"(base) : "memory");
            STAMP(32 + ks + k);
          }
        ks = kd;
        if (stage) {
          asm volatile("s_waitcnt vmcnt(0)" ::: "memory");
          if (lane == 0) __hip_atomic_store(stgf, call * 16 + kd, __ATOMIC_RELAXED, __HIP_MEMORY_SCOPE_SYSTEM);
        }
      }
    }
    if (g == 0 && s < 4) {
      const char* dp = myws + WS_DONE + s * 1024 + (lane < 34 ? lane : 0) * 16;
      while (true) {
        u32x4 v;
        asm volatile("global_load_dwordx4 %0, %1, off sc0 sc1\n\ts_waitcnt vmcnt(0)" : "=&v"(v) : "v"(dp) : "memory");
        bool ok = v.x >= call && v.y >= call && v.z >= call && v.w >= call;
        if (__all(ok)) break;
        if (give_up(t0, tmo, call)) {
          char* const lastp = obase + (size_t)unit_at(0) * (32 * COLS * 4);
          if (lane == 0) __hip_atomic_store((float*)lastp, __builtin_nanf(""), __ATOMIC_RELAXED, __HIP_MEMORY_SCOPE_SYSTEM);
          break;
        }
      }
      STAMP(3);
    }
    return;
  }

  const int w = wave - 2;
  char* const data = myws + WS_DATA + (size_t)(call & 1) * PAR_BYTES;
  const unsigned* const flags = (const unsigned*)(myws + WS_FLAG) + (lane & 4) * 4 + w * 4 + (lane & 3);
  lds_char* const lw = (lds_char*)smem + w * 32768;
  lds_char* const ltr = lw + ((lane >> 5) * 8 + ((lane & 15) >> 2)) * 64 + ((lane >> 4) & 1) * 32 + (lane & 3) * 8;
  h8 wr[32];
  {
    const unsigned* pp = flags + (128 + s) * 32;
    const char* src = data + (size_t)(128 + s) * OBJ + w * 16384;
    const unsigned svoff = lane * 16;
    const unsigned lwa = (unsigned)(size_t)lw;
    unsigned seen = 0;
#pragma unroll
    for (int r = 0; r < 4; ++r) {
      wait_round(pp, r, seen, bad, t0, tmo, call, lidle, 0);
#pragma unroll
      for (int i = 0; i < 16; ++i)
        glds16(src + r * 65536 + i * 1024, svoff, lwa + (r & 1) * 16384 + i * 1024);
      if (r & 1) asm volatile("s_waitcnt vmcnt(0)" ::: "memory");
      if (r == 1) {
#pragma unroll
        for (int i = 0; i < 32; ++i) wr[i] = tr_frag(ltr + i * 1024);
#pragma unroll
        for (int i = 0; i < 32; ++i) asm volatile("" : "+v"(wr[i]));
        asm volatile("s_waitcnt lgkmcnt(0)" ::: "memory");
      }
    }
  }
  if (w == 0) STAMP(4);
  const __amdgpu_buffer_rsrc_t rsrc =
      __builtin_amdgcn_make_buffer_rsrc(data, 0, 128 * OBJ, 0x00020000);
  const unsigned xvoff = w * 16384 + lane * 16;
  char* const pw = smem + L_PART + w * 4096 + (lane & 31) * 128;
  unsigned nextf = 0, seen, soff;
  const unsigned* pp;
  f32x16 acc;
  auto xload = [&](u32x4 (&xa)[16], int r) {
#pragma unroll
    for (int i = 0; i < 16; ++i)
      xa[i] = __builtin_amdgcn_raw_buffer_load_b128(rsrc, xvoff + (i & 3) * 1024, soff + r * 65536 + (i >> 2) * 4096, 17);
    __builtin_amdgcn_sched_barrier(0);
  };
  auto xmul = [&](u32x4 (&xa)[16], int r) {
#pragma unroll
    for (int i = 0; i < 16; ++i) {
      h8 wf = r < 2 ? wr[r * 16 + i] : tr_frag(ltr + ((r - 2) * 16 + i) * 1024);
      acc = __builtin_amdgcn_mfma_f32_32x32x16_f16(wf, __builtin_bit_cast(h8, xa[i]), acc, 0, 0, 0);
    }
  };
  auto part = [&](unsigned k) {
    if (bad) {
#pragma unroll
      for (int i = 0; i < 16; ++i) acc[i] = __builtin_nanf("");
    }
    while (rfl(__hip_atomic_load(lfree, __ATOMIC_RELAXED, __HIP_MEMORY_SCOPE_WORKGROUP)) < k) {
      if (__builtin_amdgcn_s_memrealtime() - t0 > 2 * DEADLINE) break;
      __builtin_amdgcn_s_sleep(1);
    }
    asm volatile("" ::: "memory");
#pragma unroll
    for (int gq = 0; gq < 4; ++gq) {
      f32x4 v = {acc[gq * 4], acc[gq * 4 + 1], acc[gq * 4 + 2], acc[gq * 4 + 3]};
      *(f32x4*)(pw + (((gq * 2 + (lane >> 5)) ^ (lane & 7)) * 16)) = v;
    }
    asm volatile("s_waitcnt lgkmcnt(0)" ::: "memory");
    if (lane == 0) __hip_atomic_fetch_add(lcnt, 1u, __ATOMIC_RELAXED, __HIP_MEMORY_SCOPE_WORKGROUP);
    if (w == 0) STAMP(16 + k);
  };
  for (unsigned k = 0;; ++k) {
    const unsigned e = unit_at(k);
    pp = flags + e * 32;
    soff = e * OBJ;
    seen = flag_mask(nextf, call);
    acc = f32x16{0.f, 0.f, 0.f, 0.f, 0.f, 0.f, 0.f, 0.f, 0.f, 0.f, 0.f, 0.f, 0.f, 0.f, 0.f, 0.f};
    u32x4 xa[16];
#pragma unroll
    for (int r = 0; r < 2; ++r) {
      wait_round(pp, r, seen, bad, t0, tmo, call, lidle, k + 1);
      xload(xa, r);
      xmul(xa, r);
    }
    if (k + 1 >= nu) break;
#pragma unroll
    for (int r = 2; r < 4; ++r) {
      wait_round(pp, r, seen, bad, t0, tmo, call, lidle, k + 1);
      xload(xa, r);
      if (r == 3) { nextf = flag_load(flags + unit_at(k + 1) * 32); __builtin_amdgcn_sched_barrier(0); }
      xmul(xa, r);
    }
    part(k);
  }
  {
    u32x4 xa[16], xb[16];
#pragma unroll
    for (int i = 0; i < 16; ++i) wr[i] = tr_frag(ltr + i * 1024);
#pragma unroll
    for (int i = 0; i < 16; ++i) asm volatile("" : "+v"(wr[i]));
    const bool inb = INB;
    auto full = [&](u32x4 (&x)[16]) -> bool {
      unsigned m = ~0u;
#pragma unroll
      for (int i = 0; i < 16; ++i) m = min(min(min(min(m, x[i].x), x[i].y), x[i].z), x[i].w);
      return __builtin_amdgcn_ballot_w64(m == 0) == 0;
    };
    while (((seen >> 4) & 3u) != 3u) {
      const unsigned smp = __builtin_amdgcn_raw_buffer_load_b32(rsrc, w * 16384 + (lane & 1) * 8192 + SOFF, soff + 2 * 65536, 17);
      seen = flag_mask(flag_load(pp), call);
      if (inb && __builtin_amdgcn_ballot_w64(smp == 0) == 0) break;
      __hip_atomic_store(lidle, nu, __ATOMIC_RELAXED, __HIP_MEMORY_SCOPE_WORKGROUP);
      if (give_up(t0, tmo, call)) { bad = true; break; }
      if (SSL) __builtin_amdgcn_s_sleep(SSL);
    }
    asm volatile("" ::: "memory");
    STAMP(52 + w);
    xload(xa, 2);
    if (!inb) wait_round(pp, 3, seen, bad, t0, tmo, call, lidle, nu);
    xload(xb, 3);
    unsigned f = flag_load(pp);
    auto settle = [&](u32x4 (&x)[16], int r, int rw) {
      const f32x16 c0 = acc;
#pragma unroll
      for (int i = 0; i < 16; ++i) {
        acc = __builtin_amdgcn_mfma_f32_32x32x16_f16(wr[(r - 2) * 16 + i], __builtin_bit_cast(h8, x[i]), acc, 0, 0, 0);
        if (r == 2) wr[16 + i] = tr_frag(ltr + (16 + i) * 1024);
      }
      if (r == 2) {
#pragma unroll
        for (int i = 0; i < 16; ++i) asm volatile("" : "+v"(wr[16 + i]));
      }
      if (__builtin_expect(!(full(x) | (((seen >> (2 * r)) & 3u) == 3u)), 0)) {
        bool ok;
        do {
          ok = ((flag_mask(f, call) >> (2 * r)) & 3u) == 3u;
          if (give_up(t0, tmo, call)) bad = true;
          asm volatile("" ::: "memory");
          xload(x, r);
          f = flag_load(pp);
          acc = c0;
          xmul(x, rw);
          ok = full(x) | ok | bad;
        } while (!ok);
      }
    };
    settle(xa, 2, 2);
    settle(xb, 3, 1);
    STAMP(56 + w);
    part(nu - 1);
    STAMP(60 + w);
    while (rfl(__hip_atomic_load(lcnt, __ATOMIC_RELAXED, __HIP_MEMORY_SCOPE_WORKGROUP)) < nu * 4) {
      if (__builtin_amdgcn_s_memrealtime() - t0 > 2 * DEADLINE) { bad = true; break; }
    }
    asm volatile("" ::: "memory");
    const char* const pq = pl + w * 1024;
    f32x4 y = (*(const f32x4*)pq + *(const f32x4*)(pq + 4096)) + (*(const f32x4*)(pq + 8192) + *(const f32x4*)(pq + 12288));
    y.x = gelu(y.x); y.y = gelu(y.y); y.z = gelu(y.z); y.w = gelu(y.w);
    if (bad) y.x = __builtin_nanf("");
    asm volatile("global_store_dwordx4 %0, %1, %2 sc0 sc1\n\ts_nop 1"
                 :: "v"(ovoff + (unit_at(nu - 1) * 32 + w * 8) * (COLS * 4)), "v"(y), "s"(obase) : "memory");
    if (w == 0) STAMP(32 + nu - 1);
    if (foreign) {
      asm volatile("s_waitcnt vmcnt(0)" ::: "memory");
      if (lane == 0) __hip_atomic_store(donep + w, call, __ATOMIC_RELAXED, __HIP_MEMORY_SCOPE_SYSTEM);
    }
    if (w == 0) STAMP(2);
  }
}

unsigned link_map(int rank) {
  const unsigned dflt = 0x76543210u;
  int dev = -1, n = 0, node[NR], port[NR + 1];
  if (hipGetDevice(&dev) != hipSuccess || hipGetDeviceCount(&n) != hipSuccess || dev != rank || n != NR) return dflt;
  for (int i = 0; i <= NR; ++i) port[i] = -1;
  for (int d = 0; d < NR; ++d) {
    char id[32], path[96];
    if (hipDeviceGetPCIBusId(id, sizeof id, d) != hipSuccess) return dflt;
    for (char* c = id; *c; ++c) if (*c >= 'A' && *c <= 'F') *c += 'a' - 'A';
    snprintf(path, sizeof path, "/sys/bus/pci/devices/%s/xgmi_port_num", id);
    FILE* f = fopen(path, "r");
    if (!f) return dflt;
    unsigned a, pa, z, pz;
    node[d] = -1;
    while (fscanf(f, " %x:%x -> %x:%x", &a, &pa, &z, &pz) == 4) {
      node[d] = (int)a;
      if (d == rank && z <= (unsigned)NR && pa < 8u) port[z] = (int)pa;
    }
    fclose(f);
    if (node[d] < 0 || node[d] > NR) return dflt;
  }
  unsigned map = 0, used = 0;
  const int side = rank >> 2, q = rank & 3;
  for (int c = 0; c < 7; ++c) {
    const int x = port[node[c < 3 ? side * 4 + ((q + 1 + c) & 3) : (side ^ 1) * 4 + ((q + c - 3) & 3)]];
    if (x < 0 || ((used >> x) & 1)) return dflt;
    used |= 1u << x;
    map |= (unsigned)c << (4 * x);
  }
  return map | 7u << (4 * __builtin_ctz(~used));
}

}

extern "C" void kernel_launch(int rank, int nranks, ncclComm_t comm,
                              hipStream_t stream,
                              void* const* d_in, const int* in_sizes, int n_in,
                              void* d_out, int out_size,
                              const int* in_dtypes, int out_dtype,
                              void* const* d_out_all,
                              void* const* d_ws_all, size_t ws_bytes) {
  (void)comm; (void)in_dtypes; (void)out_dtype; (void)d_out;
  if (nranks != NR || (unsigned)rank >= (unsigned)NR || n_in < 2 || ws_bytes < WS_NEED ||
      in_sizes[0] != ROWS * KDIM || in_sizes[1] != KDIM * COLS || out_size != NR * ROWS * COLS) {
    fprintf(stderr, "kernel_launch: unexpected problem shape\n");
    return;
  }
  Params P;
  const int side = rank >> 2, q = rank & 3;
  for (int p = 0; p < NR; ++p) { P.ws[p] = (char*)d_ws_all[p]; P.out[p] = (char*)d_out_all[p]; }
  unsigned xmap = link_map(rank);
  for (int x = 0; x < 8; ++x) {
    const int c = (xmap >> (4 * x)) & 7;
    const int dest = c < 3 ? side * 4 + ((q + 1 + c) & 3) : (c < 7 ? (side ^ 1) * 4 + ((q + c - 3) & 3) : rank);
    P.head[x] = Head{(const char*)d_in[0], (const char*)d_in[1], P.ws[rank], P.ws[dest], rank, c};
  }
  static const unsigned char order[NG][12] = {
    {60, 72, 0, 6, 40, 12, 45, 50, 18, 55, 24, 30},
    {61, 73, 1, 7, 43, 13, 44, 49, 19, 54, 25, 31},
    {62, 2, 8, 42, 14, 47, 48, 20, 53, 26, 32, 99},
    {63, 3, 9, 41, 46, 15, 51, 21, 52, 27, 33, 99},
    {64, 65, 66, 67, 74, 4, 10, 16, 22, 28, 34, 99},
    {68, 69, 70, 71, 75, 5, 11, 17, 23, 29, 35, 99}};
  for (int g = 0; g < NG; ++g)
    for (int cls = 0; cls < 5; ++cls) {
      unsigned u[12] = {0}, n = 0;
      for (int k = 0; k < 12; ++k) {
        int v = order[g][k];
        if (v == 99) break;
        if (cls > 0 && (v >= 60 + NA || (v >= 40 && v < 60 && ((v - 40) & 3) == cls - 1))) continue;
        if (v < 40) u[n++] = (side * 4 + ((q + 1 + v % 3) & 3)) * 16 + v / 3;
        else if (v < 60) u[n++] = ((side ^ 1) * 4 + ((v - 40) & 3)) * 16 + NA + ((v - 40) >> 2);
        else u[n++] = rank * 16 + v - 60;
      }
      for (int k = 0; k < 3; ++k)
        P.units[g][cls][k] = u[4 * k] | (u[4 * k + 1] << 8) | (u[4 * k + 2] << 16) | (u[4 * k + 3] << 24);
      P.units[g][cls][3] = n;
    }
  hipLaunchKernelGGL(ag_gemm_gelu, dim3(256), dim3(384), 0, stream, P);
  HIP_CHECK(hipGetLastError());
}
```

```cpp
#include <hip/hip_runtime.h>
#include <rccl/rccl.h>
#include <cstdio>

#define HIP_CHECK(expr) do { hipError_t _e = (expr); if (_e != hipSuccess) { \
    fprintf(stderr, "HIP error %s at %s:%d\n", hipGetErrorString(_e), __FILE__, __LINE__); \
    return; } } while (0)

namespace {

typedef unsigned u32x4 __attribute__((ext_vector_type(4)));
typedef float f32x4 __attribute__((ext_vector_type(4)));
typedef float f32x16 __attribute__((ext_vector_type(16)));
typedef _Float16 h2 __attribute__((ext_vector_type(2)));
typedef _Float16 h8 __attribute__((ext_vector_type(8)));
typedef short s4 __attribute__((ext_vector_type(4)));
typedef short s8 __attribute__((ext_vector_type(8)));
typedef unsigned long long u64;

constexpr int NR = 8;
constexpr int KDIM = 4096;
constexpr int ROWS = 512;
constexpr int COLS = 512;
constexpr unsigned OBJ = 256u * 1024u;
constexpr int NG = 6;
constexpr int NA = 12;
constexpr size_t WS_CNT = 0;
constexpr size_t WS_TMO = 16 * 1024;
constexpr size_t WS_TICK = 20 * 1024;
constexpr size_t WS_FLAG = 32 * 1024;
constexpr size_t WS_DONE = 64 * 1024;
constexpr size_t WS_STGF = 80 * 1024;
constexpr size_t WS_WIPE = 100 * 1024;
constexpr size_t WS_DIAG = 128 * 1024;
constexpr size_t WS_DATA = 1024 * 1024;
constexpr unsigned NSLOT = 160;
constexpr size_t PAR_BYTES = (size_t)NSLOT * OBJ;
constexpr size_t WS_STG = WS_DATA + 2 * PAR_BYTES;
constexpr size_t WS_NEED = WS_STG + 128 * 12 * 4096;
constexpr u64 DEADLINE = 100000000ull;
#define FWD_T0 2500
#define SMODE 1
__host__ __device__ constexpr int lent_strip(int o, int h, int i) {
  return SMODE == 0 ? h * 4 + i : (SMODE == 1 ? ((o + h) & 3) * 4 + i : i * 4 + ((o + h) & 3));
}
#define FW 5
#define KA 4
#define KC 4
#define RP 2
#define INB 1
#define WCHK 5
#define SSL 2
#define SOFF 8188
#define KL 4
#define KEEP2 2
#define AGGN 8
#define AGGC 7
#define TOUCH 3
#define LAG 30
#define KEEP 4
constexpr unsigned L_PART = 128 * 1024;
constexpr unsigned L_CTRL = L_PART + 16 * 1024;
constexpr unsigned L_TOTAL = L_CTRL + 64;

static_assert(AGGN < NA, "unit NA-1 has halves with fewer than 8 tickets");

struct alignas(64) Head {
  const char* x;
  const char* w;
  char* myws;
  char* dws;
  int rank;
  int ch;
};
struct Params {
  Head head[8];
  char* ws[NR];
  char* out[NR];
  unsigned units[NG][5][4];
};

__device__ __forceinline__ unsigned rfl(unsigned v) { return __builtin_amdgcn_readfirstlane(v); }
__device__ __forceinline__ unsigned pk(float a, float b) {
  h2 v; v.x = (_Float16)a; v.y = (_Float16)b; return __builtin_bit_cast(unsigned, v);
}
__device__ __forceinline__ u32x4 cvt8(u32x4 a, u32x4 b) {
  f32x4 fa = __builtin_bit_cast(f32x4, a), fb = __builtin_bit_cast(f32x4, b);
  u32x4 r; r.x = pk(fa.x, fa.y); r.y = pk(fa.z, fa.w); r.z = pk(fb.x, fb.y); r.w = pk(fb.z, fb.w);
  return r;
}
__device__ __forceinline__ float gelu(float y) {
  const float c2 = -2.0f * 0.7978845608028654f * 1.4426950408889634f;
  float u = y + 0.044715f * y * y * y;
  float e = __builtin_amdgcn_exp2f(c2 * u);
  return y * __builtin_amdgcn_rcpf(1.0f + e);
}

#define STAMPV(idx, val) do {} while (0)
#define STAMP(idx) STAMPV(idx, __builtin_amdgcn_s_memrealtime())

__device__ __forceinline__ bool give_up(u64 t0, unsigned* tmo, unsigned call) {
  u64 el = __builtin_amdgcn_s_memrealtime() - t0;
  if (el < 100000ull) return false;
  if (el > DEADLINE ||
      rfl(__hip_atomic_load(tmo, __ATOMIC_RELAXED, __HIP_MEMORY_SCOPE_AGENT)) == call) {
    __hip_atomic_store(tmo, call, __ATOMIC_RELAXED, __HIP_MEMORY_SCOPE_AGENT);
    return true;
  }
  return false;
}

__device__ __forceinline__ unsigned flag_word(unsigned pc) {
  return ((pc >> 4) & 1) * 16 + ((pc >> 1) & 3) * 4 + ((pc >> 3) & 1) * 2 + (pc & 1);
}
__device__ __forceinline__ unsigned flag_load(const unsigned* p) {
  return __hip_atomic_load(p, __ATOMIC_RELAXED, __HIP_MEMORY_SCOPE_SYSTEM);
}
__device__ __forceinline__ unsigned flag_mask(unsigned v, unsigned call) {
  return (unsigned)__builtin_amdgcn_ballot_w64(v >= call) & 0xFFu;
}
__device__ __forceinline__ void wait_round(const unsigned* p, int r, unsigned& seen, bool& bad, u64 t0,
                                           unsigned* tmo, unsigned call, unsigned* lidle, unsigned mark) {
  while (((seen >> (2 * r)) & 3u) != 3u) {
    seen = flag_mask(flag_load(p), call);
    if (((seen >> (2 * r)) & 3u) == 3u) break;
    if (r < 3) __hip_atomic_store(lidle, mark, __ATOMIC_RELAXED, __HIP_MEMORY_SCOPE_WORKGROUP);
    if (give_up(t0, tmo, call)) { bad = true; break; }
    __builtin_amdgcn_s_sleep(2);
  }
  asm volatile("" ::: "memory");
}

struct Piece { u32x4 r[16]; };

__device__ __forceinline__ void piece_load(Piece& s, const char* sb, unsigned fstride, unsigned voff) {
#pragma unroll
  for (int u = 0; u < 8; ++u) {
    const char* p = sb + (size_t)u * fstride;
    asm volatile("global_load_dwordx4 %0, %1, %2" : "=v"(s.r[2 * u]) : "v"(voff), "s"(p) : "memory");
    asm volatile("global_load_dwordx4 %0, %1, %2 offset:16" : "=v"(s.r[2 * u + 1]) : "v"(voff), "s"(p) : "memory");
  }
}
template <int N>
__device__ __forceinline__ void piece_wait(Piece& s) {
  asm volatile("s_waitcnt vmcnt(%0)" :: "i"(N) : "memory");
#pragma unroll
  for (int u = 0; u < 16; ++u) asm volatile("" : "+v"(s.r[u]));
}
template <int U>
__device__ __forceinline__ void piece_store1(Piece& s, char* db, unsigned voff) {
  u32x4 v = cvt8(s.r[2 * U], s.r[2 * U + 1]);
  asm volatile("global_store_dwordx4 %0, %1, %2 offset:%3 sc0 sc1\n\ts_nop 1"
               :: "v"(voff), "v"(v), "s"(db), "i"((U - 4) * 1024) : "memory");
}
__device__ __forceinline__ void piece_store(Piece& s, char* db  , unsigned voff) {
  piece_store1<0>(s, db, voff); piece_store1<1>(s, db, voff); piece_store1<2>(s, db, voff);
  piece_store1<3>(s, db, voff); piece_store1<4>(s, db, voff); piece_store1<5>(s, db, voff);
  piece_store1<6>(s, db, voff); piece_store1<7>(s, db, voff);
}

typedef __attribute__((address_space(3))) char lds_char;

__device__ __forceinline__ void glds16(const char* src, unsigned voff, unsigned lds) {
  unsigned keep;
  asm volatile("s_mov_b32 %0, m0\n\ts_mov_b32 m0, %3\n\ts_nop 0\n\t"
               "global_load_lds_dwordx4 %1, %2 sc0 sc1\n\ts_mov_b32 m0, %0"
               : "=&s"(keep) : "v"(voff), "s"(src), "s"(lds) : "memory");
}

__device__ __forceinline__ h8 tr_frag(lds_char* p) {
  s4 a = __builtin_amdgcn_ds_read_tr16_b64_v4i16((__attribute__((address_space(3))) s4*)p);
  s4 c = __builtin_amdgcn_ds_read_tr16_b64_v4i16((__attribute__((address_space(3))) s4*)(p + 256));
  s8 r = __builtin_shufflevector(a, c, 0, 1, 2, 3, 4, 5, 6, 7);
  return __builtin_bit_cast(h8, r);
}

__global__ void __launch_bounds__(384) ag_gemm_gelu(Params P) {
  __shared__ __attribute__((aligned(1024))) char smem[L_TOTAL];
  const int lane = threadIdx.x & 63;
  const int wave = rfl(threadIdx.x >> 6);
  Head H = P.head[__builtin_amdgcn_s_getreg(20 | (3 << 11)) & 7];
  asm volatile("" : "+s"(H.x), "+s"(H.w), "+s"(H.myws), "+s"(H.dws), "+s"(H.rank), "+s"(H.ch));
  const int xch = H.ch & 7;
  const int b = blockIdx.x < 64 ? xch * 8 + (blockIdx.x >> 3) : blockIdx.x;
  const int rank = H.rank & (NR - 1);
  const int side = rank >> 2, q = rank & 3;
  char* const myws = (char*)(__attribute__((address_space(1))) char*)(u64)H.myws;
  unsigned* const cntp = (unsigned*)(myws + WS_CNT + b * 32 + wave * 4);
  unsigned* const lcnt = (unsigned*)(smem + L_CTRL);
  unsigned* const lidle = (unsigned*)(smem + L_CTRL + 4);
  unsigned* const lfree = (unsigned*)(smem + L_CTRL + 8);

  if (__builtin_expect(wave == 0, 1)) {
    if (__builtin_expect(b >= 64 + 128 || (b >> 3) == 7, 0)) {
      const int c = xch;
      if ((TOUCH & 1) && (b >> 3) == 24 && c < 7) {
        const int dest = (c < 3 ? side * 4 + ((q + 1 + c) & 3) : (side ^ 1) * 4 + ((q + c - 3) & 3)) & (NR - 1);
        const char* const dd = H.dws + WS_DATA + (lane & 1) * PAR_BYTES;
        const char* a1 = lane < 2 ? dd + (size_t)(c < 3 ? rank * 16 : 144 + q * 4) * OBJ : H.dws + WS_FLAG;
        const char* a2 = lane < 32 ? dd + (size_t)(rank * 16 + (lane >> 1)) * OBJ : P.out[dest] + (size_t)(lane - 32) * OBJ;
        unsigned v1, v2;
        asm volatile("global_load_dword %0, %2, off sc0 sc1\n\tglobal_load_dword %1, %3, off sc0 sc1\n\ts_waitcnt vmcnt(0)"
                     : "=&v"(v1), "=&v"(v2) : "v"(a1), "v"(a2) : "memory");
      }
      return;
    }
    __builtin_amdgcn_s_setprio(3);
    __builtin_amdgcn_s_waitcnt(0);
    const int j = b & 7, c = b >> 3;
    int dest, nW, strip0, wslot0, xu0, nobj;
    if (c < 3)      { dest = side * 4 + ((q + 1 + c) & 3); nW = 0; strip0 = 0; wslot0 = 0; xu0 = 0; nobj = NA; }
    else if (c < 7) { dest = (side ^ 1) * 4 + ((q + c - 3) & 3); nW = 4; strip0 = lent_strip(q, dest & 3, 0); wslot0 = 16 + q * 4;
                      xu0 = NA; nobj = 4 + 16 - NA; }
    else            { dest = rank; nW = 1; strip0 = c - 8; wslot0 = c - 8; xu0 = c - 8; nobj = 2; }
    const unsigned np = nobj * 32;
    const unsigned nt = np / 2 - (c < 3 ? 2 * RP : 0);
    const unsigned laneX = (lane & 31) * (KDIM * 4) + (lane >> 5) * 32;
    const unsigned laneW = (lane >> 2) * (COLS * 4) + (lane & 3) * 32;
    const unsigned lane16 = lane * 16;
    const char* const px = H.x;
    const char* const pw = H.w;
    if (LAG && b >= 64) __builtin_amdgcn_s_sleep(LAG);
    auto load = [&](Piece& s, unsigned pc) {
      pc = pc < np ? pc : np - 1;
      const int obj = pc >> 5, f0 = (pc & 31) * 8;
      const bool isW = obj < nW;
      const char* sb = isW ? pw + (size_t)(strip0 + obj * (lent_strip(0, 0, 1) - lent_strip(0, 0, 0))) * 128 + (size_t)f0 * (16 * COLS * 4)
                           : px + (size_t)(xu0 + obj - nW) * (32 * KDIM * 4) + (size_t)f0 * 64;
      piece_load(s, sb, isW ? 16 * COLS * 4 : 64, isW ? laneW : laneX);
    };
    unsigned cnt;
    asm volatile("global_load_dword %0, %1, off sc1" : "=v"(cnt) : "v"(cntp) : "memory");
    Piece A, B;
    load(A, j);
    load(B, j + 8);
    piece_wait<16>(A);
    asm volatile("" : "+v"(cnt));
    const unsigned call = rfl(cnt) + 1;
    char* const dws = c < 8 ? H.dws : myws;
    char* const ddata = dws + WS_DATA + (size_t)(call & 1) * PAR_BYTES + 4096;
    char* const tick = myws + WS_TICK + c * 128;
    const unsigned tbase = 16 - (call - 1) * nt;
    auto slot_of = [&](unsigned pc) -> unsigned {
      const int obj = pc >> 5;
      return obj < nW ? 128 + wslot0 + obj : rank * 16 + xu0 + obj - nW;
    };
    auto store = [&](Piece& s, unsigned pc) {
      piece_store(s, ddata + (size_t)slot_of(pc) * OBJ + (pc & 31) * 8192, lane16);
    };
    const unsigned one = 1;
    auto flag = [&](unsigned pc, unsigned m) {
      char* fp = dws + WS_FLAG + slot_of(pc) * 128 + (pc & 16) * 4;
      if ((m >> (lane & 31)) & (lane < 16)) asm volatile("global_store_dword %0, %1, %2 sc0 sc1" :: "v"(lane * 4), "v"(call), "s"(fp) : "memory");
    };
    unsigned dn = 0, pend = np;
    auto count = [&](unsigned& r, unsigned off, unsigned lanes) {
      asm volatile("s_mov_b32 exec_lo, %4\n\ts_mov_b32 exec_hi, 0\n\tglobal_atomic_add %0, %1, %2, %3 sc0\n\ts_mov_b64 exec, -1"
                   : "+v"(r) : "v"(off), "v"(one), "s"(tick), "s"(lanes) : "memory");
    };
    auto done = [&](unsigned pc) {
      const bool agg = c < AGGC && (int)(pc >> 5) >= nW && pc < AGGN * 32;
      count(dn, 16 + (pc >> 4) * 4, rfl(agg));
      if (agg) pend = pc;
      else flag(pc, 5u << (flag_word(pc) & 15));
    };
    auto post = [&]() {
      asm volatile("" : "+v"(dn));
      if (pend < np && rfl(dn) + 1 == call * 8) flag(pend, 0xFFFFu);
      pend = np;
    };
    STAMP(0);
    auto piece_of = [&](unsigned t) -> unsigned {
      if (t >= NA * 16 - 24) t += t < NA * 16 - 16 - 2 * RP ? 8 + 2 * RP : (t < NA * 16 - 8 - 2 * RP ? 2 * RP - 8 : 2 * RP);
      return (t >> 3) * 16 + (t & 7);
    };
    unsigned tk = 0;
    unsigned cur = j, nxt = j + 16;
    store(A, cur);
    count(tk, 0, 1);
    load(A, nxt);
    piece_wait<25>(B);
    if (lane == 0) asm volatile("global_store_dword %0, %1, off sc1" :: "v"(cntp), "v"(call) : "memory");
    store(B, cur + 8);
    load(B, nxt + 8);
    __builtin_amdgcn_s_barrier();
    bool gate = INB && c < 3;
    do {
      piece_wait<25>(A);
      asm volatile("" : "+v"(tk));
      unsigned t = rfl(tk) + tbase;
      const unsigned nn = t < nt ? piece_of(t) : np;
      if (c < 3) asm volatile("s_waitcnt vmcnt(%0)" :: "i"(16 + KA) : "memory");
      else if (c < 7) asm volatile("s_waitcnt vmcnt(%0)" :: "i"(16 + KC) : "memory");
      post();
      if (gate && nxt >= (NA - 2) * 32) { __builtin_amdgcn_s_barrier(); gate = false; }
      store(A, nxt);
      count(tk, 0, 1);
      load(A, nn);
      piece_wait<25>(B);
      done(cur);
      if (c < 3) asm volatile("s_waitcnt vmcnt(%0)" :: "i"(18 + KA) : "memory");
      else if (c < 7) asm volatile("s_waitcnt vmcnt(%0)" :: "i"(18 + KC) : "memory");
      store(B, nxt + 8);
      load(B, nn + 8);
      cur = nxt;
      nxt = nn;
    } while (nxt < np);
    if (b < 64) __hip_atomic_store(lfree, 1u, __ATOMIC_RELAXED, __HIP_MEMORY_SCOPE_WORKGROUP);
    piece_wait<24>(A);
    asm volatile("" : "+v"(tk));
    post();
    piece_wait<0>(B);
    done(cur);
    asm volatile("s_waitcnt vmcnt(0)" ::: "memory");
    post();
    STAMP(1);
    return;
  }

  const bool fwd = b < 64;
  if (fwd && (b < 24 || b >= 56)) {
    if (INB && b < 24 && wave <= 2) {
      __builtin_amdgcn_s_barrier();
      __builtin_amdgcn_s_sleep(100);
      const unsigned call = rfl(__hip_atomic_load(cntp, __ATOMIC_RELAXED, __HIP_MEMORY_SCOPE_AGENT)) + 1;
      if (lane == 0) __hip_atomic_store(cntp, call, __ATOMIC_RELAXED, __HIP_MEMORY_SCOPE_AGENT);
      const unsigned pq = (q + 1 + (b >> 3)) & 3;
      const unsigned slot = (side * 4 + pq) * 16 + NA - wave;
      char* const zp = myws + WS_DATA + (size_t)(call & 1) * PAR_BYTES + (size_t)slot * OBJ + OBJ / 2 + (b & 7) * 16384;
      const u32x4 z = {0u, 0u, 0u, 0u};
#pragma unroll
      for (int i = 0; i < 16; ++i)
        asm volatile("global_store_dwordx4 %0, %1, %2 sc0 sc1" :: "v"(lane * 16 + i * 1024), "v"(z), "s"(zp) : "memory");
      asm volatile("s_waitcnt vmcnt(0)" ::: "memory");
      if (lane == 0) {
        const unsigned wo = ((b & 7) * 2 + wave - 1) * 4;
        asm volatile("global_store_dword %0, %1, %2 sc0 sc1" :: "v"(wo), "v"(call), "s"(H.dws + WS_WIPE + q * 128) : "memory");
        asm volatile("global_store_dword %0, %1, %2 sc0 sc1" :: "v"(wo), "v"(call), "s"(myws + WS_WIPE + 1024 + pq * 128) : "memory");
      }
      if (wave == 1) {
        const u64 t0 = __builtin_amdgcn_s_memrealtime();
        const unsigned* const wp = (const unsigned*)(myws + WS_WIPE + pq * 128) + (lane & 15);
        for (;;) {
          __builtin_amdgcn_s_sleep(64);
          if (__builtin_amdgcn_ballot_w64(flag_load(wp) < call) == 0) break;
          if (give_up(t0, (unsigned*)(myws + WS_TMO), call)) break;
        }
      }
      asm volatile("s_waitcnt vmcnt(0)" ::: "memory");
    }
    return;
  }
  const u64 t0 = __builtin_amdgcn_s_memrealtime();
  if (LAG && !fwd) __builtin_amdgcn_s_sleep(LAG);
  const unsigned call = rfl(__hip_atomic_load(cntp, __ATOMIC_RELAXED, __HIP_MEMORY_SCOPE_AGENT)) + 1;
  if (lane == 0) __hip_atomic_store(cntp, call, __ATOMIC_RELAXED, __HIP_MEMORY_SCOPE_AGENT);
  unsigned* const tmo = (unsigned*)(myws + WS_TMO);
  const int rr = lane >> 3, chn = lane & 7;
  const unsigned ovoff = rr * (COLS * 4) + chn * 16;
  bool bad = false;
  if (wave == 1) {
    STAMPV(5, t0);
    if (lane < 8) lcnt[lane] = 0;
    asm volatile("s_waitcnt lgkmcnt(0)" ::: "memory");
  }
  __builtin_amdgcn_s_barrier();

  if (fwd) {
    const int j = b & 7, p = (q + (b >> 3) - 3) & 3;
    const int owner = ((side ^ 1) * 4 + p) & (NR - 1);
    char* const obase = P.out[owner];
    unsigned* const lnext = (unsigned*)(smem + L_CTRL + 16);
    const unsigned tl = lane < 3 ? lane : 0;
    const unsigned* const stgf = (const unsigned*)(myws + WS_STGF + ((p * 4 + ((j + tl) & 3)) * 8 + 2 * tl + (j >> 2)) * 128);
    const unsigned n0 = P.units[j >> 2][1 + p][3], n1 = P.units[2 + (j >> 2)][1 + p][3], n2 = P.units[4 + (j >> 2)][1 + p][3];
    const unsigned nl = (tl == 0 ? n0 : (tl == 1 ? n1 : n2)) - 1;
    while (__builtin_amdgcn_s_memrealtime() - t0 < FWD_T0) __builtin_amdgcn_s_sleep(16);
    if (RP && wave == 5) {
      const size_t dbase = WS_DATA + (size_t)(call & 1) * PAR_BYTES + 4096;
      auto rpiece = [](unsigned n) -> unsigned { return n % (2 * RP) + n / (2 * RP) * 8; };
      if (j >= 8 - RP) {
        const unsigned slot = rank * 16 + NA - 1, pc = rpiece(p * RP + 7 - j);
        Piece A;
        piece_load(A, H.x + (size_t)(NA - 1) * (32 * KDIM * 4) + pc * 512, 64, (lane & 31) * (KDIM * 4) + (lane >> 5) * 32);
        piece_wait<0>(A);
        while (rfl(__hip_atomic_load(lfree, __ATOMIC_RELAXED, __HIP_MEMORY_SCOPE_WORKGROUP)) == 0) {
          if (__builtin_amdgcn_s_memrealtime() - t0 > 2 * DEADLINE) break;
          __builtin_amdgcn_s_sleep(2);
        }
        piece_store(A, P.ws[owner] + dbase + (size_t)slot * OBJ + pc * 8192, lane * 16);
        asm volatile("s_waitcnt vmcnt(0)" ::: "memory");
        if (lane == 0) __hip_atomic_store((unsigned*)(P.ws[owner] + WS_FLAG + slot * 128 + flag_word(pc) * 4), call, __ATOMIC_RELAXED, __HIP_MEMORY_SCOPE_SYSTEM);
      }
      for (int n = j; n < 3 * RP; n += 8) {
        const unsigned slot = ((side ^ 1) * 4 + ((p + 1 + n / RP) & 3)) * 16 + NA - 1, pc = rpiece(q * RP + n % RP);
        const size_t fo = WS_FLAG + slot * 128 + flag_word(pc) * 4;
        const size_t go = dbase + (size_t)slot * OBJ + pc * 8192;
        while (rfl(flag_load((const unsigned*)(myws + fo))) < call) {
          if (give_up(t0, tmo, call)) { bad = true; break; }
          __builtin_amdgcn_s_sleep(8);
        }
        if (bad) break;
        u32x4 y0, y1, y2, y3, y4, y5, y6, y7;
        asm volatile("global_load_dwordx4 %0, %8, %9 offset:-4096 sc0 sc1\n\t"
                     "global_load_dwordx4 %1, %8, %9 offset:-3072 sc0 sc1\n\t"
                     "global_load_dwordx4 %2, %8, %9 offset:-2048 sc0 sc1\n\t"
                     "global_load_dwordx4 %3, %8, %9 offset:-1024 sc0 sc1\n\t"
                     "global_load_dwordx4 %4, %8, %9 sc0 sc1\n\t"
                     "global_load_dwordx4 %5, %8, %9 offset:1024 sc0 sc1\n\t"
                     "global_load_dwordx4 %6, %8, %9 offset:2048 sc0 sc1\n\t"
                     "global_load_dwordx4 %7, %8, %9 offset:3072 sc0 sc1\n\t"
                     "s_waitcnt vmcnt(0)\n\t"
                     "global_store_dwordx4 %8, %0, %10 offset:-4096 sc0 sc1\n\t"
                     "global_store_dwordx4 %8, %1, %10 offset:-3072 sc0 sc1\n\t"
                     "global_store_dwordx4 %8, %2, %10 offset:-2048 sc0 sc1\n\t"
                     "global_store_dwordx4 %8, %3, %10 offset:-1024 sc0 sc1\n\t"
                     "global_store_dwordx4 %8, %4, %10 sc0 sc1\n\t"
                     "global_store_dwordx4 %8, %5, %10 offset:1024 sc0 sc1\n\t"
                     "global_store_dwordx4 %8, %6, %10 offset:2048 sc0 sc1\n\t"
                     "global_store_dwordx4 %8, %7, %10 offset:3072 sc0 sc1\n\t"
                     "s_waitcnt vmcnt(0)"
                     : "=&v"(y0), "=&v"(y1), "=&v"(y2), "=&v"(y3), "=&v"(y4), "=&v"(y5), "=&v"(y6), "=&v"(y7)
                     : "v"(lane * 16), "s"(myws + go), "s"(P.ws[owner] + go) : "memory");
        if (lane == 0) __hip_atomic_store((unsigned*)(P.ws[owner] + fo), call, __ATOMIC_RELAXED, __HIP_MEMORY_SCOPE_SYSTEM);
        STAMP(46 + (n >> 3));
      }
    }
    unsigned fv = 0;
    bool open = false;
    while (wave <= FW) {
      const unsigned nx = __hip_atomic_load(lnext + tl, __ATOMIC_RELAXED, __HIP_MEMORY_SCOPE_WORKGROUP);
      const unsigned left = (unsigned)__builtin_amdgcn_ballot_w64(nx < nl) & 7u;
      if (left == 0) break;
      const unsigned rdy = rfl(bad ? left : (unsigned)__builtin_amdgcn_ballot_w64(fv > call * 16 + nx) & left);
      if (rdy == 0) {
        fv = __hip_atomic_load(stgf, __ATOMIC_RELAXED, __HIP_MEMORY_SCOPE_SYSTEM);
        if (give_up(t0, tmo, call)) bad = true;
        __builtin_amdgcn_s_sleep(2);
        continue;
      }
      const unsigned t = __builtin_ctz(rdy);
      const unsigned k = __builtin_amdgcn_readlane(nx, t);
      unsigned got = k;
      if (lane == 0) __hip_atomic_compare_exchange_strong(lnext + t, &got, k + 1, __ATOMIC_RELAXED, __ATOMIC_RELAXED, __HIP_MEMORY_SCOPE_WORKGROUP);
      if (rfl(got) != k) continue;
      const unsigned g = 2 * t + (j >> 2), i = (j + t) & 3;
      const unsigned unit = (P.units[g][1 + p][k >> 2] >> ((k & 3) * 8)) & 127u;
      const char* const src = myws + WS_STG + (size_t)(((p * 4 + i) * 8 + g) * 12 + k) * 4096;
      f32x4 y0, y1, y2, y3;
      asm volatile("global_load_dwordx4 %0, %5, %6 sc0 sc1\n\t"
                   "global_load_dwordx4 %1, %5, %6 offset:1024 sc0 sc1\n\t"
                   "global_load_dwordx4 %2, %5, %6 offset:2048 sc0 sc1\n\t"
                   "global_load_dwordx4 %3, %5, %6 offset:3072 sc0 sc1\n\t"
                   "global_load_dword %4, %7, off sc0 sc1\n\t"
                   "s_waitcnt vmcnt(0)"
                   : "=&v"(y0), "=&v"(y1), "=&v"(y2), "=&v"(y3), "=&v"(fv)
                   : "v"(lane * 16), "s"(src), "v"(stgf) : "memory");
      if (!open) {
        while (rfl(__hip_atomic_load(lfree, __ATOMIC_RELAXED, __HIP_MEMORY_SCOPE_WORKGROUP)) == 0) {
          if (__builtin_amdgcn_s_memrealtime() - t0 > 2 * DEADLINE) break;
          __builtin_amdgcn_s_sleep(2);
        }
        open = true;
        if (wave < 5) STAMP(6 + wave);
      }
      STAMP(16 + t * 10 + k);
      if (bad) y0.x = __builtin_nanf("");
      const unsigned vo = ovoff + unit * (32 * COLS * 4) + lent_strip(p, q, i) * 128, vs = 8 * COLS * 4;
      asm volatile("global_store_dwordx4 %0, %4, %8 sc0 sc1\n\t"
                   "global_store_dwordx4 %1, %5, %8 sc0 sc1\n\t"
                   "global_store_dwordx4 %2, %6, %8 sc0 sc1\n\t"
                   "global_store_dwordx4 %3, %7, %8 sc0 sc1\n\ts_nop 1"
                   :: "v"(vo), "v"(vo + vs), "v"(vo + 2 * vs), "v"(vo + 3 * vs),
                      "v"(y0), "v"(y1), "v"(y2), "v"(y3), "s"(obase) : "memory");
    }
    asm volatile("s_waitcnt vmcnt(0)" ::: "memory");
    unsigned* const dp = (unsigned*)(P.ws[owner] + WS_DONE + q * 1024 + (j * 5 + wave - 1) * 4);
    if (lane == 0) __hip_atomic_store(dp, call, __ATOMIC_RELAXED, __HIP_MEMORY_SCOPE_SYSTEM);
    STAMP(10 + wave);
    return;
  }

  const int nq = (b & 3) * 24 + (b >> 3) - 8;
  const int g = ((b >> 2) & 1) * 3 + (nq >> 5), s = nq & 31;
  const bool foreign = s >= 16;
  const int owner = (foreign ? (side ^ 1) * 4 + ((s - 16) >> 2) : rank) & (NR - 1);
  const int cls = foreign ? 1 + (owner & 3) : 0;
  const unsigned ul0 = P.units[g][cls][0], ul1 = P.units[g][cls][1], ul2 = P.units[g][cls][2];
  const unsigned nu = P.units[g][cls][3] < 12u ? P.units[g][cls][3] : 12u;
  auto unit_at = [&](unsigned k) -> unsigned {
    unsigned wv = k < 4 ? ul0 : (k < 8 ? ul1 : ul2);
    return (wv >> ((k & 3) * 8)) & 127u;
  };

  const int col0 = (foreign ? lent_strip(owner & 3, q, s & 3) : s) * 32;
  char* const obase = P.out[owner] + col0 * 4;
  unsigned* const donep = (unsigned*)(P.ws[owner] + WS_DONE + q * 1024 + 160 + ((s & 3) * 6 + g) * 16);
  const char* const pl = smem + L_PART + rr * 128 + ((chn ^ rr) * 16);
  if (wave == 1) {
    const unsigned prod = (s & 15) * 8 + g;
    char* const stg = myws + WS_STG + (size_t)prod * (12 * 4096);
    unsigned* const stgf = (unsigned*)(myws + WS_STGF + prod * 128);
    if ((TOUCH & 2) && foreign) {
      const char* a1 = lane & 1 ? (const char*)donep : obase + (size_t)unit_at(nu - 1) * (32 * COLS * 4);
      unsigned v1;
      __builtin_amdgcn_s_sleep(100);
      asm volatile("global_load_dword %0, %1, off sc0 sc1\n\ts_waitcnt vmcnt(0)" : "=&v"(v1) : "v"(a1) : "memory");
    }
    f32x4 keep[KEEP][4];
    unsigned kd = 0, ks = 0;
    while (ks + 1 < nu) {
      const u64 st = __hip_atomic_load((u64*)lcnt, __ATOMIC_RELAXED, __HIP_MEMORY_SCOPE_WORKGROUP);
      const unsigned have = rfl((unsigned)st), idle = rfl((unsigned)(st >> 32));
      if (__builtin_amdgcn_s_memrealtime() - t0 > 2 * DEADLINE) bad = true;
      if (have >= (kd + 1) * 4 || bad) {
        asm volatile("" ::: "memory");
        f32x4 y[4];
#pragma unroll
        for (int it = 0; it < 4; ++it) {
          f32x4 p0 = *(const f32x4*)(pl + it * 1024);
          f32x4 p1 = *(const f32x4*)(pl + it * 1024 + 4096);
          f32x4 p2 = *(const f32x4*)(pl + it * 1024 + 8192);
          f32x4 p3 = *(const f32x4*)(pl + it * 1024 + 12288);
          y[it] = (p0 + p1) + (p2 + p3);
        }
#pragma unroll
        for (int it = 0; it < 4; ++it) asm volatile("" : "+v"(y[it]));
        asm volatile("s_waitcnt lgkmcnt(0)" ::: "memory");
        if (lane == 0) __hip_atomic_store(lfree, kd + 1, __ATOMIC_RELAXED, __HIP_MEMORY_SCOPE_WORKGROUP);
#pragma unroll
        for (int it = 0; it < 4; ++it) {
          y[it].x = gelu(y[it].x); y[it].y = gelu(y[it].y); y[it].z = gelu(y[it].z); y[it].w = gelu(y[it].w);
          if (bad) y[it].x = __builtin_nanf("");
        }
#pragma unroll
        for (unsigned k = 0; k < KEEP; ++k)
          if (kd - ks == k) {
#pragma unroll
            for (int it = 0; it < 4; ++it) { keep[k][it] = y[it]; asm volatile("" : "+v"(keep[k][it])); }
          }
        ++kd;
      } else if (ks == kd || idle != kd + 1) {
        __builtin_amdgcn_s_sleep(1);
        continue;
      }
      if (kd - ks >= (foreign && kd > KL ? KEEP2 : KEEP) || kd + 1 >= nu || idle == kd + 1) {
        const bool stage = foreign;
        char* const base = stage ? stg : obase;
        const unsigned step = stage ? 1024 : 8 * COLS * 4;
#pragma unroll
        for (unsigned k = 0; k < KEEP; ++k)
          if (ks + k < kd) {
            const unsigned vo = stage ? lane * 16 + (ks + k) * 4096 : ovoff + unit_at(ks + k) * (32 * COLS * 4);
#pragma unroll
            for (int it = 0; it < 4; ++it)
              asm volatile("global_store_dwordx4 %0, %1, %2 sc0 sc1\n\ts_nop 1"
                           :: "v"(vo + it * step), "v"(keep[k][it]), "s"(base) : "memory");
            STAMP(32 + ks + k);
          }
        ks = kd;
        if (stage) {
          asm volatile("s_waitcnt vmcnt(0)" ::: "memory");
          if (lane == 0) __hip_atomic_store(stgf, call * 16 + kd, __ATOMIC_RELAXED, __HIP_MEMORY_SCOPE_SYSTEM);
        }
      }
    }
    if (g == 0 && s < 4) {
      const char* dp = myws + WS_DONE + s * 1024 + (lane < 34 ? lane : 0) * 16;
      while (true) {
        u32x4 v;
        asm volatile("global_load_dwordx4 %0, %1, off sc0 sc1\n\ts_waitcnt vmcnt(0)" : "=&v"(v) : "v"(dp) : "memory");
        bool ok = v.x >= call && v.y >= call && v.z >= call && v.w >= call;
        if (__all(ok)) break;
        if (give_up(t0, tmo, call)) {
          char* const lastp = obase + (size_t)unit_at(0) * (32 * COLS * 4);
          if (lane == 0) __hip_atomic_store((float*)lastp, __builtin_nanf(""), __ATOMIC_RELAXED, __HIP_MEMORY_SCOPE_SYSTEM);
          break;
        }
      }
      STAMP(3);
    }
    return;
  }

  const int w = wave - 2;
  char* const data = myws + WS_DATA + (size_t)(call & 1) * PAR_BYTES;
  const unsigned* const flags = (const unsigned*)(myws + WS_FLAG) + (lane & 4) * 4 + w * 4 + (lane & 3);
  lds_char* const lw = (lds_char*)smem + w * 32768;
  lds_char* const ltr = lw + ((lane >> 5) * 8 + ((lane & 15) >> 2)) * 64 + ((lane >> 4) & 1) * 32 + (lane & 3) * 8;
  h8 wr[32];
  {
    const unsigned* pp = flags + (128 + s) * 32;
    const char* src = data + (size_t)(128 + s) * OBJ + w * 16384;
    const unsigned svoff = lane * 16;
    const unsigned lwa = (unsigned)(size_t)lw;
    unsigned seen = 0;
#pragma unroll
    for (int r = 0; r < 4; ++r) {
      wait_round(pp, r, seen, bad, t0, tmo, call, lidle, 0);
#pragma unroll
      for (int i = 0; i < 16; ++i)
        glds16(src + r * 65536 + i * 1024, svoff, lwa + (r & 1) * 16384 + i * 1024);
      if (r & 1) asm volatile("s_waitcnt vmcnt(0)" ::: "memory");
      if (r == 1) {
#pragma unroll
        for (int i = 0; i < 32; ++i) wr[i] = tr_frag(ltr + i * 1024);
#pragma unroll
        for (int i = 0; i < 32; ++i) asm volatile("" : "+v"(wr[i]));
        asm volatile("s_waitcnt lgkmcnt(0)" ::: "memory");
      }
    }
  }
  if (w == 0) STAMP(4);
  const __amdgpu_buffer_rsrc_t rsrc =
      __builtin_amdgcn_make_buffer_rsrc(data, 0, 128 * OBJ, 0x00020000);
  const unsigned xvoff = w * 16384 + lane * 16;
  char* const pw = smem + L_PART + w * 4096 + (lane & 31) * 128;
  unsigned nextf = 0, seen, soff;
  const unsigned* pp;
  f32x16 acc;
  auto xload = [&](u32x4 (&xa)[16], int r) {
#pragma unroll
    for (int i = 0; i < 16; ++i)
      xa[i] = __builtin_amdgcn_raw_buffer_load_b128(rsrc, xvoff + (i & 3) * 1024, soff + r * 65536 + (i >> 2) * 4096, 17);
    __builtin_amdgcn_sched_barrier(0);
  };
  auto xmul = [&](u32x4 (&xa)[16], int r) {
#pragma unroll
    for (int i = 0; i < 16; ++i) {
      h8 wf = r < 2 ? wr[r * 16 + i] : tr_frag(ltr + ((r - 2) * 16 + i) * 1024);
      acc = __builtin_amdgcn_mfma_f32_32x32x16_f16(wf, __builtin_bit_cast(h8, xa[i]), acc, 0, 0, 0);
    }
  };
  auto part = [&](unsigned k) {
    if (bad) {
#pragma unroll
      for (int i = 0; i < 16; ++i) acc[i] = __builtin_nanf("");
    }
    while (rfl(__hip_atomic_load(lfree, __ATOMIC_RELAXED, __HIP_MEMORY_SCOPE_WORKGROUP)) < k) {
      if (__builtin_amdgcn_s_memrealtime() - t0 > 2 * DEADLINE) break;
      __builtin_amdgcn_s_sleep(1);
    }
    asm volatile("" ::: "memory");
#pragma unroll
    for (int gq = 0; gq < 4; ++gq) {
      f32x4 v = {acc[gq * 4], acc[gq * 4 + 1], acc[gq * 4 + 2], acc[gq * 4 + 3]};
      *(f32x4*)(pw + (((gq * 2 + (lane >> 5)) ^ (lane & 7)) * 16)) = v;
    }
    asm volatile("s_waitcnt lgkmcnt(0)" ::: "memory");
    if (lane == 0) __hip_atomic_fetch_add(lcnt, 1u, __ATOMIC_RELAXED, __HIP_MEMORY_SCOPE_WORKGROUP);
    if (w == 0) STAMP(16 + k);
  };
  bool inb = false;
  for (unsigned k = 0;; ++k) {
    const unsigned e = unit_at(k);
    pp = flags + e * 32;
    soff = e * OBJ;
    seen = flag_mask(nextf, call);
    acc = f32x16{0.f, 0.f, 0.f, 0.f, 0.f, 0.f, 0.f, 0.f, 0.f, 0.f, 0.f, 0.f, 0.f, 0.f, 0.f, 0.f};
    u32x4 xa[16];
#pragma unroll
    for (int r = 0; r < 2; ++r) {
      wait_round(pp, r, seen, bad, t0, tmo, call, lidle, k + 1);
      xload(xa, r);
      xmul(xa, r);
    }
    if (k + 1 >= nu) break;
#pragma unroll
    for (int r = 2; r < 4; ++r) {
      wait_round(pp, r, seen, bad, t0, tmo, call, lidle, k + 1);
      xload(xa, r);
      if (r == 3) { nextf = flag_load(flags + unit_at(k + 1) * 32); __builtin_amdgcn_sched_barrier(0); }
      xmul(xa, r);
    }
    if (INB && k + WCHK == nu) {
      const unsigned* const wp = (const unsigned*)(myws + WS_WIPE + 1024 + ((unit_at(nu - 1) >> 4) & 3) * 128) + (lane & 15);
      const unsigned wv = flag_load(wp);
      part(k);
      inb = __builtin_amdgcn_ballot_w64(wv < call) == 0;
      asm volatile("" ::: "memory");
    } else {
      part(k);
    }
  }
  {
    u32x4 xa[16], xb[16];
#pragma unroll
    for (int i = 0; i < 16; ++i) wr[i] = tr_frag(ltr + i * 1024);
#pragma unroll
    for (int i = 0; i < 16; ++i) asm volatile("" : "+v"(wr[i]));
    auto full = [&](u32x4 (&x)[16]) -> bool {
      unsigned m = ~0u;
#pragma unroll
      for (int i = 0; i < 16; ++i) m = min(min(min(min(m, x[i].x), x[i].y), x[i].z), x[i].w);
      return __builtin_amdgcn_ballot_w64(m == 0) == 0;
    };
    while (((seen >> 4) & 3u) != 3u) {
      const unsigned smp = __builtin_amdgcn_raw_buffer_load_b32(rsrc, w * 16384 + (lane & 1) * 8192 + SOFF, soff + 2 * 65536, 17);
      seen = flag_mask(flag_load(pp), call);
      if (inb && __builtin_amdgcn_ballot_w64(smp == 0) == 0) break;
      __hip_atomic_store(lidle, nu, __ATOMIC_RELAXED, __HIP_MEMORY_SCOPE_WORKGROUP);
      if (give_up(t0, tmo, call)) { bad = true; break; }
      if (SSL) __builtin_amdgcn_s_sleep(SSL);
    }
    asm volatile("" ::: "memory");
    STAMP(52 + w);
    xload(xa, 2);
    if (!inb) wait_round(pp, 3, seen, bad, t0, tmo, call, lidle, nu);
    xload(xb, 3);
    unsigned f = flag_load(pp);
    auto settle = [&](u32x4 (&x)[16], int r, int rw) {
      const f32x16 c0 = acc;
#pragma unroll
      for (int i = 0; i < 16; ++i) {
        acc = __builtin_amdgcn_mfma_f32_32x32x16_f16(wr[(r - 2) * 16 + i], __builtin_bit_cast(h8, x[i]), acc, 0, 0, 0);
        if (r == 2) wr[16 + i] = tr_frag(ltr + (16 + i) * 1024);
      }
      if (r == 2) {
#pragma unroll
        for (int i = 0; i < 16; ++i) asm volatile("" : "+v"(wr[16 + i]));
      }
      if (__builtin_expect(!(full(x) | (((seen >> (2 * r)) & 3u) == 3u)), 0)) {
        bool ok;
        do {
          ok = ((flag_mask(f, call) >> (2 * r)) & 3u) == 3u;
          if (give_up(t0, tmo, call)) bad = true;
          asm volatile("" ::: "memory");
          xload(x, r);
          f = flag_load(pp);
          acc = c0;
          xmul(x, rw);
          ok = full(x) | ok | bad;
        } while (!ok);
      }
    };
    settle(xa, 2, 2);
    settle(xb, 3, 1);
    STAMP(56 + w);
    part(nu - 1);
    STAMP(60 + w);
    while (rfl(__hip_atomic_load(lcnt, __ATOMIC_RELAXED, __HIP_MEMORY_SCOPE_WORKGROUP)) < nu * 4) {
      if (__builtin_amdgcn_s_memrealtime() - t0 > 2 * DEADLINE) { bad = true; break; }
    }
    asm volatile("" ::: "memory");
    const char* const pq = pl + w * 1024;
    f32x4 y = (*(const f32x4*)pq + *(const f32x4*)(pq + 4096)) + (*(const f32x4*)(pq + 8192) + *(const f32x4*)(pq + 12288));
    y.x = gelu(y.x); y.y = gelu(y.y); y.z = gelu(y.z); y.w = gelu(y.w);
    if (bad) y.x = __builtin_nanf("");
    asm volatile("global_store_dwordx4 %0, %1, %2 sc0 sc1\n\ts_nop 1"
                 :: "v"(ovoff + (unit_at(nu - 1) * 32 + w * 8) * (COLS * 4)), "v"(y), "s"(obase) : "memory");
    if (w == 0) STAMP(32 + nu - 1);
    if (foreign) {
      asm volatile("s_waitcnt vmcnt(0)" ::: "memory");
      if (lane == 0) __hip_atomic_store(donep + w, call, __ATOMIC_RELAXED, __HIP_MEMORY_SCOPE_SYSTEM);
    }
    if (w == 0) STAMP(2);
  }
}

unsigned link_map(int rank) {
  const unsigned dflt = 0x76543210u;
  int dev = -1, n = 0, node[NR], port[NR + 1];
  if (hipGetDevice(&dev) != hipSuccess || hipGetDeviceCount(&n) != hipSuccess || dev != rank || n != NR) return dflt;
  for (int i = 0; i <= NR; ++i) port[i] = -1;
  for (int d = 0; d < NR; ++d) {
    char id[32], path[96];
    if (hipDeviceGetPCIBusId(id, sizeof id, d) != hipSuccess) return dflt;
    for (char* c = id; *c; ++c) if (*c >= 'A' && *c <= 'F') *c += 'a' - 'A';
    snprintf(path, sizeof path, "/sys/bus/pci/devices/%s/xgmi_port_num", id);
    FILE* f = fopen(path, "r");
    if (!f) return dflt;
    unsigned a, pa, z, pz;
    node[d] = -1;
    while (fscanf(f, " %x:%x -> %x:%x", &a, &pa, &z, &pz) == 4) {
      node[d] = (int)a;
      if (d == rank && z <= (unsigned)NR && pa < 8u) port[z] = (int)pa;
    }
    fclose(f);
    if (node[d] < 0 || node[d] > NR) return dflt;
  }
  unsigned map = 0, used = 0;
  const int side = rank >> 2, q = rank & 3;
  for (int c = 0; c < 7; ++c) {
    const int x = port[node[c < 3 ? side * 4 + ((q + 1 + c) & 3) : (side ^ 1) * 4 + ((q + c - 3) & 3)]];
    if (x < 0 || ((used >> x) & 1)) return dflt;
    used |= 1u << x;
    map |= (unsigned)c << (4 * x);
  }
  return map | 7u << (4 * __builtin_ctz(~used));
}

}

extern "C" void kernel_launch(int rank, int nranks, ncclComm_t comm,
                              hipStream_t stream,
                              void* const* d_in, const int* in_sizes, int n_in,
                              void* d_out, int out_size,
                              const int* in_dtypes, int out_dtype,
                              void* const* d_out_all,
                              void* const* d_ws_all, size_t ws_bytes) {
  (void)comm; (void)in_dtypes; (void)out_dtype; (void)d_out;
  if (nranks != NR || (unsigned)rank >= (unsigned)NR || n_in < 2 || ws_bytes < WS_NEED ||
      in_sizes[0] != ROWS * KDIM || in_sizes[1] != KDIM * COLS || out_size != NR * ROWS * COLS) {
    fprintf(stderr, "kernel_launch: unexpected problem shape\n");
    return;
  }
  Params P;
  const int side = rank >> 2, q = rank & 3;
  for (int p = 0; p < NR; ++p) { P.ws[p] = (char*)d_ws_all[p]; P.out[p] = (char*)d_out_all[p]; }
  unsigned xmap = link_map(rank);
  for (int x = 0; x < 8; ++x) {
    const int c = (xmap >> (4 * x)) & 7;
    const int dest = c < 3 ? side * 4 + ((q + 1 + c) & 3) : (c < 7 ? (side ^ 1) * 4 + ((q + c - 3) & 3) : rank);
    P.head[x] = Head{(const char*)d_in[0], (const char*)d_in[1], P.ws[rank], P.ws[dest], rank, c};
  }
  static const unsigned char order[NG][12] = {
    {60, 72, 0, 6, 40, 12, 45, 50, 18, 55, 24, 30},
    {61, 73, 1, 7, 43, 13, 44, 49, 19, 54, 25, 31},
    {62, 2, 8, 42, 14, 47, 48, 20, 53, 26, 32, 99},
    {63, 3, 9, 41, 46, 15, 51, 21, 52, 27, 33, 99},
    {64, 65, 66, 67, 74, 4, 10, 16, 22, 28, 34, 99},
    {68, 69, 70, 71, 75, 5, 11, 17, 23, 29, 35, 99}};
  for (int g = 0; g < NG; ++g)
    for (int cls = 0; cls < 5; ++cls) {
      unsigned u[12] = {0}, n = 0;
      for (int k = 0; k < 12; ++k) {
        int v = order[g][k];
        if (v == 99) break;
        if (cls > 0 && (v >= 60 + NA || (v >= 40 && v < 60 && ((v - 40) & 3) == cls - 1))) continue;
        if (v < 40) u[n++] = (side * 4 + ((q + 1 + v % 3) & 3)) * 16 + v / 3;
        else if (v < 60) u[n++] = ((side ^ 1) * 4 + ((v - 40) & 3)) * 16 + NA + ((v - 40) >> 2);
        else u[n++] = rank * 16 + v - 60;
      }
      for (int k = 0; k < 3; ++k)
        P.units[g][cls][k] = u[4 * k] | (u[4 * k + 1] << 8) | (u[4 * k + 2] << 16) | (u[4 * k + 3] << 24);
      P.units[g][cls][3] = n;
    }
  hipLaunchKernelGGL(ag_gemm_gelu, dim3(256), dim3(384), 0, stream, P);
  HIP_CHECK(hipGetLastError());
}
```
